```python
import numpy as np

kernel_name = "gpu_gemm_ar_m4096_k4096_n4096_bf16_none"


ABSMAX_RTOL = 0.008

COMPUTE_DTYPE = "bfloat16"
EPILOGUE = "none"
IN_QUANT = None
OUT_QUANT = None
IN_DTYPES = {"x": "bfloat16", "w_mat": "bfloat16"}
WORLD_SIZE = 8

def _to_bf16(x):
    u = np.ascontiguousarray(x, dtype=np.float32).view(np.uint32)
    return ((u + 0x7FFF + ((u >> 16) & 1)) & 0xFFFF0000).view(np.float32)

def setup_inputs():

    x = _to_bf16(np.random.randn(4096, 4096).astype(np.float32))
    w_mat = _to_bf16((np.random.randn(4096, 4096) / np.sqrt(4096)).astype(np.float32))
    return {"x": x, "w_mat": w_mat}

def _epilogue(y):
    return y

def reference(x, w_mat):
    y = (x @ w_mat).astype(np.float32)
    return _epilogue(y)
```

```cpp
#include <hip/hip_runtime.h>
#include <rccl/rccl.h>
#include <cstdio>
#include <cstdint>

namespace {
typedef unsigned u32;
typedef unsigned long long u64;
typedef unsigned short u16;
typedef u32 u32x4 __attribute__((ext_vector_type(4)));
typedef u32 u32x2 __attribute__((ext_vector_type(2)));
typedef short bf16x8 __attribute__((ext_vector_type(8)));
typedef short s16x4 __attribute__((ext_vector_type(4)));
typedef u16 u16x2 __attribute__((ext_vector_type(2)));
typedef float f32x4 __attribute__((ext_vector_type(4)));
#define LAS __attribute__((address_space(3)))
#define GAS __attribute__((address_space(1)))

constexpr int NR = 8;
constexpr int NN = 4096, KS = 512;
constexpr int NT = 512, GRID = 256;
#define PULL_S 5
constexpr int S_PER_LINK = PULL_S;
constexpr int NPULL = 7 * S_PER_LINK;
constexpr u64 DEADLINE = 40000000ull;

constexpr u32 CH_BYTES = 3072, UNIT_HDR = 8 * CH_BYTES, UNIT_BYTES = UNIT_HDR + 256;
constexpr u32 FB_CH = 2048, FB_UNIT = 8 * FB_CH;
constexpr int IN_UNITS = 96, OUT_UNITS = 128, ALL_UNITS = IN_UNITS + OUT_UNITS;
constexpr int NCH = ALL_UNITS * 8;
constexpr int NWL = S_PER_LINK * 8;

constexpr size_t OFF_CTR = 0;
constexpr size_t OFF_TMO = 4096;
constexpr size_t OFF_TICK = 5120;
constexpr size_t OFF_CNT = 8192;
constexpr size_t OFF_FIN = OFF_CNT + 24 * 8 * 128;
constexpr size_t OFF_FOUT = OFF_FIN + (size_t)8 * 256 * 128;
constexpr size_t OFF_DBG = OFF_FOUT + (size_t)8 * 128 * 128;
constexpr size_t OFF_DBG2 = OFF_DBG + 65536;
constexpr size_t OFF_PIN = OFF_DBG2 + 16384;
constexpr size_t OFF_POUT = OFF_PIN + (size_t)256 * UNIT_BYTES;
constexpr size_t OFF_FBIN = OFF_POUT + (size_t)128 * UNIT_BYTES;
constexpr size_t OFF_FBOUT = OFF_FBIN + (size_t)256 * FB_UNIT;
constexpr size_t OFF_XF = OFF_FBOUT + (size_t)128 * FB_UNIT;
constexpr size_t OFF_WF = OFF_XF + ((size_t)16 << 20);
constexpr size_t WS_NEED = OFF_WF + ((size_t)8 << 20);
static_assert(OFF_PIN % 128 == 0 && OFF_XF % 1024 == 0, "alignment");

constexpr int LDS_BYTES = 131072 + 1024;
constexpr int LDS_MISC = 131072;
constexpr int WPITCH = 288;
constexpr int OPITCH = 272;

struct Params {
    const u16* x; const u16* w; u16* out;
    char* ws[NR];
    int rank; int pad;
};

typedef __amdgpu_buffer_rsrc_t rsrc_t;
__device__ __forceinline__ rsrc_t mk_rsrc(const void* p, u32 bytes) {
    return __builtin_amdgcn_make_buffer_rsrc((void*)p, 0, bytes, 0x00020000);
}
constexpr int AUX_SYS = 17, AUX_AGENT = 16;
#define XF_AUX AUX_AGENT
#define GL_AUX 17
#define OUT_AUX 16
__device__ __forceinline__ u64 rtc() { return __builtin_amdgcn_s_memrealtime(); }
__device__ __forceinline__ u32 rfl(u32 v) { return __builtin_amdgcn_readfirstlane(v); }
__device__ __forceinline__ u32 bfi(u32 m, u32 a, u32 b) { return (m & a) | (~m & b); }
__device__ __forceinline__ u32 perm(u32 a, u32 b, u32 s) { return __builtin_amdgcn_perm(a, b, s); }
__device__ __forceinline__ void drain() { asm volatile("s_waitcnt vmcnt(0)" ::: "memory"); }
__device__ __forceinline__ u32 ld_sys(const char* p) { return __hip_atomic_load((const GAS u32*)p, __ATOMIC_RELAXED, __HIP_MEMORY_SCOPE_SYSTEM); }
__device__ __forceinline__ u32 ld_agent(const char* p) { return __hip_atomic_load((const GAS u32*)p, __ATOMIC_RELAXED, __HIP_MEMORY_SCOPE_AGENT); }
__device__ __forceinline__ void st_sys(char* p, u32 v) { __hip_atomic_store((GAS u32*)p, v, __ATOMIC_RELAXED, __HIP_MEMORY_SCOPE_SYSTEM); }
__device__ __forceinline__ void st_agent(char* p, u32 v) { __hip_atomic_store((GAS u32*)p, v, __ATOMIC_RELAXED, __HIP_MEMORY_SCOPE_AGENT); }

#define STAMPC(ws, b, call, i) do {} while (0)
#define STAMP(ws, b, i) do {} while (0)
#define STAMPW(ws, b, i) do {} while (0)

template <bool SYS>
__device__ __forceinline__ bool wait_ge(const char* flag, u32 want, char* tmo, u32 call) {
    const u64 t0 = rtc();
    for (u32 n = 1;; ++n) {
        const u32 v = rfl(SYS ? ld_sys(flag) : ld_agent(flag));
        if (v >= want) return true;
        if ((n & 7) == 0 && (rfl(ld_agent(tmo)) == call || rtc() - t0 > DEADLINE)) { st_agent(tmo, call); return false; }
        __builtin_amdgcn_s_sleep(1);
    }
}
__device__ __forceinline__ bool wait_all16(const char* cnt_lane, u32 want, char* tmo, u32 call, int lane) {
    const u64 t0 = rtc();
    for (u32 n = 1;; ++n) {
        const u32 v = lane < 16 ? ld_agent(cnt_lane) : want;
        if (__all(v >= want)) return true;
        if ((n & 7) == 0 && (rfl(ld_agent(tmo)) == call || rtc() - t0 > DEADLINE)) { st_agent(tmo, call); return false; }
        __builtin_amdgcn_s_sleep(1);
    }
}

__device__ __forceinline__ u32 wave_max(u32 v) {
    v = max(v, (u32)__builtin_amdgcn_update_dpp(0, (int)v, 0xB1, 0xF, 0xF, true));
    v = max(v, (u32)__builtin_amdgcn_update_dpp(0, (int)v, 0x4E, 0xF, 0xF, true));
    v = max(v, (u32)__builtin_amdgcn_update_dpp(0, (int)v, 0x141, 0xF, 0xF, true));
    v = max(v, (u32)__builtin_amdgcn_update_dpp(0, (int)v, 0x140, 0xF, 0xF, true));
    const u32 a = __builtin_amdgcn_readlane(v, 0), b = __builtin_amdgcn_readlane(v, 16);
    const u32 c = __builtin_amdgcn_readlane(v, 32), d = __builtin_amdgcn_readlane(v, 48);
    return max(max(a, b), max(c, d));
}

__device__ __forceinline__ void pack_chunk(const u32 (&d)[16], rsrc_t unit, rsrc_t fb, int wv, int lane) {
    u32 L[8], H[8];
#pragma unroll
    for (int t = 0; t < 8; ++t) { L[t] = perm(d[2 * t + 1], d[2 * t], 0x06040200u); H[t] = perm(d[2 * t + 1], d[2 * t], 0x07050301u); }
    u16x2 mx = {0, 0};
#pragma unroll
    for (int i = 0; i < 16; ++i) mx = __builtin_elementwise_max(mx, __builtin_bit_cast(u16x2, d[i] & 0x7FFF7FFFu));
    const u32 base = wave_max(max((u32)mx.x, (u32)mx.y) >> 8);
    const u32 B4 = (base | 0x80u) * 0x01010101u;
    u32 dl[8], N[4], ov = 0;
#pragma unroll
    for (int t = 0; t < 8; ++t) { dl[t] = B4 - H[t]; ov |= dl[t]; }
#pragma unroll
    for (int u = 0; u < 4; ++u) {
        const u32 a = bfi(0x07070707u, dl[2 * u], dl[2 * u] >> 4);
        const u32 b = bfi(0x70707070u, dl[2 * u + 1] << 4, dl[2 * u + 1]);
        N[u] = bfi(0x0F0F0F0Fu, a, b);
    }
    const u32 po = (u32)wv * CH_BYTES + (u32)lane * 16;
    __builtin_amdgcn_raw_buffer_store_b128((u32x4){L[0], L[1], L[2], L[3]}, unit, po, 0, AUX_SYS);
    __builtin_amdgcn_raw_buffer_store_b128((u32x4){L[4], L[5], L[6], L[7]}, unit, po + 1024, 0, AUX_SYS);
    __builtin_amdgcn_raw_buffer_store_b128((u32x4){N[0], N[1], N[2], N[3]}, unit, po + 2048, 0, AUX_SYS);
    u32 hdr = 0, cnt = 0;
    if (__ballot((ov & 0x78787878u) != 0)) {
#pragma unroll
        for (int t = 0; t < 8; ++t)
#pragma unroll
            for (int b = 0; b < 4; ++b) {
                u64 m = __ballot(((dl[t] >> (8 * b)) & 0x78u) != 0);
                const u32 ent = (((u32)lane * 32 + t * 4 + b) << 8) | ((H[t] >> (8 * b)) & 0xFFu);
                while (m) {
                    const int src = __builtin_ctzll(m); m &= m - 1;
                    const u32 e = __builtin_amdgcn_readlane(ent, src);
                    ++cnt;
                    if ((u32)lane == cnt) hdr = e;
                }
            }
        if (cnt > 7) {
            cnt = 255;
            const u32 fo = (u32)wv * FB_CH + (u32)lane * 16;
            __builtin_amdgcn_raw_buffer_store_b128((u32x4){H[0], H[1], H[2], H[3]}, fb, fo, 0, AUX_SYS);
            __builtin_amdgcn_raw_buffer_store_b128((u32x4){H[4], H[5], H[6], H[7]}, fb, fo + 1024, 0, AUX_SYS);
        }
    }
    if (lane == 0) hdr = base | (cnt << 8);
    if (lane < 8) __builtin_amdgcn_raw_buffer_store_b32(hdr, unit, UNIT_HDR + (u32)wv * 32 + (u32)lane * 4, 0, AUX_SYS);
}

__device__ __forceinline__ void unpack_chunk(u32 (&d)[16], u32x4 l0, u32x4 l1, u32x4 n, u32 hdr, rsrc_t fb, int wv, int lane, bool ok) {
    const u32 h0 = __builtin_amdgcn_readlane(hdr, 0);
    const u32 cnt = ok ? (h0 >> 8) & 0xFFu : 0u;
    const u32 B4 = ((h0 & 0x7Fu) | 0x80u) * 0x01010101u;
    u32 H[8];
#pragma unroll
    for (int u = 0; u < 4; ++u) {
        H[2 * u] = (B4 - (n[u] & 0x07070707u)) ^ ((n[u] << 4) & 0x80808080u);
        H[2 * u + 1] = (B4 - ((n[u] >> 4) & 0x07070707u)) ^ (n[u] & 0x80808080u);
    }
    if (cnt) {
        if (cnt == 255) {
            const u32 fo = (u32)wv * FB_CH + (u32)lane * 16;
            u32x4 a = __builtin_amdgcn_raw_buffer_load_b128(fb, fo, 0, AUX_SYS);
            u32x4 b = __builtin_amdgcn_raw_buffer_load_b128(fb, fo + 1024, 0, AUX_SYS);
            asm volatile("" : "+v"(a), "+v"(b));
#pragma unroll
            for (int t = 0; t < 4; ++t) { H[t] = a[t]; H[4 + t] = b[t]; }
        } else {
            for (u32 e = 1; e <= cnt; ++e) {
                const u32 ent = __builtin_amdgcn_readlane(hdr, e);
                const u32 pos = ent >> 8, el = pos >> 5, et = (pos >> 2) & 7, eb = pos & 3;
                const u32 msk = ((u32)lane == el) ? (0xFFu << (8 * eb)) : 0u;
                const u32 val = (ent & 0xFFu) * 0x01010101u;
#pragma unroll
                for (int t = 0; t < 8; ++t) H[t] = bfi(et == (u32)t ? msk : 0u, val, H[t]);
            }
        }
    }
    const u32 L[8] = {l0[0], l0[1], l0[2], l0[3], l1[0], l1[1], l1[2], l1[3]};
#pragma unroll
    for (int t = 0; t < 8; ++t) { d[2 * t] = perm(H[t], L[t], 0x05010400u); d[2 * t + 1] = perm(H[t], L[t], 0x07030602u); }
    if (!ok) {
#pragma unroll
        for (int i = 0; i < 16; ++i) d[i] = 0x7FC07FC0u;
    }
}

__device__ __forceinline__ void tile_order(int m, int& rg, int& cg) {
    if (m < 32) { rg = m & 3; cg = m >> 2; }
    else { rg = 4 + ((m - 32) >> 3); cg = m & 7; }
}

__global__ __launch_bounds__(NT, 2) void fused_gemm_ar(Params p) {
    extern __shared__ __attribute__((aligned(16))) unsigned char smem[];
    LAS unsigned char* lds = (LAS unsigned char*)smem;
    const int tid = threadIdx.x, lane = tid & 63, wv = (int)rfl(tid >> 6);
    const int b = blockIdx.x, rank = p.rank, ri = rank >> 2, rj = rank & 3;
    char* ws = p.ws[rank];
    char* tmo = ws + OFF_TMO;
    STAMP(ws, b, 0);
    const u32 callv = ld_agent(ws + OFF_CTR + b * 4);

    {
        u32 d[16];
        const int isw = b >> 7, g = (b & 127) >> 2, q = b & 3;
        if (!isw) {
            const u16* src = p.x + (size_t)(g * 128 + wv * 16 + (lane & 15)) * KS + q * 128 + (lane >> 4) * 8;
#pragma unroll
            for (int f = 0; f < 4; ++f) {
                const u32x4 v = *(const GAS u32x4*)(src + f * 32);
                d[4 * f] = v[0]; d[4 * f + 1] = v[1]; d[4 * f + 2] = v[2]; d[4 * f + 3] = v[3];
            }
        } else {
            const u16* src = p.w + (size_t)(q * 128 + (tid >> 4)) * NN + g * 128 + (tid & 15) * 8;
            u32x4 v[4];
#pragma unroll
            for (int i = 0; i < 4; ++i) v[i] = *(const GAS u32x4*)(src + (size_t)i * 32 * NN);
#pragma unroll
            for (int i = 0; i < 4; ++i) *(LAS u32x4*)(lds + ((tid >> 4) + 32 * i) * WPITCH + (tid & 15) * 16) = v[i];
            __syncthreads();
            const int a0 = (8 * (lane >> 4) + ((lane & 15) >> 2)) * WPITCH + (wv * 16 + 4 * (lane & 3)) * 2;
#pragma unroll
            for (int f = 0; f < 4; ++f)
#pragma unroll
                for (int h = 0; h < 2; ++h) {
                    const s16x4 r = __builtin_amdgcn_ds_read_tr16_b64_v4i16((LAS s16x4*)(lds + a0 + (f * 32 + h * 4) * WPITCH));
                    const u32x2 rr = __builtin_bit_cast(u32x2, r);
                    d[4 * f + 2 * h] = rr[0]; d[4 * f + 2 * h + 1] = rr[1];
                }
        }
        pack_chunk(d, mk_rsrc(ws + OFF_PIN + (size_t)b * UNIT_BYTES, UNIT_BYTES), mk_rsrc(ws + OFF_FBIN + (size_t)b * FB_UNIT, FB_UNIT), wv, lane);
        const bool mine = isw ? ((g >> 3) == rj) : ((g >> 4) == ri);
        const int item = isw ? 16 + (g & 7) : (g & 15);
        if (mine) {
            const rsrc_t dst = mk_rsrc(ws + (isw ? OFF_WF + ((size_t)(g & 7) << 20) : OFF_XF + ((size_t)(g & 15) << 20)), 1u << 20);
#pragma unroll
            for (int f = 0; f < 4; ++f)
                __builtin_amdgcn_raw_buffer_store_b128((u32x4){d[4 * f], d[4 * f + 1], d[4 * f + 2], d[4 * f + 3]}, dst,
                                                       (u32)(((rank * 16 + q * 4 + f) * 8 + wv) * 1024 + lane * 16), 0, AUX_AGENT);
        }
        drain();
        __syncthreads();
        const u32 call0 = rfl(callv) + 1;
        if (tid < (isw ? 2 : 4)) {
            const u32 dest = isw ? (u32)(tid * 4 + (g >> 3)) : (u32)((g >> 4) * 4 + tid);
            if (dest < (u32)NR) {
                if (dest != (u32)rank) st_sys(p.ws[dest] + OFF_FIN + (size_t)(rank * 256 + b) * 128, call0);
                else __hip_atomic_fetch_add((GAS u32*)(ws + OFF_CNT + (item * 8 + rank) * 128), 8u, __ATOMIC_RELAXED, __HIP_MEMORY_SCOPE_AGENT);
            }
        }
        if (tid == 0) st_agent(ws + OFF_CTR + b * 4, call0);
    }
    const u32 call = rfl(callv) + 1;
    STAMP(ws, b, 1);
    const int xcd = b & 7, slot = b >> 3;

    if (slot < 16) {
        const int rg = (xcd >> 1) * 4 + (slot >> 2), cg = (xcd & 1) * 4 + (slot & 3);
        LAS u32* okw = (LAS u32*)(lds + LDS_MISC);
        if (wv == 0) {
            wait_all16(ws + OFF_CNT + (((lane & 8) ? 16 + cg : rg) * 8 + (lane & 7)) * 128, call * 32, tmo, call, lane);
        }
        asm volatile("" ::: "memory"); __builtin_amdgcn_s_barrier(); asm volatile("" ::: "memory");
        STAMP(ws, b, 2);
        f32x4 acc[4][4];
#pragma unroll
        for (int m = 0; m < 4; ++m)
#pragma unroll
            for (int n = 0; n < 4; ++n) acc[m][n] = (f32x4){0.f, 0.f, 0.f, 0.f};
#define G_BAR() do { __builtin_amdgcn_s_barrier(); asm volatile("" ::: "memory"); } while (0)
        const int wr = (wv >> 1) & 1, wc = wv & 1;
        if (wv >= 4) {
            const int lw = wv - 4;
            const char* xg = ws + OFF_XF + ((size_t)rg << 20) + lw * 4096 + lane * 16;
            const char* wg = ws + OFF_WF + ((size_t)cg << 20) + lw * 4096 + lane * 16;
            const int ldst = lw * 4096;
#define L_ISSUE(s, SL) do { const char* _x = xg + (size_t)(s) * 16384; const char* _w = wg + (size_t)(s) * 16384; \
            _Pragma("unroll") for (int i = 0; i < 4; ++i) __builtin_amdgcn_global_load_lds((const GAS u32*)(_x + i * 1024), (LAS u32*)(lds + (SL) * 32768 + ldst + i * 1024), 16, 0, GL_AUX); \
            _Pragma("unroll") for (int i = 0; i < 4; ++i) __builtin_amdgcn_global_load_lds((const GAS u32*)(_w + i * 1024), (LAS u32*)(lds + (SL) * 32768 + 16384 + ldst + i * 1024), 16, 0, GL_AUX); } while (0)
#define L_WAIT(n) asm volatile("s_waitcnt vmcnt(" #n ")" ::: "memory")
            L_ISSUE(0, 0); L_ISSUE(1, 1); L_ISSUE(2, 2); L_ISSUE(3, 3);
            L_WAIT(24); G_BAR();
            for (int s = 0; s < 60; s += 4) {
                L_WAIT(16); G_BAR(); L_ISSUE(s + 4, 0);
                L_WAIT(16); G_BAR(); L_ISSUE(s + 5, 1);
                L_WAIT(16); G_BAR(); L_ISSUE(s + 6, 2);
                L_WAIT(16); G_BAR(); L_ISSUE(s + 7, 3);
            }
            L_WAIT(16); G_BAR(); L_WAIT(8); G_BAR(); L_WAIT(0); G_BAR(); G_BAR();
        } else {
            const int xro = wr * 4096 + lane * 16, wro = 16384 + wc * 4096 + lane * 16;
            bf16x8 xa[2][2][4], wb[2][2][4];
#define C_READ(P, SL) do { _Pragma("unroll") for (int k = 0; k < 2; ++k) { \
            _Pragma("unroll") for (int n = 0; n < 4; ++n) wb[P][k][n] = *(const LAS bf16x8*)(lds + (SL) * 32768 + wro + k * 8192 + n * 1024); \
            _Pragma("unroll") for (int m = 0; m < 4; ++m) xa[P][k][m] = *(const LAS bf16x8*)(lds + (SL) * 32768 + xro + k * 8192 + m * 1024); } } while (0)
#define C_MMA(P) do { _Pragma("unroll") for (int k = 0; k < 2; ++k) _Pragma("unroll") for (int m = 0; m < 4; ++m) \
            _Pragma("unroll") for (int n = 0; n < 4; ++n) acc[m][n] = __builtin_amdgcn_mfma_f32_16x16x32_bf16(wb[P][k][n], xa[P][k][m], acc[m][n], 0, 0, 0); } while (0)
#define C_SCHED() do { _Pragma("unroll") for (int i = 0; i < 16; ++i) { __builtin_amdgcn_sched_group_barrier(0x008, 2, 0); __builtin_amdgcn_sched_group_barrier(0x100, 1, 0); } } while (0)
#define C_STEP(SL, P, RD) do { asm volatile("s_waitcnt lgkmcnt(0)" ::: "memory"); G_BAR(); if (RD) C_READ((P) ^ 1, ((SL) + 1) & 3); C_MMA(P); if (RD) C_SCHED(); __builtin_amdgcn_sched_barrier(0); } while (0)
            G_BAR();
            C_READ(0, 0);
            for (int s = 0; s < 60; s += 4) { C_STEP(0, 0, true); C_STEP(1, 1, true); C_STEP(2, 0, true); C_STEP(3, 1, true); }
            C_STEP(0, 0, true); C_STEP(1, 1, true); C_STEP(2, 0, true); C_STEP(3, 1, false);
        }
        G_BAR();
        STAMP(ws, b, 3);
        if (wv < 4) {
#pragma unroll
            for (int m = 0; m < 4; ++m)
#pragma unroll
                for (int n = 0; n < 4; ++n) {
                    u32x2 o;
                    asm("v_cvt_pk_bf16_f32 %0, %1, %2" : "=v"(o.x) : "v"(acc[m][n][0]), "v"(acc[m][n][1]));
                    asm("v_cvt_pk_bf16_f32 %0, %1, %2" : "=v"(o.y) : "v"(acc[m][n][2]), "v"(acc[m][n][3]));
                    *(LAS u32x2*)(lds + (wr * 64 + m * 16 + (lane & 15)) * OPITCH + (wc * 64 + n * 16 + (lane >> 4) * 4) * 2) = o;
                }
        }
        if (tid == 0) okw[0] = (ld_agent(tmo) == call) ? 1u : 0u;
        __syncthreads();
        u32 d[16];
#pragma unroll
        for (int f = 0; f < 4; ++f) {
            const u32x4 v = *(const LAS u32x4*)(lds + (wv * 16 + f * 4 + (lane >> 4)) * OPITCH + (lane & 15) * 16);
            d[4 * f] = v[0]; d[4 * f + 1] = v[1]; d[4 * f + 2] = v[2]; d[4 * f + 3] = v[3];
        }
        if (okw[0]) {
#pragma unroll
            for (int i = 0; i < 16; ++i) d[i] = 0x7FC07FC0u;
        }
        const int t = rg * 8 + cg;
        pack_chunk(d, mk_rsrc(ws + OFF_POUT + (size_t)t * UNIT_BYTES, UNIT_BYTES), mk_rsrc(ws + OFF_FBOUT + (size_t)t * FB_UNIT, FB_UNIT), wv, lane);
        drain();
        __syncthreads();
        if (tid < NR && tid != rank) st_sys(p.ws[tid] + OFF_FOUT + (size_t)(rank * 128 + t) * 128, call);
        {
            const rsrc_t outr = mk_rsrc(p.out, 4096u * NN * 2);
            const u32 oo = (u32)((ri * 2048 + rg * 128 + wv * 16 + (lane >> 4)) * NN + rj * 1024 + cg * 128 + (lane & 15) * 8) * 2;
#pragma unroll
            for (int f = 0; f < 4; ++f)
                __builtin_amdgcn_raw_buffer_store_b128((u32x4){d[4 * f], d[4 * f + 1], d[4 * f + 2], d[4 * f + 3]}, outr, oo + f * 4 * NN * 2, 0, OUT_AUX);
        }
        STAMP(ws, b, 4);
        STAMPC(ws, b, call, 1);
        return;
    }

    const int pi = (slot - 16) * 8 + xcd;
    if (pi >= NPULL) return;
    {
        const int link = pi % 7, wl = (pi / 7) * 8 + wv;
        const u32 src = (u32)(rank + 1 + link) & 7u;
        if (src >= (u32)NR) return;
        const char* pws = p.ws[src];
        const int si = src >> 2, sj = src & 3;
        GAS u32* tick = (GAS u32*)(ws + OFF_TICK + link * 128);
        const u32 tbase = 3 * NWL - (call - 1) * (u32)(NCH - 2 * NWL);
        auto describe = [&](int u, int& uid, int& rg, int& cg, int& isw) -> const char* {
            rg = 0; cg = 0; isw = 0;
            if (u < IN_UNITS) {
                const int e = u >> 2, q = u & 3;
                isw = (e >= 4) & (e < 12);
                if (isw) { cg = e - 4; uid = 128 + (rj * 8 + cg) * 4 + q; }
                else { rg = e < 4 ? e : (e - 8); uid = (ri * 16 + rg) * 4 + q; }
                return ws + OFF_FIN + (size_t)(src * 256 + uid) * 128;
            }
            tile_order(u - IN_UNITS, rg, cg);
            uid = rg * 8 + cg;
            return ws + OFF_FOUT + (size_t)(src * 128 + uid) * 128;
        };
        const rsrc_t outr = mk_rsrc(p.out, 4096u * NN * 2);
        int cur = wl, n1 = NWL + wl, n2 = 2 * NWL + wl;
        int uid, rg, cg, isw;
        u32x4 l0 = {}, l1 = {}, nn = {}; u32 hdr = 0;
#define P_LOAD(c, uid_) do { const rsrc_t un_ = mk_rsrc(pws + (((c) >> 3) < IN_UNITS ? OFF_PIN : OFF_POUT) + (size_t)(uid_) * UNIT_BYTES, UNIT_BYTES); \
            const u32 po_ = (u32)((c) & 7) * CH_BYTES + (u32)lane * 16; \
            l0 = __builtin_amdgcn_raw_buffer_load_b128(un_, po_, 0, AUX_SYS); \
            l1 = __builtin_amdgcn_raw_buffer_load_b128(un_, po_ + 1024, 0, AUX_SYS); \
            nn = __builtin_amdgcn_raw_buffer_load_b128(un_, po_ + 2048, 0, AUX_SYS); \
            hdr = __builtin_amdgcn_raw_buffer_load_b32(un_, UNIT_HDR + (u32)((c) & 7) * 32 + (u32)(lane & 7) * 4, 0, AUX_SYS); } while (0)
        bool ok = wait_ge<true>(describe(cur >> 3, uid, rg, cg, isw), call, tmo, call);
        asm volatile("" ::: "memory");
        P_LOAD(cur, uid);
        u32 fl;
        { int a_, b_, c_, d_; fl = ld_sys(describe(n1 >> 3, a_, b_, c_, d_)); }
        for (;;) {
            drain();
            asm volatile("" : "+v"(l0), "+v"(l1), "+v"(nn), "+v"(hdr), "+v"(fl));
            const u32 flv1 = rfl(fl);
            u32 tk;
            {
                u64 sv; const u32 em = rfl(n2 < NCH ? 1u : 0u);
                asm volatile("s_mov_b64 %1, exec\n\ts_mov_b32 exec_lo, %4\n\ts_mov_b32 exec_hi, 0\n\tglobal_atomic_add %0, %2, %3, off sc0\n\ts_mov_b64 exec, %1"
                             : "=&v"(tk), "=&s"(sv) : "v"(tick), "v"(1u), "s"(em) : "memory");
            }
            const int u = cur >> 3, v = cur & 7;
            const bool isin = u < IN_UNITS;
            const int q = u & 3;
            u32 d[16];
            unpack_chunk(d, l0, l1, nn, hdr, mk_rsrc(pws + (isin ? OFF_FBIN : OFF_FBOUT) + (size_t)uid * FB_UNIT, FB_UNIT), v, lane, ok);
            if (isin) {
                const rsrc_t dst = mk_rsrc(ws + (isw ? OFF_WF + ((size_t)cg << 20) : OFF_XF + ((size_t)rg << 20)), 1u << 20);
#pragma unroll
                for (int f = 0; f < 4; ++f)
                    __builtin_amdgcn_raw_buffer_store_b128((u32x4){d[4 * f], d[4 * f + 1], d[4 * f + 2], d[4 * f + 3]}, dst,
                                                           (u32)(((src * 16 + q * 4 + f) * 8 + v) * 1024 + lane * 16), 0, XF_AUX);
            } else {
                const u32 oo = (u32)((si * 2048 + rg * 128 + v * 16 + (lane >> 4)) * NN + sj * 1024 + cg * 128 + (lane & 15) * 8) * 2;
#pragma unroll
                for (int f = 0; f < 4; ++f)
                    __builtin_amdgcn_raw_buffer_store_b128((u32x4){d[4 * f], d[4 * f + 1], d[4 * f + 2], d[4 * f + 3]}, outr, oo + f * 4 * NN * 2, 0, OUT_AUX);
            }
            GAS u32* cntp = (GAS u32*)(ws + OFF_CNT + ((isw ? 16 + cg : rg) * 8 + src) * 128);
            u32 sigm = isin ? 1u : 0u;
            if (n1 >= NCH) {
                if (isin) { drain(); if (lane == 0) __hip_atomic_fetch_add(cntp, 1u, __ATOMIC_RELAXED, __HIP_MEMORY_SCOPE_AGENT); }
                break;
            }
            const char* flag1 = describe(n1 >> 3, uid, rg, cg, isw);
            ok = true;
            if (flv1 < call) {
                drain();
                if (sigm && lane == 0) __hip_atomic_fetch_add(cntp, 1u, __ATOMIC_RELAXED, __HIP_MEMORY_SCOPE_AGENT);
                sigm = 0;
                ok = wait_ge<true>(flag1, call, tmo, call);
            }
            asm volatile("" ::: "memory");
            P_LOAD(n1, uid);
            { int a_, b_, c_, d_; fl = ld_sys(describe((n2 < NCH ? n2 : n1) >> 3, a_, b_, c_, d_)); }
            {
                u64 sv; const u32 sm = rfl(sigm);
                asm volatile("s_waitcnt vmcnt(5)\n\ts_mov_b64 %0, exec\n\ts_mov_b32 exec_lo, %3\n\ts_mov_b32 exec_hi, 0\n\tglobal_atomic_add %1, %2, off\n\ts_mov_b64 exec, %0"
                             : "=&s"(sv) : "v"(cntp), "v"(1u), "s"(sm) : "memory");
            }
            asm volatile("" : "+v"(tk));
            const int n3 = n2 < NCH ? (int)(rfl(tk) + tbase) : NCH;
            cur = n1; n1 = n2; n2 = n3;
        }
        STAMPW(ws, b, 4);
        drain();
        STAMPC(ws, b, call, 1);
    }
}
}

#define HIP_CHECK(expr) do { hipError_t _e = (expr); if (_e != hipSuccess) { \
    fprintf(stderr, "HIP error %s at %s:%d\n", hipGetErrorString(_e), __FILE__, __LINE__); \
    return; } } while (0)

extern "C" void kernel_launch(int rank, int nranks, ncclComm_t comm,
                              hipStream_t stream,
                              void* const* d_in, const int* in_sizes, int n_in,
                              void* d_out, int out_size,
                              const int* in_dtypes, int out_dtype,
                              void* const* d_out_all,
                              void* const* d_ws_all, size_t ws_bytes) {
    (void)comm; (void)in_sizes; (void)n_in; (void)out_size; (void)in_dtypes; (void)out_dtype; (void)d_out_all;
    if (nranks != NR || ws_bytes < WS_NEED || rank < 0 || rank >= NR) { fprintf(stderr, "unexpected configuration\n"); return; }
    static bool attr_done[64] = {};
    int dev = 0;
    HIP_CHECK(hipGetDevice(&dev));
    if (dev >= 0 && dev < 64 && !attr_done[dev]) {
        HIP_CHECK(hipFuncSetAttribute((const void*)fused_gemm_ar, hipFuncAttributeMaxDynamicSharedMemorySize, LDS_BYTES));
        attr_done[dev] = true;
    }
    Params p;
    p.x = (const u16*)d_in[0];
    p.w = (const u16*)d_in[1];
    p.out = (u16*)d_out;
    for (int i = 0; i < NR; ++i) p.ws[i] = (char*)d_ws_all[i];
    p.rank = rank; p.pad = 0;
    hipLaunchKernelGGL(fused_gemm_ar, dim3(GRID), dim3(NT), LDS_BYTES, stream, p);
    HIP_CHECK(hipGetLastError());
}
```

```cpp
#include <hip/hip_runtime.h>
#include <rccl/rccl.h>
#include <cstdio>
#include <cstdint>

namespace {
typedef unsigned u32;
typedef unsigned long long u64;
typedef unsigned short u16;
typedef u32 u32x4 __attribute__((ext_vector_type(4)));
typedef u32 u32x2 __attribute__((ext_vector_type(2)));
typedef u32 u32x3 __attribute__((ext_vector_type(3)));
typedef short bf16x8 __attribute__((ext_vector_type(8)));
typedef short s16x4 __attribute__((ext_vector_type(4)));
typedef u16 u16x2 __attribute__((ext_vector_type(2)));
typedef float f32x4 __attribute__((ext_vector_type(4)));
#define LAS __attribute__((address_space(3)))
#define GAS __attribute__((address_space(1)))

constexpr int NR = 8;
constexpr int NN = 4096, KS = 512;
constexpr int NT = 512, GRID = 256;
#define PULL_W 31
constexpr int NWL = PULL_W;
#define PULL_SLOTS 4
constexpr int GS = 32 - PULL_SLOTS;
static_assert(GS % 2 == 0 && GS >= 16, "rows");
constexpr int NPULL = 8 * PULL_SLOTS;
static_assert(7 * NWL <= NPULL * 8, "puller waves");
constexpr u64 DEADLINE = 40000000ull;

constexpr u32 CH_BYTES = 2944, LST_OFF = 2816, UNIT_BYTES = 8 * CH_BYTES;
constexpr u32 LST_CAP = 63;
constexpr u32 FB_CH = 2048, FB_UNIT = 8 * FB_CH;
constexpr int IN_UNITS = 96, OUT_UNITS = 128, ALL_UNITS = IN_UNITS + OUT_UNITS;
constexpr int NCH = ALL_UNITS * 8;

constexpr size_t OFF_CTR = 0;
constexpr size_t OFF_TMO = 4096;
constexpr size_t OFF_TICK = 5120;
constexpr size_t OFF_CNT = 8192;
constexpr size_t OFF_FIN = OFF_CNT + 24 * 8 * 128;
constexpr size_t OFF_FOUT = OFF_FIN + (size_t)8 * 256 * 128;
constexpr size_t OFF_DBG = OFF_FOUT + (size_t)8 * 256 * 128;
constexpr size_t OFF_DBG2 = OFF_DBG + 65536;
constexpr size_t OFF_DBG3 = OFF_DBG2 + 16384;
constexpr size_t OFF_PIN = OFF_DBG3 + 384 * 512;
constexpr size_t OFF_POUT = OFF_PIN + (size_t)256 * UNIT_BYTES;
constexpr size_t OFF_FBIN = OFF_POUT + (size_t)128 * UNIT_BYTES;
constexpr size_t OFF_FBOUT = OFF_FBIN + (size_t)256 * FB_UNIT;
constexpr size_t OFF_XF = OFF_FBOUT + (size_t)128 * FB_UNIT;
constexpr size_t OFF_WF = OFF_XF + ((size_t)16 << 20);
constexpr size_t WS_NEED = OFF_WF + ((size_t)8 << 20);
static_assert(OFF_PIN % 128 == 0 && OFF_XF % 1024 == 0, "alignment");

constexpr int LDS_BYTES = 131072 + 1024;
constexpr int LDS_MISC = 131072;
constexpr int GSLOT = 24576;
constexpr int WPITCH = 288;
constexpr int OPITCH = 144;
constexpr int SCR_OFF = 65536, SCR_WAVE = 2304;

struct Params {
    const u16* x; const u16* w; u16* out;
    char* ws[NR];
    int rank; int pad;
};

typedef __amdgpu_buffer_rsrc_t rsrc_t;
__device__ __forceinline__ rsrc_t mk_rsrc(const void* p, u32 bytes) {
    return __builtin_amdgcn_make_buffer_rsrc((void*)p, 0, bytes, 0x00020000);
}
constexpr int AUX_SYS = 17, AUX_AGENT = 16;
#define GATE0 1000
#define GATE1 150
#define POLL_SLEEP 8
#define GL_AUX 17
#define OUT_AUX 16
__device__ __forceinline__ u64 rtc() { return __builtin_amdgcn_s_memrealtime(); }
__device__ __forceinline__ u32 rfl(u32 v) { return __builtin_amdgcn_readfirstlane(v); }
__device__ __forceinline__ u64 uni64(u64 v) { return ((u64)rfl((u32)(v >> 32)) << 32) | rfl((u32)v); }
__device__ __forceinline__ u32 bfi(u32 m, u32 a, u32 b) { return (m & a) | (~m & b); }
__device__ __forceinline__ u32 perm(u32 a, u32 b, u32 s) { return __builtin_amdgcn_perm(a, b, s); }
__device__ __forceinline__ void drain() { asm volatile("s_waitcnt vmcnt(0)" ::: "memory"); }
__device__ __forceinline__ u32 ld_sys(const char* p) { return __hip_atomic_load((const GAS u32*)p, __ATOMIC_RELAXED, __HIP_MEMORY_SCOPE_SYSTEM); }
__device__ __forceinline__ u32 ld_agent(const char* p) { return __hip_atomic_load((const GAS u32*)p, __ATOMIC_RELAXED, __HIP_MEMORY_SCOPE_AGENT); }
__device__ __forceinline__ void st_sys(char* p, u32 v) { __hip_atomic_store((GAS u32*)p, v, __ATOMIC_RELAXED, __HIP_MEMORY_SCOPE_SYSTEM); }
__device__ __forceinline__ void st_agent(char* p, u32 v) { __hip_atomic_store((GAS u32*)p, v, __ATOMIC_RELAXED, __HIP_MEMORY_SCOPE_AGENT); }

#define STAMPC(ws, b, call, i) do {} while (0)
#define STAMP(ws, b, i) do {} while (0)
#define STAMPW(ws, b, i) do {} while (0)

template <bool SYS>
__device__ __forceinline__ bool wait_ge(const char* flag, u32 want, char* tmo, u32 call) {
    const u64 t0 = rtc();
    for (u32 n = 1;; ++n) {
        const u32 v = rfl(SYS ? ld_sys(flag) : ld_agent(flag));
        if (v >= want) return true;
        if ((n & 7) == 0 && (rfl(ld_agent(tmo)) == call || rtc() - t0 > DEADLINE)) { st_agent(tmo, call); return false; }
        __builtin_amdgcn_s_sleep(1);
    }
}
__device__ __forceinline__ bool wait_all16(const char* cnt_lane, u32 want, char* tmo, u32 call, int lane) {
    const u64 t0 = rtc();
    for (u32 n = 1;; ++n) {
        const u32 v = lane < 16 ? ld_agent(cnt_lane) : want;
        if (__all(v >= want)) return true;
        if ((n & 7) == 0 && (rfl(ld_agent(tmo)) == call || rtc() - t0 > DEADLINE)) { st_agent(tmo, call); return false; }
        __builtin_amdgcn_s_sleep(POLL_SLEEP);
    }
}

#define DPP_ADD(v, ctrl, rm) v += (u32)__builtin_amdgcn_update_dpp(0, (int)(v), ctrl, rm, 0xF, true)
__device__ __forceinline__ u32 wave_sum(u32 v) {
    DPP_ADD(v, 0xB1, 0xF); DPP_ADD(v, 0x4E, 0xF); DPP_ADD(v, 0x141, 0xF); DPP_ADD(v, 0x140, 0xF);
    return (__builtin_amdgcn_readlane(v, 0) + __builtin_amdgcn_readlane(v, 16)) + (__builtin_amdgcn_readlane(v, 32) + __builtin_amdgcn_readlane(v, 48));
}
__device__ __forceinline__ u32 wave_scan(u32 v) {
    DPP_ADD(v, 0x111, 0xF); DPP_ADD(v, 0x112, 0xF); DPP_ADD(v, 0x114, 0xF); DPP_ADD(v, 0x118, 0xF);
    DPP_ADD(v, 0x142, 0xA); DPP_ADD(v, 0x143, 0xC);
    return v;
}
__device__ __forceinline__ void pack_chunk(const u32 (&d)[16], rsrc_t unit, rsrc_t fb, int v, int lane, LAS unsigned char* scr) {
    u32 SM[8], E[8], sum = 0;
#pragma unroll
    for (int t = 0; t < 8; ++t) {
        const u32 lo = perm(d[2 * t + 1], d[2 * t], 0x06040200u), hi = perm(d[2 * t + 1], d[2 * t], 0x07050301u);
        SM[t] = bfi(0x7F7F7F7Fu, lo, hi);
        E[t] = bfi(0xFEFEFEFEu, hi << 1, lo >> 7);
        sum = __builtin_amdgcn_udot4(E[t], 0x01010101u, sum, false);
    }
    const u32 po = (u32)v * CH_BYTES;
    __builtin_amdgcn_raw_buffer_store_b128((u32x4){SM[0], SM[1], SM[2], SM[3]}, unit, po + lane * 16, 0, AUX_SYS);
    __builtin_amdgcn_raw_buffer_store_b128((u32x4){SM[4], SM[5], SM[6], SM[7]}, unit, po + 1024 + lane * 16, 0, AUX_SYS);
    const u32 base = min((wave_sum(sum) + 5755u) >> 11, 255u);
    const u16x2 b2 = {(u16)base, (u16)base}, eight = {8, 8};
    u32 P0 = 0, P1 = 0, P2 = 0, M = 0;
#pragma unroll
    for (int t = 7; t >= 0; --t) {
        const u16x2 ev = __builtin_bit_cast(u16x2, E[t] & 0x00FF00FFu), od = __builtin_bit_cast(u16x2, perm(E[t], E[t], 0x0C030C01u));
        const u32 ca = __builtin_bit_cast(u32, __builtin_elementwise_min((u16x2)(b2 - ev), eight));
        const u32 cb = __builtin_bit_cast(u32, __builtin_elementwise_min((u16x2)(b2 - od), eight));
        const u32 c = ca | (cb << 8);
        P0 = (P0 << 1) | (c & 0x01010101u);
        P1 = (P1 << 1) | ((c >> 1) & 0x01010101u);
        P2 = (P2 << 1) | ((c >> 2) & 0x01010101u);
        M = (M << 1) | ((c >> 3) & 0x01010101u);
    }
    __builtin_amdgcn_raw_buffer_store_b96((u32x3){P0, P1, P2}, unit, po + 2048 + lane * 12, 0, AUX_SYS);
    const u32 n = __builtin_popcount(M);
    const u32 incl = wave_scan(n);
    u32 cnt = __builtin_amdgcn_readlane(incl, 63), lst = 0;
    bool raw = cnt > LST_CAP;
    if (!raw && cnt) {
        *(LAS u32x4*)(scr + lane * 32) = (u32x4){E[0], E[1], E[2], E[3]};
        *(LAS u32x4*)(scr + lane * 32 + 16) = (u32x4){E[4], E[5], E[6], E[7]};
        u32 idx = incl - n + 1;
        while (M) {
            const u32 bit = __builtin_ctz(M); M &= M - 1;
            ((LAS u16*)(scr + 2048))[idx & 63] = (u16)(lane * 32 + (bit & 7) * 4 + (bit >> 3));
            ++idx;
        }
        asm volatile("" ::: "memory");
        bool bad = false;
        if (lane < 32) {
            const u32 pp = *(const LAS u32*)(scr + 2048 + lane * 4);
#pragma unroll
            for (int k = 0; k < 2; ++k) {
                const u32 pos = (pp >> (16 * k)) & 2047u, e = scr[pos];
                const int dl = (int)base - (int)e;
                const u32 c5 = e == 0 ? 15u : (dl > 0 ? (u32)(dl - 8) : (u32)(15 - dl));
                if ((u32)(lane * 2 + k - 1) < cnt) { bad |= e != 0 && (dl > 22 || dl < -16); lst |= ((pos << 5) | (c5 & 31u)) << (16 * k); }
            }
        }
        raw = __ballot(bad) != 0;
    }
    if (raw) {
        cnt = 255;
        const u32 fo = (u32)v * FB_CH + (u32)lane * 16;
        __builtin_amdgcn_raw_buffer_store_b128((u32x4){E[0], E[1], E[2], E[3]}, fb, fo, 0, AUX_SYS);
        __builtin_amdgcn_raw_buffer_store_b128((u32x4){E[4], E[5], E[6], E[7]}, fb, fo + 1024, 0, AUX_SYS);
    }
    if (lane == 0) lst = (lst & 0xFFFF0000u) | base | (cnt << 8);
    if (lane < 32) __builtin_amdgcn_raw_buffer_store_b32(lst, unit, po + LST_OFF + (u32)lane * 4, 0, AUX_SYS);
}

__device__ __forceinline__ void unpack_chunk(u32 (&d)[16], u32x4 s0, u32x4 s1, u32x3 cp, u32 lst, rsrc_t fb, u32 fbo, int lane, bool ok,
                                             LAS unsigned char* scr) {
    const u32 h0 = __builtin_amdgcn_readlane(lst, 0);
    const u32 cnt = ok ? (h0 >> 8) & 0xFFu : 0u, base = h0 & 0xFFu;
    const u32 B4 = base * 0x01010101u;
    const u32 lut_lo = B4 - 0x03020100u, lut_hi = B4 - 0x07060504u;
    u32 E[8];
#pragma unroll
    for (int t = 0; t < 8; ++t) {
        const u32 c = ((cp[0] >> t) & 0x01010101u) | (((cp[1] >> t) & 0x01010101u) << 1) | (((cp[2] >> t) & 0x01010101u) << 2);
        E[t] = perm(lut_hi, lut_lo, c);
    }
    if (cnt) {
        if (cnt == 255) {
            u32x4 a = __builtin_amdgcn_raw_buffer_load_b128(fb, lane * 16, fbo, AUX_SYS);
            u32x4 b = __builtin_amdgcn_raw_buffer_load_b128(fb, lane * 16 + 1024, fbo, AUX_SYS);
            asm volatile("" : "+v"(a), "+v"(b));
#pragma unroll
            for (int t = 0; t < 4; ++t) { E[t] = a[t]; E[4 + t] = b[t]; }
        } else {
            *(LAS u32x4*)(scr + lane * 32) = (u32x4){E[0], E[1], E[2], E[3]};
            *(LAS u32x4*)(scr + lane * 32 + 16) = (u32x4){E[4], E[5], E[6], E[7]};
            asm volatile("" ::: "memory");
#pragma unroll
            for (int k = 0; k < 2; ++k) {
                const u32 ent = (lst >> (16 * k)) & 0xFFFFu, c5 = ent & 31u;
                const u32 e = c5 < 16 ? (c5 == 15 ? 0u : base - 8 - c5) : base + c5 - 15;
                if (lane < 32 && (u32)(lane * 2 + k - 1) < cnt) scr[ent >> 5] = (unsigned char)e;
            }
            asm volatile("" ::: "memory");
            const u32x4 a = *(const LAS u32x4*)(scr + lane * 32), b = *(const LAS u32x4*)(scr + lane * 32 + 16);
#pragma unroll
            for (int t = 0; t < 4; ++t) { E[t] = a[t]; E[4 + t] = b[t]; }
        }
    }
#pragma unroll
    for (int t = 0; t < 8; ++t) {
        const u32 sm = t < 4 ? s0[t & 3] : s1[t & 3];
        const u32 lo = bfi(0x7F7F7F7Fu, sm, E[t] << 7), hi = bfi(0x80808080u, sm, E[t] >> 1);
        d[2 * t] = perm(hi, lo, 0x05010400u); d[2 * t + 1] = perm(hi, lo, 0x07030602u);
    }
    if (!ok) {
#pragma unroll
        for (int i = 0; i < 16; ++i) d[i] = 0x7FC07FC0u;
    }
}

__host__ __device__ __forceinline__ void tile_order(u32 m, u32& rg, u32& cg) {
    rg = m < 32 ? (m & 3) : (m >> 3);
    cg = m < 32 ? (m >> 2) : (m & 7);
}
__host__ __device__ __forceinline__ void half_tile(int h, int& rg, int& c64) {
    if (h < 64) { rg = h & 3; c64 = h >> 2; }
    else if (h < 192) { const int k = (h - 64) >> 5, i = h & 31; rg = 4 + (k >> 1) * 4 + (i >> 3); c64 = (k & 1) * 8 + (i & 7); }
    else { rg = 12 + ((h - 192) >> 4); c64 = h & 15; }
}

__global__ __launch_bounds__(NT, 2) void fused_gemm_ar(Params p) {
    extern __shared__ __attribute__((aligned(16))) unsigned char smem[];
    LAS unsigned char* lds = (LAS unsigned char*)smem;
    const int tid = threadIdx.x, lane = tid & 63, wv = (int)rfl(tid >> 6);
    const int b = blockIdx.x, rank = p.rank, ri = rank >> 2, rj = rank & 3;
    char* ws = p.ws[rank];
    char* tmo = ws + OFF_TMO;
    STAMP(ws, b, 0);
    const u64 t_start = rtc();
    const u32 callv = ld_agent(ws + OFF_CTR + b * 4);

    {
        u32 d[16];
        const int isw = b >> 7, g = (b & 127) >> 2, q = b & 3;
        if (!isw) {
            const u16* src = p.x + (size_t)(g * 128 + wv * 16 + (lane & 15)) * KS + q * 128 + (lane >> 4) * 8;
#pragma unroll
            for (int f = 0; f < 4; ++f) {
                const u32x4 v = *(const GAS u32x4*)(src + f * 32);
                d[4 * f] = v[0]; d[4 * f + 1] = v[1]; d[4 * f + 2] = v[2]; d[4 * f + 3] = v[3];
            }
        } else {
            const u16* src = p.w + (size_t)(q * 128 + (tid >> 4)) * NN + g * 128 + (tid & 15) * 8;
            u32x4 v[4];
#pragma unroll
            for (int i = 0; i < 4; ++i) v[i] = *(const GAS u32x4*)(src + (size_t)i * 32 * NN);
#pragma unroll
            for (int i = 0; i < 4; ++i) *(LAS u32x4*)(lds + ((tid >> 4) + 32 * i) * WPITCH + (tid & 15) * 16) = v[i];
            __syncthreads();
            const int a0 = (8 * (lane >> 4) + ((lane & 15) >> 2)) * WPITCH + (wv * 16 + 4 * (lane & 3)) * 2;
#pragma unroll
            for (int f = 0; f < 4; ++f)
#pragma unroll
                for (int h = 0; h < 2; ++h) {
                    const s16x4 r = __builtin_amdgcn_ds_read_tr16_b64_v4i16((LAS s16x4*)(lds + a0 + (f * 32 + h * 4) * WPITCH));
                    const u32x2 rr = __builtin_bit_cast(u32x2, r);
                    d[4 * f + 2 * h] = rr[0]; d[4 * f + 2 * h + 1] = rr[1];
                }
        }
        pack_chunk(d, mk_rsrc(ws + OFF_PIN + (size_t)b * UNIT_BYTES, UNIT_BYTES), mk_rsrc(ws + OFF_FBIN + (size_t)b * FB_UNIT, FB_UNIT), wv, lane, lds + SCR_OFF + wv * SCR_WAVE);
        const bool mine = isw ? ((g >> 3) == rj) : ((g >> 4) == ri);
        const int item = isw ? 16 + (g & 7) : (g & 15);
        if (mine) {
            const rsrc_t dst = mk_rsrc(ws + (isw ? OFF_WF + ((size_t)(g & 7) << 20) : OFF_XF + ((size_t)(g & 15) << 20)), 1u << 20);
#pragma unroll
            for (int f = 0; f < 4; ++f)
                __builtin_amdgcn_raw_buffer_store_b128((u32x4){d[4 * f], d[4 * f + 1], d[4 * f + 2], d[4 * f + 3]}, dst,
                                                       (u32)(((rank * 16 + q * 4 + f) * 8 + wv) * 1024 + lane * 16), 0, AUX_AGENT);
        }
        drain();
        __syncthreads();
        const u32 call0 = rfl(callv) + 1;
        if (tid < (isw ? 2 : 4)) {
            const u32 dest = isw ? (u32)(tid * 4 + (g >> 3)) : (u32)((g >> 4) * 4 + tid);
            if (dest < (u32)NR) {
                if (dest != (u32)rank) st_sys(p.ws[dest] + OFF_FIN + (size_t)(rank * 256 + b) * 128, call0);
                else __hip_atomic_fetch_add((GAS u32*)(ws + OFF_CNT + (item * 8 + rank) * 128), 8u, __ATOMIC_RELAXED, __HIP_MEMORY_SCOPE_AGENT);
            }
        }
        if (tid == 0) st_agent(ws + OFF_CTR + b * 4, call0);
    }
    const u32 call = rfl(callv) + 1;
    STAMP(ws, b, 1);
    const int xcd = b & 7, slot = b >> 3;

    if (slot < GS) {
        const int h0 = xcd * GS + slot;
        LAS u32* okw = (LAS u32*)(lds + LDS_MISC);
#define G_BAR() do { __builtin_amdgcn_s_barrier(); asm volatile("" ::: "memory"); } while (0)
        for (int round = 0; round < (h0 < 256 - 8 * GS ? 2 : 1); ++round) {
            int rg, c64;
            half_tile(h0 + round * 8 * GS, rg, c64);
            const int cg = c64 >> 1, hn = c64 & 1;
            if (wv == 0) {
                const u32 e = max(rg < 4 ? rg : rg + 8, 4 + cg);
                while (rtc() - t_start < (u64)(GATE0 + GATE1 * e)) __builtin_amdgcn_s_sleep(8);
                wait_all16(ws + OFF_CNT + (((lane & 8) ? 16 + cg : rg) * 8 + (lane & 7)) * 128, call * 32, tmo, call, lane);
            }
            asm volatile("" ::: "memory"); __builtin_amdgcn_s_barrier(); asm volatile("" ::: "memory");
            STAMP(ws, b, 2 + 3 * round);
            f32x4 acc[4][2];
#pragma unroll
            for (int m = 0; m < 4; ++m) { acc[m][0] = (f32x4){0.f, 0.f, 0.f, 0.f}; acc[m][1] = (f32x4){0.f, 0.f, 0.f, 0.f}; }
            const int wr = (wv >> 1) & 1, wc = wv & 1;
            if (wv >= 4) {
                const int lw = wv - 4;
                const char* xg = ws + OFF_XF + ((size_t)rg << 20) + lw * 4096 + lane * 16;
                const char* wg = ws + OFF_WF + ((size_t)cg << 20) + (lw >> 1) * 8192 + hn * 4096 + (lw & 1) * 2048 + lane * 16;
#define L_ISSUE(s, SL) do { const char* _x = xg + (size_t)(s) * 16384; const char* _w = wg + (size_t)(s) * 16384; \
                _Pragma("unroll") for (int i = 0; i < 4; ++i) __builtin_amdgcn_global_load_lds((const GAS u32*)(_x + i * 1024), (LAS u32*)(lds + (SL) * GSLOT + lw * 4096 + i * 1024), 16, 0, GL_AUX); \
                _Pragma("unroll") for (int i = 0; i < 2; ++i) __builtin_amdgcn_global_load_lds((const GAS u32*)(_w + i * 1024), (LAS u32*)(lds + (SL) * GSLOT + 16384 + lw * 2048 + i * 1024), 16, 0, GL_AUX); } while (0)
#define L_WAIT(n) asm volatile("s_waitcnt vmcnt(" #n ")" ::: "memory")
                L_ISSUE(0, 0); L_ISSUE(1, 1); L_ISSUE(2, 2); L_ISSUE(3, 3);
                L_WAIT(18); G_BAR();
                for (int s = 0; s < 60; s += 4) {
                    L_WAIT(12); G_BAR(); L_ISSUE(s + 4, 0);
                    L_WAIT(12); G_BAR(); L_ISSUE(s + 5, 1);
                    L_WAIT(12); G_BAR(); L_ISSUE(s + 6, 2);
                    L_WAIT(12); G_BAR(); L_ISSUE(s + 7, 3);
                }
                L_WAIT(12); G_BAR(); L_WAIT(6); G_BAR(); L_WAIT(0); G_BAR(); G_BAR();
            } else {
                const int xro = wr * 4096 + lane * 16, wro = 16384 + wc * 2048 + lane * 16;
                bf16x8 xa[2][2][4], wb[2][2][2];
#define C_READ(P, SL) do { _Pragma("unroll") for (int k = 0; k < 2; ++k) { \
                _Pragma("unroll") for (int n = 0; n < 2; ++n) wb[P][k][n] = *(const LAS bf16x8*)(lds + (SL) * GSLOT + wro + k * 4096 + n * 1024); \
                _Pragma("unroll") for (int m = 0; m < 4; ++m) xa[P][k][m] = *(const LAS bf16x8*)(lds + (SL) * GSLOT + xro + k * 8192 + m * 1024); } } while (0)
#define C_MMA(P) do { _Pragma("unroll") for (int k = 0; k < 2; ++k) _Pragma("unroll") for (int m = 0; m < 4; ++m) \
                _Pragma("unroll") for (int n = 0; n < 2; ++n) acc[m][n] = __builtin_amdgcn_mfma_f32_16x16x32_bf16(wb[P][k][n], xa[P][k][m], acc[m][n], 0, 0, 0); } while (0)
#define C_SCHED() do { _Pragma("unroll") for (int i = 0; i < 12; ++i) { __builtin_amdgcn_sched_group_barrier(0x008, 1, 0); __builtin_amdgcn_sched_group_barrier(0x100, 1, 0); } \
                __builtin_amdgcn_sched_group_barrier(0x008, 4, 0); } while (0)
#define C_STEP(SL, P, RD) do { asm volatile("s_waitcnt lgkmcnt(0)" ::: "memory"); G_BAR(); if (RD) C_READ((P) ^ 1, ((SL) + 1) & 3); C_MMA(P); if (RD) C_SCHED(); __builtin_amdgcn_sched_barrier(0); } while (0)
                G_BAR();
                C_READ(0, 0);
                for (int s = 0; s < 60; s += 4) { C_STEP(0, 0, true); C_STEP(1, 1, true); C_STEP(2, 0, true); C_STEP(3, 1, true); }
                C_STEP(0, 0, true); C_STEP(1, 1, true); C_STEP(2, 0, true); C_STEP(3, 1, false);
            }
            G_BAR();
            STAMP(ws, b, 3 + 3 * round);
            if (wv < 4) {
#pragma unroll
                for (int m = 0; m < 4; ++m)
#pragma unroll
                    for (int n = 0; n < 2; ++n) {
                        u32x2 o;
                        asm("v_cvt_pk_bf16_f32 %0, %1, %2" : "=v"(o.x) : "v"(acc[m][n][0]), "v"(acc[m][n][1]));
                        asm("v_cvt_pk_bf16_f32 %0, %1, %2" : "=v"(o.y) : "v"(acc[m][n][2]), "v"(acc[m][n][3]));
                        *(LAS u32x2*)(lds + (wr * 64 + m * 16 + (lane & 15)) * OPITCH + (wc * 32 + n * 16 + (lane >> 4) * 4) * 2) = o;
                    }
            }
            if (tid == 0) okw[0] = (ld_agent(tmo) == call) ? 1u : 0u;
            __syncthreads();
            u32 d[16];
            const int j = wv & 3;
#pragma unroll
            for (int f = 0; f < 4; ++f) {
                const u32x4 v = *(const LAS u32x4*)(lds + (j * 32 + f * 8 + (lane >> 3)) * OPITCH + (lane & 7) * 16);
                d[4 * f] = v[0]; d[4 * f + 1] = v[1]; d[4 * f + 2] = v[2]; d[4 * f + 3] = v[3];
            }
            if (okw[0]) {
#pragma unroll
                for (int i = 0; i < 16; ++i) d[i] = 0x7FC07FC0u;
            }
            const int t = rg * 8 + cg;
            if (wv < 4) {
                pack_chunk(d, mk_rsrc(ws + OFF_POUT + (size_t)t * UNIT_BYTES, UNIT_BYTES), mk_rsrc(ws + OFF_FBOUT + (size_t)t * FB_UNIT, FB_UNIT), hn * 4 + j, lane,
                           lds + SCR_OFF + wv * SCR_WAVE);
                drain();
            } else {
                const rsrc_t outr = mk_rsrc(p.out, 4096u * NN * 2);
                const u32 oo = (u32)((ri * 2048 + rg * 128 + j * 32 + (lane >> 3)) * NN + rj * 1024 + c64 * 64 + (lane & 7) * 8) * 2;
#pragma unroll
                for (int f = 0; f < 4; ++f)
                    __builtin_amdgcn_raw_buffer_store_b128((u32x4){d[4 * f], d[4 * f + 1], d[4 * f + 2], d[4 * f + 3]}, outr, oo + f * 8 * NN * 2, 0, OUT_AUX);
            }
            __syncthreads();
            if (tid < NR && tid != rank) st_sys(p.ws[tid] + OFF_FOUT + (size_t)(rank * 256 + t * 2 + hn) * 128, call);
            STAMP(ws, b, 4 + 3 * round);
        }
        STAMPC(ws, b, call, 1);
        return;
    }

    const int pi = (slot - GS) * 8 + xcd;
    {
        const int gw = pi * 8 + wv;
        if (gw >= 7 * NWL) return;
        const int link = gw % 7, wl = gw / 7;
        const u32 src = rfl((u32)(rank + 1 + link) & 7u);
        if (src >= (u32)NR) return;
        const u64 pws = uni64((u64)p.ws[src]), wsu = uni64((u64)ws);
        LAS unsigned char* scr = lds + wv * SCR_WAVE;
        GAS u32* tick = (GAS u32*)(ws + OFF_TICK + link * 128);
        const u32 tbase = 3 * NWL - (call - 1) * (u32)(NCH - 2 * NWL);
        struct Desc { u32 rem, fb, flg, dst, cnt; };
        const u32 xbase = ri * 64, wbase = 128 + rj * 32;
        const u32 obase = ((src >> 2) * 2048 * NN + (src & 3) * 1024) * 2;
        auto mk = [&](u32 c) -> Desc {
            c = min(c, (u32)NCH - 1);
            const u32 u = c >> 3, v = c & 7, e = u >> 2, q = u & 3, m = u - IN_UNITS;
            const bool in = u < IN_UNITS, w = (e - 4u) < 8u;
            const u32 pan = w ? e - 4 : (e < 4 ? e : e - 8);
            const u32 uidi = (w ? wbase : xbase) + pan * 4 + q;
            u32 rgo, cgo;
            tile_order(m, rgo, cgo);
            const u32 uid = in ? uidi : rgo * 8 + cgo;
            Desc d;
            d.rem = (in ? (u32)OFF_PIN : (u32)OFF_POUT) + uid * UNIT_BYTES + v * CH_BYTES;
            d.fb = (in ? (u32)OFF_FBIN : (u32)OFF_FBOUT) + (uid * 8 + v) * FB_CH;
            d.flg = in ? (u32)OFF_FIN + (src * 256 + uid) * 128 : (u32)OFF_FOUT + (src * 256 + uid * 2 + (v >> 2)) * 128;
            d.dst = in ? (w ? (u32)OFF_WF : (u32)OFF_XF) + (pan << 20) + (src * 128 + q * 32 + v) * 1024
                       : obase + ((rgo * 128 + (v & 3) * 32) * NN + cgo * 128 + (v >> 2) * 64) * 2;
            d.cnt = in ? (u32)OFF_CNT + (((w ? 16 : 0) + pan) * 8 + src) * 128 : 0u;
            return d;
        };
        const rsrc_t wsr = mk_rsrc(ws, (u32)WS_NEED), outr = mk_rsrc(p.out, 4096u * NN * 2), pwr = mk_rsrc((const void*)pws, (u32)WS_NEED);
        const u32 vin = lane * 16, vout = (lane >> 3) * (NN * 2) + (lane & 7) * 16, vl4 = LST_OFF + (lane & 31) * 4;
        u32 n1 = NWL + wl, n2 = 2 * NWL + wl;
        Desc D0 = mk(wl), D1 = mk(n1), D2 = mk(n2);
#define P_CLOB "a0", "a1", "a2", "a3", "a4", "a5", "a6", "a7", "a8", "a9", "a10", "a11"
#define P_ISSUE(DA) do { \
            const u64 un_ = pws + (DA).rem; \
            asm volatile("s_nop 4\n\tglobal_load_dwordx4 a[0:3], %0, %2 sc0 sc1\n\tglobal_load_dwordx4 a[4:7], %0, %2 offset:1024 sc0 sc1\n\t" \
                         "global_load_dwordx3 a[8:10], %1, %2 offset:2048 sc0 sc1\n\tglobal_load_dword a11, %3, %2 sc0 sc1" \
                         :: "v"(vin), "v"(lane * 12), "s"(un_), "v"(vl4) : P_CLOB, "memory"); } while (0)
#define P_FLAG(DB) do { const u64 fa_ = wsu + (DB).flg; \
            asm volatile("s_nop 4\n\tglobal_load_dword a12, %0, %1 sc0 sc1" :: "v"(0), "s"(fa_) : "a12", "memory"); } while (0)
#define P_TICKET(want_) do { u64 sv_; const u32 em_ = rfl((want_) ? 1u : 0u); \
            asm volatile("v_accvgpr_write_b32 a14, 1\n\ts_mov_b64 %0, exec\n\ts_mov_b32 exec_lo, %2\n\ts_mov_b32 exec_hi, 0\n\ts_nop 1\n\t" \
                         "global_atomic_add a13, %1, a14, off sc0\n\ts_mov_b64 exec, %0" \
                         : "=&s"(sv_) : "v"(tick), "s"(em_) : "a13", "a14", "memory"); } while (0)
#define P_FETCH() asm volatile("v_accvgpr_read_b32 %0, a0\n\tv_accvgpr_read_b32 %1, a1\n\tv_accvgpr_read_b32 %2, a2\n\tv_accvgpr_read_b32 %3, a3\n\t" \
                         "v_accvgpr_read_b32 %4, a4\n\tv_accvgpr_read_b32 %5, a5\n\tv_accvgpr_read_b32 %6, a6\n\tv_accvgpr_read_b32 %7, a7\n\t" \
                         "v_accvgpr_read_b32 %8, a8\n\tv_accvgpr_read_b32 %9, a9\n\tv_accvgpr_read_b32 %10, a10\n\tv_accvgpr_read_b32 %11, a11\n\t" \
                         "v_accvgpr_read_b32 %12, a12\n\tv_accvgpr_read_b32 %13, a13" \
                         : "=v"(c0.x), "=v"(c0.y), "=v"(c0.z), "=v"(c0.w), "=v"(c1.x), "=v"(c1.y), "=v"(c1.z), "=v"(c1.w), \
                           "=v"(cc.x), "=v"(cc.y), "=v"(cc.z), "=v"(cl), "=v"(fl), "=v"(tk) :: "memory")
#define P_SIGNAL(c_) do { u64 sv_; const u64 ca_ = wsu + (c_); \
            asm volatile("s_nop 4\n\ts_mov_b64 %0, exec\n\ts_mov_b64 exec, 1\n\tglobal_atomic_add %1, %2, %3\n\ts_mov_b64 exec, %0" \
                         : "=&s"(sv_) : "v"(0), "v"(1u), "s"(ca_) : "memory"); } while (0)
        bool ok = wait_ge<true>(ws + D0.flg, call, tmo, call);
        asm volatile("" ::: "memory");
        P_ISSUE(D0); P_FLAG(D1);
        bool tkv = false;
        drain();
        u32 pendA = 0, pendB = 0;
#define TS() do {} while (0)
        for (;;) {
            asm volatile("s_waitcnt vmcnt(4)" ::: "memory");
            TS();
            u32x4 c0, c1; u32x3 cc; u32 cl, fl, tk;
            P_FETCH();
            const bool cok = ok;
            const bool more = n1 < (u32)NCH;
            const bool early = more && rfl(fl) >= call;
            if (early) { ok = true; asm volatile("" ::: "memory"); P_ISSUE(D1); }
            if (tkv) { n2 = rfl(tk) + tbase; D2 = mk(n2); }
            if (early) { P_FLAG(D2); tkv = n2 < (u32)NCH; P_TICKET(tkv); }
            if (pendA) P_SIGNAL(pendA);
            pendA = pendB;
            u32 d[16];
            unpack_chunk(d, c0, c1, cc, cl, pwr, D0.fb, lane, cok, scr);
            {
                const bool isin = D0.cnt != 0;
                const rsrc_t dr = isin ? wsr : outr;
                const u32 vo = isin ? vin : vout, step = isin ? 8192u : 8u * NN * 2;
#pragma unroll
                for (int f = 0; f < 4; ++f)
                    __builtin_amdgcn_raw_buffer_store_b128((u32x4){d[4 * f], d[4 * f + 1], d[4 * f + 2], d[4 * f + 3]}, dr, vo, D0.dst + f * step, AUX_AGENT);
            }
            u32 mine = D0.cnt;
            if (!early) {
                if (pendA | mine) { drain(); if (pendA) P_SIGNAL(pendA); if (mine) P_SIGNAL(mine); }
                pendA = 0; mine = 0;
                if (!more) break;
                ok = wait_ge<true>(ws + D1.flg, call, tmo, call);
                asm volatile("" ::: "memory");
                P_ISSUE(D1); P_FLAG(D2); tkv = n2 < (u32)NCH; P_TICKET(tkv);
                drain();
            }
            pendB = mine;
            n1 = n2; if (!tkv) n2 = NCH;
            D0 = D1; D1 = D2;
        }
        STAMPW(ws, b, 4);
        drain();
        STAMPC(ws, b, call, 1);
    }
}
}

#define HIP_CHECK(expr) do { hipError_t _e = (expr); if (_e != hipSuccess) { \
    fprintf(stderr, "HIP error %s at %s:%d\n", hipGetErrorString(_e), __FILE__, __LINE__); \
    return; } } while (0)

extern "C" void kernel_launch(int rank, int nranks, ncclComm_t comm,
                              hipStream_t stream,
                              void* const* d_in, const int* in_sizes, int n_in,
                              void* d_out, int out_size,
                              const int* in_dtypes, int out_dtype,
                              void* const* d_out_all,
                              void* const* d_ws_all, size_t ws_bytes) {
    (void)comm; (void)in_sizes; (void)n_in; (void)out_size; (void)in_dtypes; (void)out_dtype; (void)d_out_all;
    if (nranks != NR || ws_bytes < WS_NEED || rank < 0 || rank >= NR) { fprintf(stderr, "unexpected configuration\n"); return; }
    static bool attr_done[64] = {};
    int dev = 0;
    HIP_CHECK(hipGetDevice(&dev));
    if (dev >= 0 && dev < 64 && !attr_done[dev]) {
        HIP_CHECK(hipFuncSetAttribute((const void*)fused_gemm_ar, hipFuncAttributeMaxDynamicSharedMemorySize, LDS_BYTES));
        attr_done[dev] = true;
    }
    Params p;
    p.x = (const u16*)d_in[0];
    p.w = (const u16*)d_in[1];
    p.out = (u16*)d_out;
    for (int i = 0; i < NR; ++i) p.ws[i] = (char*)d_ws_all[i];
    p.rank = rank; p.pad = 0;
    hipLaunchKernelGGL(fused_gemm_ar, dim3(GRID), dim3(NT), LDS_BYTES, stream, p);
    HIP_CHECK(hipGetLastError());
}
```

```cpp
#include <hip/hip_runtime.h>
#include <rccl/rccl.h>
#include <cstdio>
#include <cstdint>

namespace {
typedef unsigned u32;
typedef unsigned long long u64;
typedef unsigned short u16;
typedef u32 u32x4 __attribute__((ext_vector_type(4)));
typedef u32 u32x2 __attribute__((ext_vector_type(2)));
typedef u32 u32x3 __attribute__((ext_vector_type(3)));
typedef short bf16x8 __attribute__((ext_vector_type(8)));
typedef short s16x4 __attribute__((ext_vector_type(4)));
typedef u16 u16x2 __attribute__((ext_vector_type(2)));
typedef float f32x4 __attribute__((ext_vector_type(4)));
#define LAS __attribute__((address_space(3)))
#define GAS __attribute__((address_space(1)))

constexpr int NR = 8;
constexpr int NN = 4096, KS = 512;
constexpr int NT = 512, GRID = 256;
#define PULL_W 31
constexpr int NWL = PULL_W;
#define PULL_SLOTS 4
constexpr int GS = 32 - PULL_SLOTS;
static_assert(GS % 2 == 0 && GS >= 16, "rows");
constexpr int NPULL = 8 * PULL_SLOTS;
static_assert(7 * NWL <= NPULL * 8, "puller waves");
constexpr u64 DEADLINE = 40000000ull;

constexpr u32 CH_BYTES = 2944, LST_OFF = 2816, UNIT_BYTES = 8 * CH_BYTES;
constexpr u32 LST_CAP = 63;
constexpr u32 FB_CH = 2048, FB_UNIT = 8 * FB_CH;
constexpr int IN_UNITS = 96, OUT_UNITS = 128, ALL_UNITS = IN_UNITS + OUT_UNITS;
constexpr int NCH = ALL_UNITS * 8;
constexpr int NPUSH = 64;

constexpr size_t OFF_CTR = 0;
constexpr size_t OFF_TMO = 4096;
constexpr size_t OFF_TICK = 5120;
constexpr size_t OFF_CNT = 8192;
constexpr size_t OFF_FIN = OFF_CNT + 24 * 8 * 128;
constexpr size_t OFF_FOUT = OFF_FIN + (size_t)8 * 256 * 128;
constexpr size_t OFF_DBG = OFF_FOUT + (size_t)8 * 256 * 128;
constexpr size_t OFF_DBG2 = OFF_DBG + 65536;
constexpr size_t OFF_DBG3 = OFF_DBG2 + 16384;
constexpr size_t OFF_PIN = OFF_DBG3 + 384 * 512;
constexpr size_t OFF_PUSH = OFF_PIN + (size_t)256 * UNIT_BYTES;
constexpr size_t OFF_POUT = OFF_PUSH + (size_t)8 * NPUSH * CH_BYTES;
constexpr size_t OFF_FBIN = OFF_POUT + (size_t)128 * UNIT_BYTES;
constexpr size_t OFF_FBOUT = OFF_FBIN + (size_t)256 * FB_UNIT;
constexpr size_t OFF_XF = OFF_FBOUT + (size_t)128 * FB_UNIT;
constexpr size_t OFF_WF = OFF_XF + ((size_t)16 << 20);
constexpr size_t WS_NEED = OFF_WF + ((size_t)8 << 20);
static_assert(OFF_PIN % 128 == 0 && OFF_XF % 1024 == 0, "alignment");

constexpr int LDS_BYTES = 131072 + 1024;
constexpr int LDS_MISC = 131072;
constexpr int GSLOT = 24576;
constexpr int WPITCH = 288;
constexpr int OPITCH = 144;
constexpr int SCR_OFF = 65536, SCR_WAVE = 2304;

struct Params {
    const u16* x; const u16* w; u16* out;
    char* ws[NR];
    int rank; int pad;
};

typedef __amdgpu_buffer_rsrc_t rsrc_t;
__device__ __forceinline__ rsrc_t mk_rsrc(const void* p, u32 bytes) {
    return __builtin_amdgcn_make_buffer_rsrc((void*)p, 0, bytes, 0x00020000);
}
constexpr int AUX_SYS = 17, AUX_AGENT = 16;
#define GATE0 1000
#define GATE1 150
#define POLL_SLEEP 8
#define GL_AUX 17
#define OUT_AUX 16
__device__ __forceinline__ u64 rtc() { return __builtin_amdgcn_s_memrealtime(); }
__device__ __forceinline__ u32 rfl(u32 v) { return __builtin_amdgcn_readfirstlane(v); }
__device__ __forceinline__ u64 uni64(u64 v) { return ((u64)rfl((u32)(v >> 32)) << 32) | rfl((u32)v); }
__device__ __forceinline__ u32 bfi(u32 m, u32 a, u32 b) { return (m & a) | (~m & b); }
__device__ __forceinline__ u32 perm(u32 a, u32 b, u32 s) { return __builtin_amdgcn_perm(a, b, s); }
__device__ __forceinline__ void drain() { asm volatile("s_waitcnt vmcnt(0)" ::: "memory"); }
__device__ __forceinline__ u32 ld_sys(const char* p) { return __hip_atomic_load((const GAS u32*)p, __ATOMIC_RELAXED, __HIP_MEMORY_SCOPE_SYSTEM); }
__device__ __forceinline__ u32 ld_agent(const char* p) { return __hip_atomic_load((const GAS u32*)p, __ATOMIC_RELAXED, __HIP_MEMORY_SCOPE_AGENT); }
__device__ __forceinline__ void st_sys(char* p, u32 v) { __hip_atomic_store((GAS u32*)p, v, __ATOMIC_RELAXED, __HIP_MEMORY_SCOPE_SYSTEM); }
__device__ __forceinline__ void st_agent(char* p, u32 v) { __hip_atomic_store((GAS u32*)p, v, __ATOMIC_RELAXED, __HIP_MEMORY_SCOPE_AGENT); }

#define STAMPC(ws, b, call, i) do {} while (0)
#define STAMP(ws, b, i) do {} while (0)
#define STAMPW(ws, b, i) do {} while (0)

template <bool SYS>
__device__ __forceinline__ bool wait_ge(const char* flag, u32 want, char* tmo, u32 call) {
    const u64 t0 = rtc();
    for (u32 n = 1;; ++n) {
        const u32 v = rfl(SYS ? ld_sys(flag) : ld_agent(flag));
        if (v >= want) return true;
        if ((n & 7) == 0 && (rfl(ld_agent(tmo)) == call || rtc() - t0 > DEADLINE)) { st_agent(tmo, call); return false; }
        __builtin_amdgcn_s_sleep(1);
    }
}
__device__ __forceinline__ bool wait_all16(const char* cnt_lane, u32 want, char* tmo, u32 call, int lane) {
    const u64 t0 = rtc();
    for (u32 n = 1;; ++n) {
        const u32 v = lane < 16 ? ld_agent(cnt_lane) : want;
        if (__all(v >= want)) return true;
        if ((n & 7) == 0 && (rfl(ld_agent(tmo)) == call || rtc() - t0 > DEADLINE)) { st_agent(tmo, call); return false; }
        __builtin_amdgcn_s_sleep(POLL_SLEEP);
    }
}

#define DPP_ADD(v, ctrl, rm) v += (u32)__builtin_amdgcn_update_dpp(0, (int)(v), ctrl, rm, 0xF, true)
__device__ __forceinline__ u32 wave_sum(u32 v) {
    DPP_ADD(v, 0xB1, 0xF); DPP_ADD(v, 0x4E, 0xF); DPP_ADD(v, 0x141, 0xF); DPP_ADD(v, 0x140, 0xF);
    return (__builtin_amdgcn_readlane(v, 0) + __builtin_amdgcn_readlane(v, 16)) + (__builtin_amdgcn_readlane(v, 32) + __builtin_amdgcn_readlane(v, 48));
}
__device__ __forceinline__ u32 wave_scan(u32 v) {
    DPP_ADD(v, 0x111, 0xF); DPP_ADD(v, 0x112, 0xF); DPP_ADD(v, 0x114, 0xF); DPP_ADD(v, 0x118, 0xF);
    DPP_ADD(v, 0x142, 0xA); DPP_ADD(v, 0x143, 0xC);
    return v;
}
struct Packed { u32x4 s0, s1; u32x3 cp; u32 lst; };
__device__ __forceinline__ void store_packed(const Packed& k, rsrc_t dst, u32 off, int lane) {
    __builtin_amdgcn_raw_buffer_store_b128(k.s0, dst, lane * 16, off, AUX_SYS);
    __builtin_amdgcn_raw_buffer_store_b128(k.s1, dst, lane * 16 + 1024, off, AUX_SYS);
    __builtin_amdgcn_raw_buffer_store_b96(k.cp, dst, lane * 12 + 2048, off, AUX_SYS);
    if (lane < 32) __builtin_amdgcn_raw_buffer_store_b32(k.lst, dst, lane * 4 + LST_OFF, off, AUX_SYS);
}
__device__ __forceinline__ Packed pack_chunk(const u32 (&d)[16], rsrc_t fb, u32 fbo, int lane, LAS unsigned char* scr) {
    u32 SM[8], E[8], sum = 0;
#pragma unroll
    for (int t = 0; t < 8; ++t) {
        const u32 lo = perm(d[2 * t + 1], d[2 * t], 0x06040200u), hi = perm(d[2 * t + 1], d[2 * t], 0x07050301u);
        SM[t] = bfi(0x7F7F7F7Fu, lo, hi);
        E[t] = bfi(0xFEFEFEFEu, hi << 1, lo >> 7);
        sum = __builtin_amdgcn_udot4(E[t], 0x01010101u, sum, false);
    }
    const u32 base = min((wave_sum(sum) + 5755u) >> 11, 255u);
    const u16x2 b2 = {(u16)base, (u16)base}, eight = {8, 8};
    u32 P0 = 0, P1 = 0, P2 = 0, M = 0;
#pragma unroll
    for (int t = 7; t >= 0; --t) {
        const u16x2 ev = __builtin_bit_cast(u16x2, E[t] & 0x00FF00FFu), od = __builtin_bit_cast(u16x2, perm(E[t], E[t], 0x0C030C01u));
        const u32 ca = __builtin_bit_cast(u32, __builtin_elementwise_min((u16x2)(b2 - ev), eight));
        const u32 cb = __builtin_bit_cast(u32, __builtin_elementwise_min((u16x2)(b2 - od), eight));
        const u32 c = ca | (cb << 8);
        P0 = (P0 << 1) | (c & 0x01010101u);
        P1 = (P1 << 1) | ((c >> 1) & 0x01010101u);
        P2 = (P2 << 1) | ((c >> 2) & 0x01010101u);
        M = (M << 1) | ((c >> 3) & 0x01010101u);
    }
    const u32 n = __builtin_popcount(M);
    const u32 incl = wave_scan(n);
    u32 cnt = __builtin_amdgcn_readlane(incl, 63), lst = 0;
    bool raw = cnt > LST_CAP;
    if (!raw && cnt) {
        *(LAS u32x4*)(scr + lane * 32) = (u32x4){E[0], E[1], E[2], E[3]};
        *(LAS u32x4*)(scr + lane * 32 + 16) = (u32x4){E[4], E[5], E[6], E[7]};
        u32 idx = incl - n + 1;
        while (M) {
            const u32 bit = __builtin_ctz(M); M &= M - 1;
            ((LAS u16*)(scr + 2048))[idx & 63] = (u16)(lane * 32 + (bit & 7) * 4 + (bit >> 3));
            ++idx;
        }
        asm volatile("" ::: "memory");
        bool bad = false;
        if (lane < 32) {
            const u32 pp = *(const LAS u32*)(scr + 2048 + lane * 4);
#pragma unroll
            for (int k = 0; k < 2; ++k) {
                const u32 pos = (pp >> (16 * k)) & 2047u, e = scr[pos];
                const int dl = (int)base - (int)e;
                const u32 c5 = e == 0 ? 15u : (dl > 0 ? (u32)(dl - 8) : (u32)(15 - dl));
                if ((u32)(lane * 2 + k - 1) < cnt) { bad |= e != 0 && (dl > 22 || dl < -16); lst |= ((pos << 5) | (c5 & 31u)) << (16 * k); }
            }
        }
        raw = __ballot(bad) != 0;
    }
    if (raw) {
        cnt = 255;
        __builtin_amdgcn_raw_buffer_store_b128((u32x4){E[0], E[1], E[2], E[3]}, fb, lane * 16, fbo, AUX_SYS);
        __builtin_amdgcn_raw_buffer_store_b128((u32x4){E[4], E[5], E[6], E[7]}, fb, lane * 16 + 1024, fbo, AUX_SYS);
    }
    if (lane == 0) lst = (lst & 0xFFFF0000u) | base | (cnt << 8);
    return Packed{(u32x4){SM[0], SM[1], SM[2], SM[3]}, (u32x4){SM[4], SM[5], SM[6], SM[7]}, (u32x3){P0, P1, P2}, lst};
}

__device__ __forceinline__ void unpack_chunk(u32 (&d)[16], u32x4 s0, u32x4 s1, u32x3 cp, u32 lst, rsrc_t fb, u32 fbo, int lane, bool ok,
                                             LAS unsigned char* scr) {
    const u32 h0 = __builtin_amdgcn_readlane(lst, 0);
    const u32 cnt = ok ? (h0 >> 8) & 0xFFu : 0u, base = h0 & 0xFFu;
    const u32 B4 = base * 0x01010101u;
    const u32 lut_lo = B4 - 0x03020100u, lut_hi = B4 - 0x07060504u;
    u32 E[8];
#pragma unroll
    for (int t = 0; t < 8; ++t) {
        const u32 c = ((cp[0] >> t) & 0x01010101u) | (((cp[1] >> t) & 0x01010101u) << 1) | (((cp[2] >> t) & 0x01010101u) << 2);
        E[t] = perm(lut_hi, lut_lo, c);
    }
    if (cnt) {
        if (cnt == 255) {
            u32x4 a = __builtin_amdgcn_raw_buffer_load_b128(fb, lane * 16, fbo, AUX_SYS);
            u32x4 b = __builtin_amdgcn_raw_buffer_load_b128(fb, lane * 16 + 1024, fbo, AUX_SYS);
            asm volatile("" : "+v"(a), "+v"(b));
#pragma unroll
            for (int t = 0; t < 4; ++t) { E[t] = a[t]; E[4 + t] = b[t]; }
        } else {
            *(LAS u32x4*)(scr + lane * 32) = (u32x4){E[0], E[1], E[2], E[3]};
            *(LAS u32x4*)(scr + lane * 32 + 16) = (u32x4){E[4], E[5], E[6], E[7]};
            asm volatile("" ::: "memory");
#pragma unroll
            for (int k = 0; k < 2; ++k) {
                const u32 ent = (lst >> (16 * k)) & 0xFFFFu, c5 = ent & 31u;
                const u32 e = c5 < 16 ? (c5 == 15 ? 0u : base - 8 - c5) : base + c5 - 15;
                if (lane < 32 && (u32)(lane * 2 + k - 1) < cnt) scr[ent >> 5] = (unsigned char)e;
            }
            asm volatile("" ::: "memory");
            const u32x4 a = *(const LAS u32x4*)(scr + lane * 32), b = *(const LAS u32x4*)(scr + lane * 32 + 16);
#pragma unroll
            for (int t = 0; t < 4; ++t) { E[t] = a[t]; E[4 + t] = b[t]; }
        }
    }
#pragma unroll
    for (int t = 0; t < 8; ++t) {
        const u32 sm = t < 4 ? s0[t & 3] : s1[t & 3];
        const u32 lo = bfi(0x7F7F7F7Fu, sm, E[t] << 7), hi = bfi(0x80808080u, sm, E[t] >> 1);
        d[2 * t] = perm(hi, lo, 0x05010400u); d[2 * t + 1] = perm(hi, lo, 0x07030602u);
    }
    if (!ok) {
#pragma unroll
        for (int i = 0; i < 16; ++i) d[i] = 0x7FC07FC0u;
    }
}

__host__ __device__ __forceinline__ void tile_order(u32 m, u32& rg, u32& cg) {
    rg = m < 32 ? (m & 3) : (m >> 3);
    cg = m < 32 ? (m >> 2) : (m & 7);
}
__host__ __device__ __forceinline__ void half_tile(int h, int& rg, int& c64) {
    if (h < 64) { rg = h & 3; c64 = h >> 2; }
    else if (h < 192) { const int k = (h - 64) >> 5, i = h & 31; rg = 4 + (k >> 1) * 4 + (i >> 3); c64 = (k & 1) * 8 + (i & 7); }
    else { rg = 12 + ((h - 192) >> 4); c64 = h & 15; }
}

__global__ __launch_bounds__(NT, 2) void fused_gemm_ar(Params p) {
    extern __shared__ __attribute__((aligned(16))) unsigned char smem[];
    LAS unsigned char* lds = (LAS unsigned char*)smem;
    const int tid = threadIdx.x, lane = tid & 63, wv = (int)rfl(tid >> 6);
    const int b = blockIdx.x, rank = p.rank, ri = rank >> 2, rj = rank & 3;
    char* ws = p.ws[rank];
    char* tmo = ws + OFF_TMO;
    STAMP(ws, b, 0);
    const u64 t_start = rtc();
    const u32 callv = ld_agent(ws + OFF_CTR + b * 4);

    {
        u32 d[16];
        const int isw = b >> 7, g = (b & 127) >> 2, q = b & 3;
        if (!isw) {
            const u16* src = p.x + (size_t)(g * 128 + wv * 16 + (lane & 15)) * KS + q * 128 + (lane >> 4) * 8;
#pragma unroll
            for (int f = 0; f < 4; ++f) {
                const u32x4 v = *(const GAS u32x4*)(src + f * 32);
                d[4 * f] = v[0]; d[4 * f + 1] = v[1]; d[4 * f + 2] = v[2]; d[4 * f + 3] = v[3];
            }
        } else {
            const u16* src = p.w + (size_t)(q * 128 + (tid >> 4)) * NN + g * 128 + (tid & 15) * 8;
            u32x4 v[4];
#pragma unroll
            for (int i = 0; i < 4; ++i) v[i] = *(const GAS u32x4*)(src + (size_t)i * 32 * NN);
#pragma unroll
            for (int i = 0; i < 4; ++i) *(LAS u32x4*)(lds + ((tid >> 4) + 32 * i) * WPITCH + (tid & 15) * 16) = v[i];
            __syncthreads();
            const int a0 = (8 * (lane >> 4) + ((lane & 15) >> 2)) * WPITCH + (wv * 16 + 4 * (lane & 3)) * 2;
#pragma unroll
            for (int f = 0; f < 4; ++f)
#pragma unroll
                for (int h = 0; h < 2; ++h) {
                    const s16x4 r = __builtin_amdgcn_ds_read_tr16_b64_v4i16((LAS s16x4*)(lds + a0 + (f * 32 + h * 4) * WPITCH));
                    const u32x2 rr = __builtin_bit_cast(u32x2, r);
                    d[4 * f + 2 * h] = rr[0]; d[4 * f + 2 * h + 1] = rr[1];
                }
        }
        const Packed pk = pack_chunk(d, mk_rsrc(ws + OFF_FBIN + (size_t)b * FB_UNIT, FB_UNIT), wv * FB_CH, lane, lds + SCR_OFF + wv * SCR_WAVE);
        if (!isw && (g & 15) < NPUSH / 32) {
            for (int j = 0; j < 4; ++j) {
                const u32 dest = (u32)((g >> 4) * 4 + j);
                if (dest != (u32)rank && dest < (u32)NR)
                    store_packed(pk, mk_rsrc(p.ws[dest] + OFF_PUSH + (size_t)rank * NPUSH * CH_BYTES, NPUSH * CH_BYTES),
                                 (u32)(((g & 15) * 4 + q) * 8 + wv) * CH_BYTES, lane);
            }
        } else {
            store_packed(pk, mk_rsrc(ws + OFF_PIN + (size_t)b * UNIT_BYTES, UNIT_BYTES), wv * CH_BYTES, lane);
        }
        const bool mine = isw ? ((g >> 3) == rj) : ((g >> 4) == ri);
        const int item = isw ? 16 + (g & 7) : (g & 15);
        if (mine) {
            const rsrc_t dst = mk_rsrc(ws + (isw ? OFF_WF + ((size_t)(g & 7) << 20) : OFF_XF + ((size_t)(g & 15) << 20)), 1u << 20);
#pragma unroll
            for (int f = 0; f < 4; ++f)
                __builtin_amdgcn_raw_buffer_store_b128((u32x4){d[4 * f], d[4 * f + 1], d[4 * f + 2], d[4 * f + 3]}, dst,
                                                       (u32)(((rank * 16 + q * 4 + f) * 8 + wv) * 1024 + lane * 16), 0, AUX_AGENT);
        }
        drain();
        __syncthreads();
        const u32 call0 = rfl(callv) + 1;
        if (tid < (isw ? 2 : 4)) {
            const u32 dest = isw ? (u32)(tid * 4 + (g >> 3)) : (u32)((g >> 4) * 4 + tid);
            if (dest < (u32)NR) {
                if (dest != (u32)rank) st_sys(p.ws[dest] + OFF_FIN + (size_t)(rank * 256 + b) * 128, call0);
                else __hip_atomic_fetch_add((GAS u32*)(ws + OFF_CNT + (item * 8 + rank) * 128), 8u, __ATOMIC_RELAXED, __HIP_MEMORY_SCOPE_AGENT);
            }
        }
        if (tid == 0) st_agent(ws + OFF_CTR + b * 4, call0);
    }
    const u32 call = rfl(callv) + 1;
    STAMP(ws, b, 1);
    const int xcd = b & 7, slot = b >> 3;

    if (slot < GS) {
        const int h0 = xcd * GS + slot;
        LAS u32* okw = (LAS u32*)(lds + LDS_MISC);
        {
            const u32 i = (u32)h0 + (u32)wv * (8 * GS);
            if (i < 7u * NPUSH) {
                const u32 src = (u32)(rank + 1 + i % 7) & 7u, c = i / 7, uid = ri * 64 + (c >> 3), v = c & 7;
                const bool ok = wait_ge<true>(ws + OFF_FIN + (size_t)(src * 256 + uid) * 128, call, tmo, call);
                asm volatile("" ::: "memory");
                const rsrc_t pr = mk_rsrc(ws + OFF_PUSH + (size_t)(src * NPUSH + c) * CH_BYTES, CH_BYTES);
                const u32x4 s0 = __builtin_amdgcn_raw_buffer_load_b128(pr, lane * 16, 0, AUX_SYS);
                const u32x4 s1 = __builtin_amdgcn_raw_buffer_load_b128(pr, lane * 16 + 1024, 0, AUX_SYS);
                const u32x3 cp = __builtin_amdgcn_raw_buffer_load_b96(pr, lane * 12 + 2048, 0, AUX_SYS);
                const u32 lst = __builtin_amdgcn_raw_buffer_load_b32(pr, (lane & 31) * 4 + LST_OFF, 0, AUX_SYS);
                u32 d[16];
                unpack_chunk(d, s0, s1, cp, lst, mk_rsrc(p.ws[src] + OFF_FBIN, 256 * FB_UNIT), (uid * 8 + v) * FB_CH, lane, ok, lds + wv * SCR_WAVE);
                const rsrc_t dst = mk_rsrc(ws + OFF_XF + ((size_t)(c >> 5) << 20), 1u << 20);
#pragma unroll
                for (int f = 0; f < 4; ++f)
                    __builtin_amdgcn_raw_buffer_store_b128((u32x4){d[4 * f], d[4 * f + 1], d[4 * f + 2], d[4 * f + 3]}, dst,
                                                           (u32)(((src * 16 + ((c >> 3) & 3) * 4 + f) * 8 + v) * 1024 + lane * 16), 0, AUX_AGENT);
                drain();
                if (lane == 0) __hip_atomic_fetch_add((GAS u32*)(ws + OFF_CNT + ((c >> 5) * 8 + src) * 128), 1u, __ATOMIC_RELAXED, __HIP_MEMORY_SCOPE_AGENT);
            }
        }
#define G_BAR() do { __builtin_amdgcn_s_barrier(); asm volatile("" ::: "memory"); } while (0)
        for (int round = 0; round < (h0 < 256 - 8 * GS ? 2 : 1); ++round) {
            int rg, c64;
            half_tile(h0 + round * 8 * GS, rg, c64);
            const int cg = c64 >> 1, hn = c64 & 1;
            if (wv == 0) {
                const u32 e = max(rg < 4 ? rg : rg + 8, 4 + cg);
                while (rtc() - t_start < (u64)(GATE0 + GATE1 * e)) __builtin_amdgcn_s_sleep(8);
                wait_all16(ws + OFF_CNT + (((lane & 8) ? 16 + cg : rg) * 8 + (lane & 7)) * 128, call * 32, tmo, call, lane);
            }
            asm volatile("" ::: "memory"); __builtin_amdgcn_s_barrier(); asm volatile("" ::: "memory");
            STAMP(ws, b, 2 + 3 * round);
            f32x4 acc[4][2];
#pragma unroll
            for (int m = 0; m < 4; ++m) { acc[m][0] = (f32x4){0.f, 0.f, 0.f, 0.f}; acc[m][1] = (f32x4){0.f, 0.f, 0.f, 0.f}; }
            const int wr = (wv >> 1) & 1, wc = wv & 1;
            if (wv >= 4) {
                const int lw = wv - 4;
                const char* xg = ws + OFF_XF + ((size_t)rg << 20) + lw * 4096 + lane * 16;
                const char* wg = ws + OFF_WF + ((size_t)cg << 20) + (lw >> 1) * 8192 + hn * 4096 + (lw & 1) * 2048 + lane * 16;
#define L_ISSUE(s, SL) do { const char* _x = xg + (size_t)(s) * 16384; const char* _w = wg + (size_t)(s) * 16384; \
                _Pragma("unroll") for (int i = 0; i < 4; ++i) __builtin_amdgcn_global_load_lds((const GAS u32*)(_x + i * 1024), (LAS u32*)(lds + (SL) * GSLOT + lw * 4096 + i * 1024), 16, 0, GL_AUX); \
                _Pragma("unroll") for (int i = 0; i < 2; ++i) __builtin_amdgcn_global_load_lds((const GAS u32*)(_w + i * 1024), (LAS u32*)(lds + (SL) * GSLOT + 16384 + lw * 2048 + i * 1024), 16, 0, GL_AUX); } while (0)
#define L_WAIT(n) asm volatile("s_waitcnt vmcnt(" #n ")" ::: "memory")
                L_ISSUE(0, 0); L_ISSUE(1, 1); L_ISSUE(2, 2); L_ISSUE(3, 3);
                L_WAIT(18); G_BAR();
                for (int s = 0; s < 60; s += 4) {
                    L_WAIT(12); G_BAR(); L_ISSUE(s + 4, 0);
                    L_WAIT(12); G_BAR(); L_ISSUE(s + 5, 1);
                    L_WAIT(12); G_BAR(); L_ISSUE(s + 6, 2);
                    L_WAIT(12); G_BAR(); L_ISSUE(s + 7, 3);
                }
                L_WAIT(12); G_BAR(); L_WAIT(6); G_BAR(); L_WAIT(0); G_BAR(); G_BAR();
            } else {
                const int xro = wr * 4096 + lane * 16, wro = 16384 + wc * 2048 + lane * 16;
                bf16x8 xa[2][2][4], wb[2][2][2];
#define C_READ(P, SL) do { _Pragma("unroll") for (int k = 0; k < 2; ++k) { \
                _Pragma("unroll") for (int n = 0; n < 2; ++n) wb[P][k][n] = *(const LAS bf16x8*)(lds + (SL) * GSLOT + wro + k * 4096 + n * 1024); \
                _Pragma("unroll") for (int m = 0; m < 4; ++m) xa[P][k][m] = *(const LAS bf16x8*)(lds + (SL) * GSLOT + xro + k * 8192 + m * 1024); } } while (0)
#define C_MMA(P) do { _Pragma("unroll") for (int k = 0; k < 2; ++k) _Pragma("unroll") for (int m = 0; m < 4; ++m) \
                _Pragma("unroll") for (int n = 0; n < 2; ++n) acc[m][n] = __builtin_amdgcn_mfma_f32_16x16x32_bf16(wb[P][k][n], xa[P][k][m], acc[m][n], 0, 0, 0); } while (0)
#define C_SCHED() do { _Pragma("unroll") for (int i = 0; i < 12; ++i) { __builtin_amdgcn_sched_group_barrier(0x008, 1, 0); __builtin_amdgcn_sched_group_barrier(0x100, 1, 0); } \
                __builtin_amdgcn_sched_group_barrier(0x008, 4, 0); } while (0)
#define C_STEP(SL, P, RD) do { asm volatile("s_waitcnt lgkmcnt(0)" ::: "memory"); G_BAR(); if (RD) C_READ((P) ^ 1, ((SL) + 1) & 3); C_MMA(P); if (RD) C_SCHED(); __builtin_amdgcn_sched_barrier(0); } while (0)
                G_BAR();
                C_READ(0, 0);
                for (int s = 0; s < 60; s += 4) { C_STEP(0, 0, true); C_STEP(1, 1, true); C_STEP(2, 0, true); C_STEP(3, 1, true); }
                C_STEP(0, 0, true); C_STEP(1, 1, true); C_STEP(2, 0, true); C_STEP(3, 1, false);
            }
            G_BAR();
            STAMP(ws, b, 3 + 3 * round);
            if (wv < 4) {
#pragma unroll
                for (int m = 0; m < 4; ++m)
#pragma unroll
                    for (int n = 0; n < 2; ++n) {
                        u32x2 o;
                        asm("v_cvt_pk_bf16_f32 %0, %1, %2" : "=v"(o.x) : "v"(acc[m][n][0]), "v"(acc[m][n][1]));
                        asm("v_cvt_pk_bf16_f32 %0, %1, %2" : "=v"(o.y) : "v"(acc[m][n][2]), "v"(acc[m][n][3]));
                        *(LAS u32x2*)(lds + (wr * 64 + m * 16 + (lane & 15)) * OPITCH + (wc * 32 + n * 16 + (lane >> 4) * 4) * 2) = o;
                    }
            }
            if (tid == 0) okw[0] = (ld_agent(tmo) == call) ? 1u : 0u;
            __syncthreads();
            u32 d[16];
            const int j = wv & 3;
#pragma unroll
            for (int f = 0; f < 4; ++f) {
                const u32x4 v = *(const LAS u32x4*)(lds + (j * 32 + f * 8 + (lane >> 3)) * OPITCH + (lane & 7) * 16);
                d[4 * f] = v[0]; d[4 * f + 1] = v[1]; d[4 * f + 2] = v[2]; d[4 * f + 3] = v[3];
            }
            if (okw[0]) {
#pragma unroll
                for (int i = 0; i < 16; ++i) d[i] = 0x7FC07FC0u;
            }
            const int t = rg * 8 + cg;
            if (wv < 4) {
                const Packed pk = pack_chunk(d, mk_rsrc(ws + OFF_FBOUT + (size_t)t * FB_UNIT, FB_UNIT), (hn * 4 + j) * FB_CH, lane, lds + SCR_OFF + wv * SCR_WAVE);
                store_packed(pk, mk_rsrc(ws + OFF_POUT + (size_t)t * UNIT_BYTES, UNIT_BYTES), (hn * 4 + j) * CH_BYTES, lane);
                drain();
            } else {
                const rsrc_t outr = mk_rsrc(p.out, 4096u * NN * 2);
                const u32 oo = (u32)((ri * 2048 + rg * 128 + j * 32 + (lane >> 3)) * NN + rj * 1024 + c64 * 64 + (lane & 7) * 8) * 2;
#pragma unroll
                for (int f = 0; f < 4; ++f)
                    __builtin_amdgcn_raw_buffer_store_b128((u32x4){d[4 * f], d[4 * f + 1], d[4 * f + 2], d[4 * f + 3]}, outr, oo + f * 8 * NN * 2, 0, OUT_AUX);
            }
            __syncthreads();
            if (tid < NR && tid != rank) st_sys(p.ws[tid] + OFF_FOUT + (size_t)(rank * 256 + t * 2 + hn) * 128, call);
            STAMP(ws, b, 4 + 3 * round);
        }
        STAMPC(ws, b, call, 1);
        return;
    }

    const int pi = (slot - GS) * 8 + xcd;
    {
        const int gw = pi * 8 + wv;
        if (gw >= 7 * NWL) return;
        const int link = gw % 7, wl = gw / 7;
        const u32 src = rfl((u32)(rank + 1 + link) & 7u);
        if (src >= (u32)NR) return;
        const u64 pws = uni64((u64)p.ws[src]), wsu = uni64((u64)ws);
        LAS unsigned char* scr = lds + wv * SCR_WAVE;
        GAS u32* tick = (GAS u32*)(ws + OFF_TICK + link * 128);
        const u32 tbase = NPUSH + 3 * NWL - (call - 1) * (u32)(NCH - NPUSH - 2 * NWL);
        struct Desc { u32 rem, fb, flg, dst, cnt; };
        const u32 xbase = ri * 64, wbase = 128 + rj * 32;
        const u32 obase = ((src >> 2) * 2048 * NN + (src & 3) * 1024) * 2;
        auto mk = [&](u32 c) -> Desc {
            c = min(c, (u32)NCH - 1);
            const u32 u = c >> 3, v = c & 7, e = u >> 2, q = u & 3, m = u - IN_UNITS;
            const bool in = u < IN_UNITS, w = (e - 4u) < 8u;
            const u32 pan = w ? e - 4 : (e < 4 ? e : e - 8);
            const u32 uidi = (w ? wbase : xbase) + pan * 4 + q;
            u32 rgo, cgo;
            tile_order(m, rgo, cgo);
            const u32 uid = in ? uidi : rgo * 8 + cgo;
            Desc d;
            d.rem = (in ? (u32)OFF_PIN : (u32)OFF_POUT) + uid * UNIT_BYTES + v * CH_BYTES;
            d.fb = (in ? (u32)OFF_FBIN : (u32)OFF_FBOUT) + (uid * 8 + v) * FB_CH;
            d.flg = in ? (u32)OFF_FIN + (src * 256 + uid) * 128 : (u32)OFF_FOUT + (src * 256 + uid * 2 + (v >> 2)) * 128;
            d.dst = in ? (w ? (u32)OFF_WF : (u32)OFF_XF) + (pan << 20) + (src * 128 + q * 32 + v) * 1024
                       : obase + ((rgo * 128 + (v & 3) * 32) * NN + cgo * 128 + (v >> 2) * 64) * 2;
            d.cnt = in ? (u32)OFF_CNT + (((w ? 16 : 0) + pan) * 8 + src) * 128 : 0u;
            return d;
        };
        const rsrc_t wsr = mk_rsrc(ws, (u32)WS_NEED), outr = mk_rsrc(p.out, 4096u * NN * 2), pwr = mk_rsrc((const void*)pws, (u32)WS_NEED);
        const u32 vin = lane * 16, vout = (lane >> 3) * (NN * 2) + (lane & 7) * 16, vl4 = LST_OFF + (lane & 31) * 4;
        u32 n1 = NPUSH + NWL + wl, n2 = NPUSH + 2 * NWL + wl;
        Desc D0 = mk(NPUSH + wl), D1 = mk(n1), D2 = mk(n2);
#define P_CLOB "a0", "a1", "a2", "a3", "a4", "a5", "a6", "a7", "a8", "a9", "a10", "a11"
#define P_ISSUE(DA) do { \
            const u64 un_ = pws + (DA).rem; \
            asm volatile("s_nop 4\n\tglobal_load_dwordx4 a[0:3], %0, %2 sc0 sc1\n\tglobal_load_dwordx4 a[4:7], %0, %2 offset:1024 sc0 sc1\n\t" \
                         "global_load_dwordx3 a[8:10], %1, %2 offset:2048 sc0 sc1\n\tglobal_load_dword a11, %3, %2 sc0 sc1" \
                         :: "v"(vin), "v"(lane * 12), "s"(un_), "v"(vl4) : P_CLOB, "memory"); } while (0)
#define P_FLAG(DB) do { const u64 fa_ = wsu + (DB).flg; \
            asm volatile("s_nop 4\n\tglobal_load_dword a12, %0, %1 sc0 sc1" :: "v"(0), "s"(fa_) : "a12", "memory"); } while (0)
#define P_TICKET(want_) do { u64 sv_; const u32 em_ = rfl((want_) ? 1u : 0u); \
            asm volatile("v_accvgpr_write_b32 a14, 1\n\ts_mov_b64 %0, exec\n\ts_mov_b32 exec_lo, %2\n\ts_mov_b32 exec_hi, 0\n\ts_nop 1\n\t" \
                         "global_atomic_add a13, %1, a14, off sc0\n\ts_mov_b64 exec, %0" \
                         : "=&s"(sv_) : "v"(tick), "s"(em_) : "a13", "a14", "memory"); } while (0)
#define P_FETCH() asm volatile("v_accvgpr_read_b32 %0, a0\n\tv_accvgpr_read_b32 %1, a1\n\tv_accvgpr_read_b32 %2, a2\n\tv_accvgpr_read_b32 %3, a3\n\t" \
                         "v_accvgpr_read_b32 %4, a4\n\tv_accvgpr_read_b32 %5, a5\n\tv_accvgpr_read_b32 %6, a6\n\tv_accvgpr_read_b32 %7, a7\n\t" \
                         "v_accvgpr_read_b32 %8, a8\n\tv_accvgpr_read_b32 %9, a9\n\tv_accvgpr_read_b32 %10, a10\n\tv_accvgpr_read_b32 %11, a11\n\t" \
                         "v_accvgpr_read_b32 %12, a12\n\tv_accvgpr_read_b32 %13, a13" \
                         : "=v"(c0.x), "=v"(c0.y), "=v"(c0.z), "=v"(c0.w), "=v"(c1.x), "=v"(c1.y), "=v"(c1.z), "=v"(c1.w), \
                           "=v"(cc.x), "=v"(cc.y), "=v"(cc.z), "=v"(cl), "=v"(fl), "=v"(tk) :: "memory")
#define P_SIGNAL(c_) do { u64 sv_; const u64 ca_ = wsu + (c_); \
            asm volatile("s_nop 4\n\ts_mov_b64 %0, exec\n\ts_mov_b64 exec, 1\n\tglobal_atomic_add %1, %2, %3\n\ts_mov_b64 exec, %0" \
                         : "=&s"(sv_) : "v"(0), "v"(1u), "s"(ca_) : "memory"); } while (0)
        bool ok = wait_ge<true>(ws + D0.flg, call, tmo, call);
        asm volatile("" ::: "memory");
        P_ISSUE(D0); P_FLAG(D1);
        bool tkv = false;
        drain();
        u32 pendA = 0, pendB = 0;
#define TS() do {} while (0)
        for (;;) {
            asm volatile("s_waitcnt vmcnt(4)" ::: "memory");
            TS();
            u32x4 c0, c1; u32x3 cc; u32 cl, fl, tk;
            P_FETCH();
            const bool cok = ok;
            const bool more = n1 < (u32)NCH;
            const bool early = more && rfl(fl) >= call;
            if (early) { ok = true; asm volatile("" ::: "memory"); P_ISSUE(D1); }
            if (tkv) { n2 = rfl(tk) + tbase; D2 = mk(n2); }
            if (early) { P_FLAG(D2); tkv = n2 < (u32)NCH; P_TICKET(tkv); }
            if (pendA) P_SIGNAL(pendA);
            pendA = pendB;
            u32 d[16];
            unpack_chunk(d, c0, c1, cc, cl, pwr, D0.fb, lane, cok, scr);
            {
                const bool isin = D0.cnt != 0;
                const rsrc_t dr = isin ? wsr : outr;
                const u32 vo = isin ? vin : vout, step = isin ? 8192u : 8u * NN * 2;
#pragma unroll
                for (int f = 0; f < 4; ++f)
                    __builtin_amdgcn_raw_buffer_store_b128((u32x4){d[4 * f], d[4 * f + 1], d[4 * f + 2], d[4 * f + 3]}, dr, vo, D0.dst + f * step, AUX_AGENT);
            }
            u32 mine = D0.cnt;
            if (!early) {
                if (pendA | mine) { drain(); if (pendA) P_SIGNAL(pendA); if (mine) P_SIGNAL(mine); }
                pendA = 0; mine = 0;
                if (!more) break;
                ok = wait_ge<true>(ws + D1.flg, call, tmo, call);
                asm volatile("" ::: "memory");
                P_ISSUE(D1); P_FLAG(D2); tkv = n2 < (u32)NCH; P_TICKET(tkv);
                drain();
            }
            pendB = mine;
            n1 = n2; if (!tkv) n2 = NCH;
            D0 = D1; D1 = D2;
        }
        STAMPW(ws, b, 4);
        drain();
        STAMPC(ws, b, call, 1);
    }
}
}

#define HIP_CHECK(expr) do { hipError_t _e = (expr); if (_e != hipSuccess) { \
    fprintf(stderr, "HIP error %s at %s:%d\n", hipGetErrorString(_e), __FILE__, __LINE__); \
    return; } } while (0)

extern "C" void kernel_launch(int rank, int nranks, ncclComm_t comm,
                              hipStream_t stream,
                              void* const* d_in, const int* in_sizes, int n_in,
                              void* d_out, int out_size,
                              const int* in_dtypes, int out_dtype,
                              void* const* d_out_all,
                              void* const* d_ws_all, size_t ws_bytes) {
    (void)comm; (void)in_sizes; (void)n_in; (void)out_size; (void)in_dtypes; (void)out_dtype; (void)d_out_all;
    if (nranks != NR || ws_bytes < WS_NEED || rank < 0 || rank >= NR) { fprintf(stderr, "unexpected configuration\n"); return; }
    static bool attr_done[64] = {};
    int dev = 0;
    HIP_CHECK(hipGetDevice(&dev));
    if (dev >= 0 && dev < 64 && !attr_done[dev]) {
        HIP_CHECK(hipFuncSetAttribute((const void*)fused_gemm_ar, hipFuncAttributeMaxDynamicSharedMemorySize, LDS_BYTES));
        attr_done[dev] = true;
    }
    Params p;
    p.x = (const u16*)d_in[0];
    p.w = (const u16*)d_in[1];
    p.out = (u16*)d_out;
    for (int i = 0; i < NR; ++i) p.ws[i] = (char*)d_ws_all[i];
    p.rank = rank; p.pad = 0;
    hipLaunchKernelGGL(fused_gemm_ar, dim3(GRID), dim3(NT), LDS_BYTES, stream, p);
    HIP_CHECK(hipGetLastError());
}
```

```cpp
#include <hip/hip_runtime.h>
#include <rccl/rccl.h>
#include <cstdio>
#include <cstdint>

namespace {
typedef unsigned u32;
typedef unsigned long long u64;
typedef unsigned short u16;
typedef u32 u32x4 __attribute__((ext_vector_type(4)));
typedef u32 u32x2 __attribute__((ext_vector_type(2)));
typedef u32 u32x3 __attribute__((ext_vector_type(3)));
typedef short bf16x8 __attribute__((ext_vector_type(8)));
typedef short s16x4 __attribute__((ext_vector_type(4)));
typedef u16 u16x2 __attribute__((ext_vector_type(2)));
typedef float f32x4 __attribute__((ext_vector_type(4)));
#define LAS __attribute__((address_space(3)))
#define GAS __attribute__((address_space(1)))

constexpr int NR = 8;
constexpr int NN = 4096, KS = 512;
constexpr int NT = 512, GRID = 256;
#define PULL_W 25
constexpr int NWL = PULL_W;
#define PULL_SLOTS 8
constexpr int GS = 32 - PULL_SLOTS;
static_assert(GS % 2 == 0 && GS >= 16, "rows");
constexpr int NPULL = 8 * PULL_SLOTS;
#define PULL_WPW 4
constexpr int PWPW = PULL_WPW;
static_assert(7 * NWL <= NPULL * PWPW, "puller waves");
constexpr u64 DEADLINE = 40000000ull;

constexpr u32 CH_BYTES = 2944, LST_OFF = 2816, UNIT_BYTES = 8 * CH_BYTES;
constexpr u32 LST_CAP = 63;
constexpr u32 FB_CH = 2048, FB_UNIT = 8 * FB_CH;
constexpr int IN_UNITS = 96, OUT_UNITS = 128, ALL_UNITS = IN_UNITS + OUT_UNITS;
constexpr int NCH = ALL_UNITS * 8;
#define PUSH_N 64
constexpr int NPUSH = PUSH_N;

constexpr size_t OFF_CTR = 0;
constexpr size_t OFF_TMO = 4096;
constexpr size_t OFF_TICK = 5120;
constexpr size_t OFF_CNT = 8192;
constexpr size_t OFF_FIN = OFF_CNT + 24 * 8 * 128;
constexpr size_t OFF_FOUT = OFF_FIN + (size_t)8 * 256 * 128;
constexpr size_t OFF_DBG = OFF_FOUT + (size_t)8 * 256 * 128;
constexpr size_t OFF_DBG2 = OFF_DBG + 65536;
constexpr size_t OFF_DBG3 = OFF_DBG2 + 16384;
constexpr size_t OFF_PIN = OFF_DBG3 + 384 * 512;
constexpr size_t OFF_PUSH = OFF_PIN + (size_t)256 * UNIT_BYTES;
constexpr size_t OFF_POUT = OFF_PUSH + (size_t)8 * NPUSH * CH_BYTES;
constexpr size_t OFF_FBIN = OFF_POUT + (size_t)128 * UNIT_BYTES;
constexpr size_t OFF_FBOUT = OFF_FBIN + (size_t)256 * FB_UNIT;
constexpr size_t OFF_XF = OFF_FBOUT + (size_t)128 * FB_UNIT;
constexpr size_t OFF_WF = OFF_XF + ((size_t)16 << 20);
constexpr size_t WS_NEED = OFF_WF + ((size_t)8 << 20);
static_assert(OFF_PIN % 128 == 0 && OFF_XF % 1024 == 0, "alignment");

constexpr int LDS_BYTES = 131072 + 1024;
constexpr int LDS_MISC = 131072;
constexpr int GSLOT = 24576;
constexpr int WPITCH = 288;
constexpr int OPITCH = 144;
constexpr int SCR_OFF = 65536, SCR_WAVE = 2304;

struct Params {
    const u16* x; const u16* w; u16* out;
    char* ws[NR];
    int rank; int pad;
};

typedef __amdgpu_buffer_rsrc_t rsrc_t;
__device__ __forceinline__ rsrc_t mk_rsrc(const void* p, u32 bytes) {
    return __builtin_amdgcn_make_buffer_rsrc((void*)p, 0, bytes, 0x00020000);
}
constexpr int AUX_SYS = 17, AUX_AGENT = 16;
#define GATE0 1000
#define GATE1 150
#define POLL_SLEEP 8
#define PULL_GATE 0
#define GL_AUX 17
#define OUT_AUX 16
__device__ __forceinline__ u64 rtc() { return __builtin_amdgcn_s_memrealtime(); }
__device__ __forceinline__ u32 rfl(u32 v) { return __builtin_amdgcn_readfirstlane(v); }
__device__ __forceinline__ u64 uni64(u64 v) { return ((u64)rfl((u32)(v >> 32)) << 32) | rfl((u32)v); }
__device__ __forceinline__ u32 bfi(u32 m, u32 a, u32 b) { return (m & a) | (~m & b); }
__device__ __forceinline__ u32 perm(u32 a, u32 b, u32 s) { return __builtin_amdgcn_perm(a, b, s); }
__device__ __forceinline__ void drain() { asm volatile("s_waitcnt vmcnt(0)" ::: "memory"); }
__device__ __forceinline__ u32 ld_sys(const char* p) { return __hip_atomic_load((const GAS u32*)p, __ATOMIC_RELAXED, __HIP_MEMORY_SCOPE_SYSTEM); }
__device__ __forceinline__ u32 ld_agent(const char* p) { return __hip_atomic_load((const GAS u32*)p, __ATOMIC_RELAXED, __HIP_MEMORY_SCOPE_AGENT); }
__device__ __forceinline__ void st_sys(char* p, u32 v) { __hip_atomic_store((GAS u32*)p, v, __ATOMIC_RELAXED, __HIP_MEMORY_SCOPE_SYSTEM); }
__device__ __forceinline__ void st_agent(char* p, u32 v) { __hip_atomic_store((GAS u32*)p, v, __ATOMIC_RELAXED, __HIP_MEMORY_SCOPE_AGENT); }

#define STAMPC(ws, b, call, i) do {} while (0)
#define STAMP(ws, b, i) do {} while (0)
#define STAMPW(ws, b, i) do {} while (0)

template <bool SYS>
__device__ __forceinline__ bool wait_ge(const char* flag, u32 want, char* tmo, u32 call) {
    const u64 t0 = rtc();
    for (u32 n = 1;; ++n) {
        const u32 v = rfl(SYS ? ld_sys(flag) : ld_agent(flag));
        if (v >= want) return true;
        if ((n & 7) == 0 && (rfl(ld_agent(tmo)) == call || rtc() - t0 > DEADLINE)) { st_agent(tmo, call); return false; }
        __builtin_amdgcn_s_sleep(1);
    }
}
__device__ __forceinline__ bool wait_all16(const char* cnt_lane, u32 want, char* tmo, u32 call, int lane) {
    const u64 t0 = rtc();
    for (u32 n = 1;; ++n) {
        const u32 v = lane < 16 ? ld_agent(cnt_lane) : want;
        if (__all(v >= want)) return true;
        if ((n & 7) == 0 && (rfl(ld_agent(tmo)) == call || rtc() - t0 > DEADLINE)) { st_agent(tmo, call); return false; }
        __builtin_amdgcn_s_sleep(POLL_SLEEP);
    }
}

#define DPP_ADD(v, ctrl, rm) v += (u32)__builtin_amdgcn_update_dpp(0, (int)(v), ctrl, rm, 0xF, true)
__device__ __forceinline__ u32 wave_sum(u32 v) {
    DPP_ADD(v, 0xB1, 0xF); DPP_ADD(v, 0x4E, 0xF); DPP_ADD(v, 0x141, 0xF); DPP_ADD(v, 0x140, 0xF);
    return (__builtin_amdgcn_readlane(v, 0) + __builtin_amdgcn_readlane(v, 16)) + (__builtin_amdgcn_readlane(v, 32) + __builtin_amdgcn_readlane(v, 48));
}
__device__ __forceinline__ u32 wave_scan(u32 v) {
    DPP_ADD(v, 0x111, 0xF); DPP_ADD(v, 0x112, 0xF); DPP_ADD(v, 0x114, 0xF); DPP_ADD(v, 0x118, 0xF);
    DPP_ADD(v, 0x142, 0xA); DPP_ADD(v, 0x143, 0xC);
    return v;
}
struct Packed { u32x4 s0, s1; u32x3 cp; u32 lst; };
__device__ __forceinline__ void store_packed(const Packed& k, rsrc_t dst, u32 off, int lane) {
    __builtin_amdgcn_raw_buffer_store_b128(k.s0, dst, lane * 16, off, AUX_SYS);
    __builtin_amdgcn_raw_buffer_store_b128(k.s1, dst, lane * 16 + 1024, off, AUX_SYS);
    __builtin_amdgcn_raw_buffer_store_b96(k.cp, dst, lane * 12 + 2048, off, AUX_SYS);
    if (lane < 32) __builtin_amdgcn_raw_buffer_store_b32(k.lst, dst, lane * 4 + LST_OFF, off, AUX_SYS);
}
__device__ __forceinline__ Packed pack_chunk(const u32 (&d)[16], rsrc_t fb, u32 fbo, int lane, LAS unsigned char* scr) {
    u32 SM[8], E[8], sum = 0;
#pragma unroll
    for (int t = 0; t < 8; ++t) {
        const u32 lo = perm(d[2 * t + 1], d[2 * t], 0x06040200u), hi = perm(d[2 * t + 1], d[2 * t], 0x07050301u);
        SM[t] = bfi(0x7F7F7F7Fu, lo, hi);
        E[t] = bfi(0xFEFEFEFEu, hi << 1, lo >> 7);
        sum = __builtin_amdgcn_udot4(E[t], 0x01010101u, sum, false);
    }
    const u32 base = min((wave_sum(sum) + 5755u) >> 11, 255u);
    const u16x2 b2 = {(u16)base, (u16)base}, eight = {8, 8};
    u32 P0 = 0, P1 = 0, P2 = 0, M = 0;
#pragma unroll
    for (int t = 7; t >= 0; --t) {
        const u16x2 ev = __builtin_bit_cast(u16x2, E[t] & 0x00FF00FFu), od = __builtin_bit_cast(u16x2, perm(E[t], E[t], 0x0C030C01u));
        const u32 ca = __builtin_bit_cast(u32, __builtin_elementwise_min((u16x2)(b2 - ev), eight));
        const u32 cb = __builtin_bit_cast(u32, __builtin_elementwise_min((u16x2)(b2 - od), eight));
        const u32 c = ca | (cb << 8);
        if (t < 4) P0 = (P0 << 2) | (c & 0x03030303u); else P1 = (P1 << 2) | (c & 0x03030303u);
        P2 = (P2 << 1) | ((c >> 2) & 0x01010101u);
        M = (M << 1) | ((c >> 3) & 0x01010101u);
    }
    const u32 n = __builtin_popcount(M);
    const u32 incl = wave_scan(n);
    u32 cnt = __builtin_amdgcn_readlane(incl, 63), lst = 0;
    bool raw = cnt > LST_CAP;
    if (!raw && cnt) {
        *(LAS u32x4*)(scr + lane * 32) = (u32x4){E[0], E[1], E[2], E[3]};
        *(LAS u32x4*)(scr + lane * 32 + 16) = (u32x4){E[4], E[5], E[6], E[7]};
        u32 idx = incl - n + 1;
        while (M) {
            const u32 bit = __builtin_ctz(M); M &= M - 1;
            ((LAS u16*)(scr + 2048))[idx & 63] = (u16)(lane * 32 + (bit & 7) * 4 + (bit >> 3));
            ++idx;
        }
        asm volatile("" ::: "memory");
        bool bad = false;
        if (lane < 32) {
            const u32 pp = *(const LAS u32*)(scr + 2048 + lane * 4);
#pragma unroll
            for (int k = 0; k < 2; ++k) {
                const u32 pos = (pp >> (16 * k)) & 2047u, e = scr[pos];
                const int dl = (int)base - (int)e;
                const u32 c5 = e == 0 ? 15u : (dl > 0 ? (u32)(dl - 8) : (u32)(15 - dl));
                if ((u32)(lane * 2 + k - 1) < cnt) { bad |= e != 0 && (dl > 22 || dl < -16); lst |= ((pos << 5) | (c5 & 31u)) << (16 * k); }
            }
        }
        raw = __ballot(bad) != 0;
    }
    if (raw) {
        cnt = 255;
        __builtin_amdgcn_raw_buffer_store_b128((u32x4){E[0], E[1], E[2], E[3]}, fb, lane * 16, fbo, AUX_SYS);
        __builtin_amdgcn_raw_buffer_store_b128((u32x4){E[4], E[5], E[6], E[7]}, fb, lane * 16 + 1024, fbo, AUX_SYS);
    }
    if (lane == 0) lst = (lst & 0xFFFF0000u) | base | (cnt << 8);
    return Packed{(u32x4){SM[0], SM[1], SM[2], SM[3]}, (u32x4){SM[4], SM[5], SM[6], SM[7]}, (u32x3){P0, P1, P2}, lst};
}

__device__ __forceinline__ void unpack_chunk(u32 (&d)[16], u32x4 s0, u32x4 s1, u32x3 cp, u32 lst, rsrc_t fb, u32 fbo, int lane, bool ok,
                                             LAS unsigned char* scr) {
    const u32 h0 = __builtin_amdgcn_readlane(lst, 0);
    const u32 cnt = ok ? (h0 >> 8) & 0xFFu : 0u, base = h0 & 0xFFu;
    const u32 B4 = base * 0x01010101u;
    const u32 lut_lo = B4 - 0x03020100u, lut_hi = B4 - 0x07060504u;
    u32 E[8], m3 = 0x03030303u, m4 = 0x04040404u;
    asm("" : "+s"(m3), "+s"(m4));
#pragma unroll
    for (int t = 0; t < 8; ++t) {
        const u32 c = ((cp[t >> 2] >> (2 * (t & 3))) & m3) | ((t < 2 ? cp[2] << (2 - t) : cp[2] >> (t - 2)) & m4);
        E[t] = perm(lut_hi, lut_lo, c);
    }
    if (__builtin_expect(cnt != 0, 1)) {
        if (__builtin_expect(cnt == 255, 0)) {
            u32x4 a = __builtin_amdgcn_raw_buffer_load_b128(fb, lane * 16, fbo, AUX_SYS);
            u32x4 b = __builtin_amdgcn_raw_buffer_load_b128(fb, lane * 16 + 1024, fbo, AUX_SYS);
            asm volatile("" : "+v"(a), "+v"(b));
#pragma unroll
            for (int t = 0; t < 4; ++t) { E[t] = a[t]; E[4 + t] = b[t]; }
        } else {
            *(LAS u32x4*)(scr + lane * 32) = (u32x4){E[0], E[1], E[2], E[3]};
            *(LAS u32x4*)(scr + lane * 32 + 16) = (u32x4){E[4], E[5], E[6], E[7]};
            asm volatile("" ::: "memory");
#pragma unroll
            for (int k = 0; k < 2; ++k) {
                const u32 ent = (lst >> (16 * k)) & 0xFFFFu, c5 = ent & 31u;
                const u32 e = c5 < 16 ? (c5 == 15 ? 0u : base - 8 - c5) : base + c5 - 15;
                if (lane < 32 && (u32)(lane * 2 + k - 1) < cnt) scr[ent >> 5] = (unsigned char)e;
            }
            asm volatile("" ::: "memory");
            const u32x4 a = *(const LAS u32x4*)(scr + lane * 32), b = *(const LAS u32x4*)(scr + lane * 32 + 16);
#pragma unroll
            for (int t = 0; t < 4; ++t) { E[t] = a[t]; E[4 + t] = b[t]; }
        }
    }
#pragma unroll
    for (int t = 0; t < 8; ++t) {
        const u32 sm = t < 4 ? s0[t & 3] : s1[t & 3];
        const u32 lo = bfi(0x7F7F7F7Fu, sm, E[t] << 7), hi = bfi(0x80808080u, sm, E[t] >> 1);
        d[2 * t] = perm(hi, lo, 0x05010400u); d[2 * t + 1] = perm(hi, lo, 0x07030602u);
    }
    if (__builtin_expect(!ok, 0)) {
        asm volatile("" ::: "memory");
#pragma unroll
        for (int i = 0; i < 16; ++i) d[i] = 0x7FC07FC0u;
    }
}

__host__ __device__ __forceinline__ void tile_order(u32 m, u32& rg, u32& cg) {
    rg = m < 32 ? (m & 3) : (m >> 3);
    cg = m < 32 ? (m >> 2) : (m & 7);
}
__host__ __device__ __forceinline__ void half_tile(int h, int& rg, int& c64) {
    if (h < 64) { rg = h & 3; c64 = h >> 2; }
    else if (h < 192) { const int k = (h - 64) >> 5, i = h & 31; rg = 4 + (k >> 1) * 4 + (i >> 3); c64 = (k & 1) * 8 + (i & 7); }
    else { rg = 12 + ((h - 192) >> 4); c64 = h & 15; }
}

__global__ __launch_bounds__(NT, 2) void fused_gemm_ar(Params p) {
    extern __shared__ __attribute__((aligned(16))) unsigned char smem[];
    LAS unsigned char* lds = (LAS unsigned char*)smem;
    const int tid = threadIdx.x, lane = tid & 63, wv = (int)rfl(tid >> 6);
    const int b = blockIdx.x, rank = p.rank, ri = rank >> 2, rj = rank & 3;
    char* ws = p.ws[rank];
    char* tmo = ws + OFF_TMO;
    STAMP(ws, b, 0);
    const u64 t_start = rtc();
    const u32 callv = ld_agent(ws + OFF_CTR + b * 4);

    {
        u32 d[16];
        const int isw = b >> 7, g = (b & 127) >> 2, q = b & 3;
        if (!isw) {
            const u16* src = p.x + (size_t)(g * 128 + wv * 16 + (lane & 15)) * KS + q * 128 + (lane >> 4) * 8;
#pragma unroll
            for (int f = 0; f < 4; ++f) {
                const u32x4 v = *(const GAS u32x4*)(src + f * 32);
                d[4 * f] = v[0]; d[4 * f + 1] = v[1]; d[4 * f + 2] = v[2]; d[4 * f + 3] = v[3];
            }
        } else {
            const u16* src = p.w + (size_t)(q * 128 + (tid >> 4)) * NN + g * 128 + (tid & 15) * 8;
            u32x4 v[4];
#pragma unroll
            for (int i = 0; i < 4; ++i) v[i] = *(const GAS u32x4*)(src + (size_t)i * 32 * NN);
#pragma unroll
            for (int i = 0; i < 4; ++i) *(LAS u32x4*)(lds + ((tid >> 4) + 32 * i) * WPITCH + (tid & 15) * 16) = v[i];
            __syncthreads();
            const int a0 = (8 * (lane >> 4) + ((lane & 15) >> 2)) * WPITCH + (wv * 16 + 4 * (lane & 3)) * 2;
#pragma unroll
            for (int f = 0; f < 4; ++f)
#pragma unroll
                for (int h = 0; h < 2; ++h) {
                    const s16x4 r = __builtin_amdgcn_ds_read_tr16_b64_v4i16((LAS s16x4*)(lds + a0 + (f * 32 + h * 4) * WPITCH));
                    const u32x2 rr = __builtin_bit_cast(u32x2, r);
                    d[4 * f + 2 * h] = rr[0]; d[4 * f + 2 * h + 1] = rr[1];
                }
        }
        const Packed pk = pack_chunk(d, mk_rsrc(ws + OFF_FBIN + (size_t)b * FB_UNIT, FB_UNIT), wv * FB_CH, lane, lds + SCR_OFF + wv * SCR_WAVE);
        if (!isw && (g & 15) < NPUSH / 32) {
            for (int j = 0; j < 4; ++j) {
                const u32 dest = (u32)((g >> 4) * 4 + j);
                if (dest != (u32)rank && dest < (u32)NR)
                    store_packed(pk, mk_rsrc(p.ws[dest] + OFF_PUSH + (size_t)rank * NPUSH * CH_BYTES, NPUSH * CH_BYTES),
                                 (u32)(((g & 15) * 4 + q) * 8 + wv) * CH_BYTES, lane);
            }
        } else {
            store_packed(pk, mk_rsrc(ws + OFF_PIN + (size_t)b * UNIT_BYTES, UNIT_BYTES), wv * CH_BYTES, lane);
        }
        const bool mine = isw ? ((g >> 3) == rj) : ((g >> 4) == ri);
        const int item = isw ? 16 + (g & 7) : (g & 15);
        if (mine) {
            const rsrc_t dst = mk_rsrc(ws + (isw ? OFF_WF + ((size_t)(g & 7) << 20) : OFF_XF + ((size_t)(g & 15) << 20)), 1u << 20);
#pragma unroll
            for (int f = 0; f < 4; ++f)
                __builtin_amdgcn_raw_buffer_store_b128((u32x4){d[4 * f], d[4 * f + 1], d[4 * f + 2], d[4 * f + 3]}, dst,
                                                       (u32)(((rank * 16 + q * 4 + f) * 8 + wv) * 1024 + lane * 16), 0, AUX_AGENT);
        }
        drain();
        __syncthreads();
        const u32 call0 = rfl(callv) + 1;
        if (tid < (isw ? 2 : 4)) {
            const u32 dest = isw ? (u32)(tid * 4 + (g >> 3)) : (u32)((g >> 4) * 4 + tid);
            if (dest < (u32)NR) {
                if (dest != (u32)rank) st_sys(p.ws[dest] + OFF_FIN + (size_t)(rank * 256 + b) * 128, call0);
                else __hip_atomic_fetch_add((GAS u32*)(ws + OFF_CNT + (item * 8 + rank) * 128), 8u, __ATOMIC_RELAXED, __HIP_MEMORY_SCOPE_AGENT);
            }
        }
        if (tid == 0) st_agent(ws + OFF_CTR + b * 4, call0);
    }
    const u32 call = rfl(callv) + 1;
    STAMP(ws, b, 1);
    const int xcd = b & 7, slot = b >> 3;

    if (slot < GS) {
        const int h0 = xcd * GS + slot;
        LAS u32* okw = (LAS u32*)(lds + LDS_MISC);
        {
            const u32 i = (u32)h0 + (u32)wv * (8 * GS);
            if (i < 7u * NPUSH) {
                const u32 src = (u32)(rank + 1 + i % 7) & 7u, c = i / 7, uid = ri * 64 + (c >> 3), v = c & 7;
                const bool ok = wait_ge<true>(ws + OFF_FIN + (size_t)(src * 256 + uid) * 128, call, tmo, call);
                asm volatile("" ::: "memory");
                const rsrc_t pr = mk_rsrc(ws + OFF_PUSH + (size_t)(src * NPUSH + c) * CH_BYTES, CH_BYTES);
                const u32x4 s0 = __builtin_amdgcn_raw_buffer_load_b128(pr, lane * 16, 0, AUX_SYS);
                const u32x4 s1 = __builtin_amdgcn_raw_buffer_load_b128(pr, lane * 16 + 1024, 0, AUX_SYS);
                const u32x3 cp = __builtin_amdgcn_raw_buffer_load_b96(pr, lane * 12 + 2048, 0, AUX_SYS);
                const u32 lst = __builtin_amdgcn_raw_buffer_load_b32(pr, (lane & 31) * 4 + LST_OFF, 0, AUX_SYS);
                u32 d[16];
                unpack_chunk(d, s0, s1, cp, lst, mk_rsrc(p.ws[src] + OFF_FBIN, 256 * FB_UNIT), (uid * 8 + v) * FB_CH, lane, ok, lds + wv * SCR_WAVE);
                const rsrc_t dst = mk_rsrc(ws + OFF_XF + ((size_t)(c >> 5) << 20), 1u << 20);
#pragma unroll
                for (int f = 0; f < 4; ++f)
                    __builtin_amdgcn_raw_buffer_store_b128((u32x4){d[4 * f], d[4 * f + 1], d[4 * f + 2], d[4 * f + 3]}, dst,
                                                           (u32)(((src * 16 + ((c >> 3) & 3) * 4 + f) * 8 + v) * 1024 + lane * 16), 0, AUX_AGENT);
                drain();
                if (lane == 0) __hip_atomic_fetch_add((GAS u32*)(ws + OFF_CNT + ((c >> 5) * 8 + src) * 128), 1u, __ATOMIC_RELAXED, __HIP_MEMORY_SCOPE_AGENT);
            }
        }
#define G_BAR() do { __builtin_amdgcn_s_barrier(); asm volatile("" ::: "memory"); } while (0)
        for (int round = 0; round < (h0 < 256 - 8 * GS ? 2 : 1); ++round) {
            int rg, c64;
            half_tile(h0 + round * 8 * GS, rg, c64);
            const int cg = c64 >> 1, hn = c64 & 1;
            if (wv == 0) {
                const u32 e = max(rg < 4 ? rg : rg + 8, 4 + cg);
                while (rtc() - t_start < (u64)(GATE0 + GATE1 * e)) __builtin_amdgcn_s_sleep(8);
                wait_all16(ws + OFF_CNT + (((lane & 8) ? 16 + cg : rg) * 8 + (lane & 7)) * 128, call * 32, tmo, call, lane);
            }
            asm volatile("" ::: "memory"); __builtin_amdgcn_s_barrier(); asm volatile("" ::: "memory");
            STAMP(ws, b, 2 + 3 * round);
            f32x4 acc[4][2];
#pragma unroll
            for (int m = 0; m < 4; ++m) { acc[m][0] = (f32x4){0.f, 0.f, 0.f, 0.f}; acc[m][1] = (f32x4){0.f, 0.f, 0.f, 0.f}; }
            const int wr = (wv >> 1) & 1, wc = wv & 1;
            if (wv >= 4) {
                const int lw = wv - 4;
                const char* xg = ws + OFF_XF + ((size_t)rg << 20) + lw * 4096 + lane * 16;
                const char* wg = ws + OFF_WF + ((size_t)cg << 20) + (lw >> 1) * 8192 + hn * 4096 + (lw & 1) * 2048 + lane * 16;
#define L_ISSUE(s, SL) do { const char* _x = xg + (size_t)(s) * 16384; const char* _w = wg + (size_t)(s) * 16384; \
                _Pragma("unroll") for (int i = 0; i < 4; ++i) __builtin_amdgcn_global_load_lds((const GAS u32*)(_x + i * 1024), (LAS u32*)(lds + (SL) * GSLOT + lw * 4096 + i * 1024), 16, 0, GL_AUX); \
                _Pragma("unroll") for (int i = 0; i < 2; ++i) __builtin_amdgcn_global_load_lds((const GAS u32*)(_w + i * 1024), (LAS u32*)(lds + (SL) * GSLOT + 16384 + lw * 2048 + i * 1024), 16, 0, GL_AUX); } while (0)
#define L_WAIT(n) asm volatile("s_waitcnt vmcnt(" #n ")" ::: "memory")
                L_ISSUE(0, 0); L_ISSUE(1, 1); L_ISSUE(2, 2); L_ISSUE(3, 3);
                L_WAIT(18); G_BAR();
                for (int s = 0; s < 60; s += 4) {
                    L_WAIT(12); G_BAR(); L_ISSUE(s + 4, 0);
                    L_WAIT(12); G_BAR(); L_ISSUE(s + 5, 1);
                    L_WAIT(12); G_BAR(); L_ISSUE(s + 6, 2);
                    L_WAIT(12); G_BAR(); L_ISSUE(s + 7, 3);
                }
                L_WAIT(12); G_BAR(); L_WAIT(6); G_BAR(); L_WAIT(0); G_BAR(); G_BAR();
            } else {
                const int xro = wr * 4096 + lane * 16, wro = 16384 + wc * 2048 + lane * 16;
                bf16x8 xa[2][2][4], wb[2][2][2];
#define C_READ(P, SL) do { _Pragma("unroll") for (int k = 0; k < 2; ++k) { \
                _Pragma("unroll") for (int n = 0; n < 2; ++n) wb[P][k][n] = *(const LAS bf16x8*)(lds + (SL) * GSLOT + wro + k * 4096 + n * 1024); \
                _Pragma("unroll") for (int m = 0; m < 4; ++m) xa[P][k][m] = *(const LAS bf16x8*)(lds + (SL) * GSLOT + xro + k * 8192 + m * 1024); } } while (0)
#define C_MMA(P) do { _Pragma("unroll") for (int k = 0; k < 2; ++k) _Pragma("unroll") for (int m = 0; m < 4; ++m) \
                _Pragma("unroll") for (int n = 0; n < 2; ++n) acc[m][n] = __builtin_amdgcn_mfma_f32_16x16x32_bf16(wb[P][k][n], xa[P][k][m], acc[m][n], 0, 0, 0); } while (0)
#define C_SCHED() do { _Pragma("unroll") for (int i = 0; i < 12; ++i) { __builtin_amdgcn_sched_group_barrier(0x008, 1, 0); __builtin_amdgcn_sched_group_barrier(0x100, 1, 0); } \
                __builtin_amdgcn_sched_group_barrier(0x008, 4, 0); } while (0)
#define C_STEP(SL, P, RD) do { asm volatile("s_waitcnt lgkmcnt(0)" ::: "memory"); G_BAR(); if (RD) C_READ((P) ^ 1, ((SL) + 1) & 3); C_MMA(P); if (RD) C_SCHED(); __builtin_amdgcn_sched_barrier(0); } while (0)
                G_BAR();
                C_READ(0, 0);
                for (int s = 0; s < 60; s += 4) { C_STEP(0, 0, true); C_STEP(1, 1, true); C_STEP(2, 0, true); C_STEP(3, 1, true); }
                C_STEP(0, 0, true); C_STEP(1, 1, true); C_STEP(2, 0, true); C_STEP(3, 1, false);
            }
            G_BAR();
            STAMP(ws, b, 3 + 3 * round);
            if (wv < 4) {
#pragma unroll
                for (int m = 0; m < 4; ++m)
#pragma unroll
                    for (int n = 0; n < 2; ++n) {
                        u32x2 o;
                        asm("v_cvt_pk_bf16_f32 %0, %1, %2" : "=v"(o.x) : "v"(acc[m][n][0]), "v"(acc[m][n][1]));
                        asm("v_cvt_pk_bf16_f32 %0, %1, %2" : "=v"(o.y) : "v"(acc[m][n][2]), "v"(acc[m][n][3]));
                        *(LAS u32x2*)(lds + (wr * 64 + m * 16 + (lane & 15)) * OPITCH + (wc * 32 + n * 16 + (lane >> 4) * 4) * 2) = o;
                    }
            }
            if (tid == 0) okw[0] = (ld_agent(tmo) == call) ? 1u : 0u;
            __syncthreads();
            u32 d[16];
            const int j = wv & 3;
#pragma unroll
            for (int f = 0; f < 4; ++f) {
                const u32x4 v = *(const LAS u32x4*)(lds + (j * 32 + f * 8 + (lane >> 3)) * OPITCH + (lane & 7) * 16);
                d[4 * f] = v[0]; d[4 * f + 1] = v[1]; d[4 * f + 2] = v[2]; d[4 * f + 3] = v[3];
            }
            if (okw[0]) {
#pragma unroll
                for (int i = 0; i < 16; ++i) d[i] = 0x7FC07FC0u;
            }
            const int t = rg < 4 ? cg * 4 + rg : rg * 8 + cg;
            if (wv < 4) {
                const Packed pk = pack_chunk(d, mk_rsrc(ws + OFF_FBOUT + (size_t)t * FB_UNIT, FB_UNIT), (hn * 4 + j) * FB_CH, lane, lds + SCR_OFF + wv * SCR_WAVE);
                store_packed(pk, mk_rsrc(ws + OFF_POUT + (size_t)t * UNIT_BYTES, UNIT_BYTES), (hn * 4 + j) * CH_BYTES, lane);
                drain();
            } else {
                const rsrc_t outr = mk_rsrc(p.out, 4096u * NN * 2);
                const u32 oo = (u32)((ri * 2048 + rg * 128 + j * 32 + (lane >> 3)) * NN + rj * 1024 + c64 * 64 + (lane & 7) * 8) * 2;
#pragma unroll
                for (int f = 0; f < 4; ++f)
                    __builtin_amdgcn_raw_buffer_store_b128((u32x4){d[4 * f], d[4 * f + 1], d[4 * f + 2], d[4 * f + 3]}, outr, oo + f * 8 * NN * 2, 0, OUT_AUX);
            }
            __syncthreads();
            if (tid < NR && tid != rank) st_sys(p.ws[tid] + OFF_FOUT + (size_t)(rank * 256 + t * 2 + hn) * 128, call);
            STAMP(ws, b, 4 + 3 * round);
        }
        STAMPC(ws, b, call, 1);
        return;
    }

    const int pi = (slot - GS) * 8 + xcd;
    {
        const int gw = pi * PWPW + wv;
        if (wv >= PWPW || gw >= 7 * NWL) return;
        const int link = gw % 7, wl = gw / 7;
        const u32 src = rfl((u32)(rank + 1 + link) & 7u);
        if (src >= (u32)NR) return;
        const u64 pws = uni64((u64)p.ws[src]), wsu = uni64((u64)ws);
        LAS unsigned char* scr = lds + wv * SCR_WAVE;
        GAS u32* tick = (GAS u32*)(ws + OFF_TICK + link * 128);
        const u32 tbase = NPUSH + 3 * NWL - (call - 1) * (u32)(NCH - NPUSH - 2 * NWL);
        struct Desc { u32 rem, fb, flg, dst, cnt; };
        const u32 xbase = ri * 64, wbase = 128 + rj * 32;
        const u32 obase = ((src >> 2) * 2048 * NN + (src & 3) * 1024) * 2;
        const u32 fin0 = (u32)OFF_FIN + src * 256 * 128, fout0 = (u32)OFF_FOUT + src * 256 * 128 - IN_UNITS * 256;
        const u32 cnt0 = (u32)OFF_CNT + src * 128, din0 = src * 128 * 1024;
        auto mk = [&](u32 c) -> Desc {
            c = min(c, (u32)NCH - 1);
            const u32 u = c >> 3, v = c & 7, e = u >> 2, m = u - IN_UNITS;
            u32 s4 = e >= 4 ? 4u : 0u, s12 = e >= 12 ? 4u : 0u, dx = e < 4 ? xbase : xbase - 32, dw = wbase - 16;
            u32 rgo = m < 32 ? (m & 3) : (m >> 3), cgo = m < 32 ? (m >> 2) : (m & 7);
            asm("" : "+s"(s4), "+s"(s12), "+s"(dx), "+s"(rgo), "+s"(cgo));
            const bool in = u < IN_UNITS, w = s4 != s12;
            const u32 pan = e - s4 - s12;
            u32 uidi = u + (w ? dw : dx), pbase = w ? (u32)OFF_WF : (u32)OFF_XF, cw = w ? 16u : 0u;
            asm("" : "+s"(uidi), "+s"(pbase), "+s"(cw));
            u32 remi = (u32)OFF_PIN + uidi * UNIT_BYTES, remo = (u32)OFF_POUT + m * UNIT_BYTES;
            u32 fbi = (u32)OFF_FBIN + uidi * FB_UNIT, fbo = (u32)OFF_FBOUT + m * FB_UNIT;
            u32 flgi = fin0 + uidi * 128, flgo = fout0 + u * 256 + (v >> 2) * 128;
            u32 dsti = pbase + (pan << 20) + din0 + ((u & 3) * 32 + v) * 1024;
            u32 dsto = obase + ((rgo * 128 + (v & 3) * 32) * NN + cgo * 128 + (v >> 2) * 64) * 2;
            u32 cnti = cnt0 + (cw + pan) * 1024;
            asm("" : "+s"(remi), "+s"(remo), "+s"(fbi), "+s"(fbo), "+s"(flgi), "+s"(flgo), "+s"(dsti), "+s"(dsto), "+s"(cnti));
            Desc d;
            d.rem = (in ? remi : remo) + v * CH_BYTES;
            d.fb = (in ? fbi : fbo) + v * FB_CH;
            d.flg = in ? flgi : flgo;
            d.dst = in ? dsti : dsto;
            d.cnt = in ? cnti : 0u;
            return d;
        };
        const rsrc_t wsr = mk_rsrc(ws, (u32)WS_NEED), outr = mk_rsrc(p.out, 4096u * NN * 2), pwr = mk_rsrc((const void*)pws, (u32)WS_NEED);
        const u32 vin = lane * 16, vout = (lane >> 3) * (NN * 2) + (lane & 7) * 16, vl4 = LST_OFF + (lane & 31) * 4;
        u32 n1 = NPUSH + NWL + wl, n2 = NPUSH + 2 * NWL + wl;
        Desc D0 = mk(NPUSH + wl), D1 = mk(n1), D2 = mk(n2);
#define P_CLOB "a0", "a1", "a2", "a3", "a4", "a5", "a6", "a7", "a8", "a9", "a10", "a11"
#define P_ISSUE(DA) do { \
            const u64 un_ = pws + (DA).rem; \
            asm volatile("s_nop 4\n\tglobal_load_dwordx4 a[0:3], %0, %2 sc0 sc1\n\tglobal_load_dwordx4 a[4:7], %0, %2 offset:1024 sc0 sc1\n\t" \
                         "global_load_dwordx3 a[8:10], %1, %2 offset:2048 sc0 sc1\n\tglobal_load_dword a11, %3, %2 sc0 sc1" \
                         :: "v"(vin), "v"(lane * 12), "s"(un_), "v"(vl4) : P_CLOB, "memory"); } while (0)
#define P_FLAG(DB) do { const u64 fa_ = wsu + (DB).flg; \
            asm volatile("s_nop 4\n\tglobal_load_dword a12, %0, %1 sc0 sc1" :: "v"(0), "s"(fa_) : "a12", "memory"); } while (0)
#define P_TICKET(want_) do { u64 sv_; const u32 em_ = rfl((want_) ? 1u : 0u); \
            asm volatile("v_accvgpr_write_b32 a14, 1\n\ts_mov_b64 %0, exec\n\ts_mov_b32 exec_lo, %2\n\ts_mov_b32 exec_hi, 0\n\ts_nop 1\n\t" \
                         "global_atomic_add a13, %1, a14, off sc0\n\ts_mov_b64 exec, %0" \
                         : "=&s"(sv_) : "v"(tick), "s"(em_) : "a13", "a14", "memory"); } while (0)
#define P_FETCH() asm volatile("v_accvgpr_read_b32 %0, a0\n\tv_accvgpr_read_b32 %1, a1\n\tv_accvgpr_read_b32 %2, a2\n\tv_accvgpr_read_b32 %3, a3\n\t" \
                         "v_accvgpr_read_b32 %4, a4\n\tv_accvgpr_read_b32 %5, a5\n\tv_accvgpr_read_b32 %6, a6\n\tv_accvgpr_read_b32 %7, a7\n\t" \
                         "v_accvgpr_read_b32 %8, a8\n\tv_accvgpr_read_b32 %9, a9\n\tv_accvgpr_read_b32 %10, a10\n\tv_accvgpr_read_b32 %11, a11\n\t" \
                         "v_accvgpr_read_b32 %12, a12\n\tv_accvgpr_read_b32 %13, a13" \
                         : "=v"(c0.x), "=v"(c0.y), "=v"(c0.z), "=v"(c0.w), "=v"(c1.x), "=v"(c1.y), "=v"(c1.z), "=v"(c1.w), \
                           "=v"(cc.x), "=v"(cc.y), "=v"(cc.z), "=v"(cl), "=v"(fl), "=v"(tk) :: "memory")
#define P_SIGNAL(c_) do { u64 sv_; const u64 ca_ = wsu + (c_); const u32 em_ = rfl((c_) ? 1u : 0u); \
            asm volatile("s_nop 4\n\ts_mov_b64 %0, exec\n\ts_mov_b32 exec_lo, %4\n\ts_mov_b32 exec_hi, 0\n\tglobal_atomic_add %1, %2, %3\n\ts_mov_b64 exec, %0" \
                         : "=&s"(sv_) : "v"(0), "v"(1u), "s"(ca_), "s"(em_) : "memory"); } while (0)
        bool ok = wait_ge<true>(ws + D0.flg, call, tmo, call);
        asm volatile("" ::: "memory");
        P_ISSUE(D0); P_FLAG(D1);
        bool tkv = false;
        drain();
        u32 pendA = 0, pendB = 0;
#define TS() do {} while (0)
        for (;;) {
            asm volatile("s_waitcnt vmcnt(4)" ::: "memory");
            TS();
            u32x4 c0, c1; u32x3 cc; u32 cl, fl, tk;
            P_FETCH();
            const bool cok = ok;
            const bool more = n1 < (u32)NCH;
            const bool early = more && rfl(fl) >= call;
            if (__builtin_expect(early, 1)) { ok = true; asm volatile("" ::: "memory"); P_ISSUE(D1); }
            n2 = tkv ? rfl(tk) + tbase : n2;
            D2 = mk(n2);
            if (__builtin_expect(early, 1)) { P_FLAG(D2); tkv = n2 < (u32)NCH; P_TICKET(tkv); }
            P_SIGNAL(pendA);
            pendA = pendB;
            u32 d[16];
            unpack_chunk(d, c0, c1, cc, cl, pwr, D0.fb, lane, cok, scr);

            {
                const bool isin = D0.cnt != 0;
                const rsrc_t dr = isin ? wsr : outr;
                const u32 vo = isin ? vin : vout, step = isin ? 8192u : 8u * NN * 2;
#pragma unroll
                for (int f = 0; f < 4; ++f)
                    __builtin_amdgcn_raw_buffer_store_b128((u32x4){d[4 * f], d[4 * f + 1], d[4 * f + 2], d[4 * f + 3]}, dr, vo, D0.dst + f * step, AUX_AGENT);
            }
            u32 mine = D0.cnt;
            if (__builtin_expect(!early, 0)) {
                drain(); P_SIGNAL(pendA); P_SIGNAL(mine);
                pendA = 0; mine = 0;
                if (!more) break;
                ok = wait_ge<true>(ws + D1.flg, call, tmo, call);
                asm volatile("" ::: "memory");
                P_ISSUE(D1); P_FLAG(D2); tkv = n2 < (u32)NCH; P_TICKET(tkv);
                drain();
            }
            pendB = mine;
            n1 = n2; if (!tkv) n2 = NCH;
            D0 = D1; D1 = D2;
        }
        STAMPW(ws, b, 4);
        drain();
        STAMPC(ws, b, call, 1);
    }
}
}

#define HIP_CHECK(expr) do { hipError_t _e = (expr); if (_e != hipSuccess) { \
    fprintf(stderr, "HIP error %s at %s:%d\n", hipGetErrorString(_e), __FILE__, __LINE__); \
    return; } } while (0)

extern "C" void kernel_launch(int rank, int nranks, ncclComm_t comm,
                              hipStream_t stream,
                              void* const* d_in, const int* in_sizes, int n_in,
                              void* d_out, int out_size,
                              const int* in_dtypes, int out_dtype,
                              void* const* d_out_all,
                              void* const* d_ws_all, size_t ws_bytes) {
    (void)comm; (void)in_sizes; (void)n_in; (void)out_size; (void)in_dtypes; (void)out_dtype; (void)d_out_all;
    if (nranks != NR || ws_bytes < WS_NEED || rank < 0 || rank >= NR) { fprintf(stderr, "unexpected configuration\n"); return; }
    static bool attr_done[64] = {};
    int dev = 0;
    HIP_CHECK(hipGetDevice(&dev));
    if (dev >= 0 && dev < 64 && !attr_done[dev]) {
        HIP_CHECK(hipFuncSetAttribute((const void*)fused_gemm_ar, hipFuncAttributeMaxDynamicSharedMemorySize, LDS_BYTES));
        attr_done[dev] = true;
    }
    Params p;
    p.x = (const u16*)d_in[0];
    p.w = (const u16*)d_in[1];
    p.out = (u16*)d_out;
    for (int i = 0; i < NR; ++i) p.ws[i] = (char*)d_ws_all[i];
    p.rank = rank; p.pad = 0;
    hipLaunchKernelGGL(fused_gemm_ar, dim3(GRID), dim3(NT), LDS_BYTES, stream, p);
    HIP_CHECK(hipGetLastError());
}
```

```cpp
#include <hip/hip_runtime.h>
#include <rccl/rccl.h>
#include <cstdio>
#include <cstdint>

namespace {
typedef unsigned u32;
typedef unsigned long long u64;
typedef unsigned short u16;
typedef u32 u32x4 __attribute__((ext_vector_type(4)));
typedef u32 u32x2 __attribute__((ext_vector_type(2)));
typedef u32 u32x3 __attribute__((ext_vector_type(3)));
typedef short bf16x8 __attribute__((ext_vector_type(8)));
typedef short s16x4 __attribute__((ext_vector_type(4)));
typedef u16 u16x2 __attribute__((ext_vector_type(2)));
typedef float f32x4 __attribute__((ext_vector_type(4)));
#define LAS __attribute__((address_space(3)))
#define GAS __attribute__((address_space(1)))

constexpr int NR = 8;
constexpr int NN = 4096, KS = 512;
constexpr int NT = 512, GRID = 256;
#define PULL_W 26
constexpr int NWL = PULL_W;
#define PULL_SLOTS 8
constexpr int GS = 32 - PULL_SLOTS;
static_assert(GS % 2 == 0 && GS >= 16, "rows");
constexpr int NPULL = 8 * PULL_SLOTS;
#define PULL_WPW 3
constexpr int PWPW = PULL_WPW;
static_assert(7 * NWL <= NPULL * PWPW, "puller waves");
constexpr u64 DEADLINE = 40000000ull;

constexpr u32 CH_BYTES = 2816, UNIT_BYTES = 8 * CH_BYTES;
constexpr u32 FB_CH = 2048, FB_UNIT = 8 * FB_CH;
constexpr int IN_UNITS = 96, OUT_UNITS = 128, ALL_UNITS = IN_UNITS + OUT_UNITS;
constexpr int NCH = ALL_UNITS * 8;
#define PUSH_N 64
constexpr int NPUSH = PUSH_N;

constexpr size_t OFF_CTR = 0;
constexpr size_t OFF_TMO = 4096;
constexpr size_t OFF_TICK = 5120;
constexpr size_t OFF_CNT = 8192;
constexpr size_t OFF_FIN = OFF_CNT + 24 * 8 * 128;
constexpr size_t OFF_FOUT = OFF_FIN + (size_t)8 * 256 * 128;
constexpr size_t OFF_DBG = OFF_FOUT + (size_t)8 * 256 * 128;
constexpr size_t OFF_DBG2 = OFF_DBG + 65536;
constexpr size_t OFF_DBG3 = OFF_DBG2 + 16384;
constexpr size_t OFF_PIN = OFF_DBG3 + 384 * 512;
constexpr size_t OFF_PUSH = OFF_PIN + (size_t)256 * UNIT_BYTES;
constexpr size_t OFF_POUT = OFF_PUSH + (size_t)8 * NPUSH * CH_BYTES;
constexpr size_t OFF_FBIN = OFF_POUT + (size_t)128 * UNIT_BYTES;
constexpr size_t OFF_FBOUT = OFF_FBIN + (size_t)256 * FB_UNIT;
constexpr size_t OFF_XF = OFF_FBOUT + (size_t)128 * FB_UNIT;
constexpr size_t OFF_WF = OFF_XF + ((size_t)16 << 20);
constexpr size_t WS_NEED = OFF_WF + ((size_t)8 << 20);
static_assert(OFF_PIN % 128 == 0 && OFF_XF % 1024 == 0, "alignment");

constexpr int LDS_BYTES = 131072 + 1024;
constexpr int LDS_MISC = 131072;
constexpr int GSLOT = 24576;
constexpr int WPITCH = 288;
constexpr int OPITCH = 144;
constexpr int SCR_OFF = 65536, SCR_R = 2048, SCR_POS = 2304, SCR_SEL = 2432, SCR_CMP = 2496, SCR_WAVE = 2560;

struct Params {
    const u16* x; const u16* w; u16* out;
    char* ws[NR];
    int rank; int pad;
};

typedef __amdgpu_buffer_rsrc_t rsrc_t;
__device__ __forceinline__ rsrc_t mk_rsrc(const void* p, u32 bytes) {
    return __builtin_amdgcn_make_buffer_rsrc((void*)p, 0, bytes, 0x00020000);
}
constexpr int AUX_SYS = 17, AUX_AGENT = 16;
#define GATE0 1000
#define GATE1 150
#define POLL_SLEEP 8
#define PULL_GATE 0
#define GL_AUX 17
#define OUT_AUX 16
__device__ __forceinline__ u64 rtc() { return __builtin_amdgcn_s_memrealtime(); }
__device__ __forceinline__ u32 rfl(u32 v) { return __builtin_amdgcn_readfirstlane(v); }
__device__ __forceinline__ u64 uni64(u64 v) { return ((u64)rfl((u32)(v >> 32)) << 32) | rfl((u32)v); }
__device__ __forceinline__ u32 bfi(u32 m, u32 a, u32 b) { return (m & a) | (~m & b); }
__device__ __forceinline__ u32 perm(u32 a, u32 b, u32 s) { return __builtin_amdgcn_perm(a, b, s); }
__device__ __forceinline__ void drain() { asm volatile("s_waitcnt vmcnt(0)" ::: "memory"); }
__device__ __forceinline__ u32 ld_sys(const char* p) { return __hip_atomic_load((const GAS u32*)p, __ATOMIC_RELAXED, __HIP_MEMORY_SCOPE_SYSTEM); }
__device__ __forceinline__ u32 ld_agent(const char* p) { return __hip_atomic_load((const GAS u32*)p, __ATOMIC_RELAXED, __HIP_MEMORY_SCOPE_AGENT); }
__device__ __forceinline__ void st_sys(char* p, u32 v) { __hip_atomic_store((GAS u32*)p, v, __ATOMIC_RELAXED, __HIP_MEMORY_SCOPE_SYSTEM); }
__device__ __forceinline__ void st_agent(char* p, u32 v) { __hip_atomic_store((GAS u32*)p, v, __ATOMIC_RELAXED, __HIP_MEMORY_SCOPE_AGENT); }

#define STAMPC(ws, b, call, i) do {} while (0)
#define STAMP(ws, b, i) do {} while (0)
#define STAMPW(ws, b, i) do {} while (0)

template <bool SYS>
__device__ __forceinline__ bool wait_ge(const char* flag, u32 want, char* tmo, u32 call) {
    const u64 t0 = rtc();
    for (u32 n = 1;; ++n) {
        const u32 v = rfl(SYS ? ld_sys(flag) : ld_agent(flag));
        if (v >= want) return true;
        if ((n & 7) == 0 && (rfl(ld_agent(tmo)) == call || rtc() - t0 > DEADLINE)) { st_agent(tmo, call); return false; }
        __builtin_amdgcn_s_sleep(1);
    }
}
__device__ __forceinline__ bool wait_all16(const char* cnt_lane, u32 want, char* tmo, u32 call, int lane) {
    const u64 t0 = rtc();
    for (u32 n = 1;; ++n) {
        const u32 v = lane < 16 ? ld_agent(cnt_lane) : want;
        if (__all(v >= want)) return true;
        if ((n & 7) == 0 && (rfl(ld_agent(tmo)) == call || rtc() - t0 > DEADLINE)) { st_agent(tmo, call); return false; }
        __builtin_amdgcn_s_sleep(POLL_SLEEP);
    }
}

#define DPP_ADD(v, ctrl, rm) v += (u32)__builtin_amdgcn_update_dpp(0, (int)(v), ctrl, rm, 0xF, true)
__device__ __forceinline__ u32 wave_sum(u32 v) {
    DPP_ADD(v, 0xB1, 0xF); DPP_ADD(v, 0x4E, 0xF); DPP_ADD(v, 0x141, 0xF); DPP_ADD(v, 0x140, 0xF);
    return (__builtin_amdgcn_readlane(v, 0) + __builtin_amdgcn_readlane(v, 16)) + (__builtin_amdgcn_readlane(v, 32) + __builtin_amdgcn_readlane(v, 48));
}
__device__ __forceinline__ u32 wave_scan(u32 v) {
    DPP_ADD(v, 0x111, 0xF); DPP_ADD(v, 0x112, 0xF); DPP_ADD(v, 0x114, 0xF); DPP_ADD(v, 0x118, 0xF);
    DPP_ADD(v, 0x142, 0xA); DPP_ADD(v, 0x143, 0xC);
    return v;
}
__device__ __forceinline__ void init_tables(LAS unsigned char* scr, int lane) {
    if (lane < 16) {
        const u32 m = ((u32)lane * 0x00204081u) & 0x01010101u;
        ((LAS u32*)(scr + SCR_SEL))[lane] = bfi((m << 8) - m, (m << 8) + (m << 16) + (m << 24) + 0x04040404u, 0x03020100u);
        u32 c = 0x0C0C0C0Cu; int q = 0;
#pragma unroll
        for (int b = 0; b < 4; ++b) if ((lane >> b) & 1) { c = bfi(0xFFu << (8 * q), (u32)b << (8 * q), c); ++q; }
        ((LAS u32*)(scr + SCR_CMP))[lane] = c;
    }
    asm volatile("" ::: "memory");
}
struct Packed { u32x4 s0, s1; u32x2 ab; u32 r; };
__device__ __forceinline__ void store_packed(const Packed& k, rsrc_t dst, u32 off, int lane) {
    __builtin_amdgcn_raw_buffer_store_b128(k.s0, dst, lane * 16, off, AUX_SYS);
    __builtin_amdgcn_raw_buffer_store_b128(k.s1, dst, lane * 16 + 1024, off, AUX_SYS);
    __builtin_amdgcn_raw_buffer_store_b64(k.ab, dst, lane * 8 + 2048, off, AUX_SYS);
    __builtin_amdgcn_raw_buffer_store_b32(k.r, dst, lane * 4 + 2560, off, AUX_SYS);
}
__device__ __forceinline__ Packed pack_chunk(const u32 (&d)[16], rsrc_t fb, u32 fbo, int lane, LAS unsigned char* scr) {
    u32 SM[8], E[8], sum = 0;
#pragma unroll
    for (int t = 0; t < 8; ++t) {
        const u32 lo = perm(d[2 * t + 1], d[2 * t], 0x06040200u), hi = perm(d[2 * t + 1], d[2 * t], 0x07050301u);
        SM[t] = bfi(0x7F7F7F7Fu, lo, hi);
        E[t] = bfi(0xFEFEFEFEu, hi << 1, lo >> 7);
        sum = __builtin_amdgcn_udot4(E[t], 0x01010101u, sum, false);
    }
    const u32 base = min((wave_sum(sum) + 4035u) >> 11, 255u);
    const u16x2 b2 = {(u16)base, (u16)base}, c1 = {1, 1}, c3 = {3, 3}, c11 = {11, 11};
    u32 A = 0, B = 0, T = 0, k = 0;
    u64 acc = 0;
#pragma unroll
    for (int t = 0; t < 8; ++t) {
        const u16x2 da = b2 - __builtin_bit_cast(u16x2, E[t] & 0x00FF00FFu), db = b2 - __builtin_bit_cast(u16x2, perm(E[t], E[t], 0x0C030C01u));
        const u32 c2 = __builtin_bit_cast(u32, __builtin_elementwise_min(da, c3)) | (__builtin_bit_cast(u32, __builtin_elementwise_min(db, c3)) << 8);
        const u16x2 sa = __builtin_elementwise_max(__builtin_elementwise_min((u16x2)(da + c1), c11), c3) - c3;
        const u16x2 sb = __builtin_elementwise_max(__builtin_elementwise_min((u16x2)(db + c1), c11), c3) - c3;
        const u32 s8 = __builtin_bit_cast(u32, sa) | (__builtin_bit_cast(u32, sb) << 8);
        const u32 m = c2 & (c2 >> 1) & 0x01010101u;
        T |= ((s8 >> 3) & 0x01010101u) << t;
        const u32 cs = *(const LAS u32*)(scr + SCR_CMP + __builtin_amdgcn_udot4(m, 0x20100804u, 0u, false));
        const u32 x = perm(0u, s8 & 0x07070707u, cs);
        u32 v = (x | (x >> 5)) & 0x003F003Fu;
        v = (v | (v >> 10)) & 0xFFFu;
        acc |= (u64)v << min(3u * k, 63u);
        k += __builtin_popcount(m);
        if (t < 4) A |= c2 << (2 * t); else B |= c2 << (2 * (t - 4));
    }
    const u32 nt = __builtin_popcount(T);
    const u32 incl = wave_scan(k | (nt << 16));
    const u32 tot = __builtin_amdgcn_readlane(incl, 63), K = tot & 0xFFFFu;
    u32 cnt = tot >> 16, R = 0;
    bool raw = __ballot(k > 21) != 0 || cnt > 63 || 32 + 3 * K > 16 * (128 - cnt);
    if (!raw) {
        LAS u32* rl = (LAS u32*)(scr + SCR_R);
        rl[lane] = lane == 0 ? (base | (cnt << 8)) : 0u;
        asm volatile("" ::: "memory");
        const u32 off = 32 + 3 * ((incl & 0xFFFFu) - k), i = off >> 5, sh = off & 31;
        const u32 lo = (u32)acc, hi = (u32)(acc >> 32);
        __hip_atomic_fetch_or(rl + i, lo << sh, __ATOMIC_RELAXED, __HIP_MEMORY_SCOPE_WORKGROUP);
        __hip_atomic_fetch_or(rl + ((i + 1) & 63), (u32)((acc << sh) >> 32), __ATOMIC_RELAXED, __HIP_MEMORY_SCOPE_WORKGROUP);
        __hip_atomic_fetch_or(rl + ((i + 2) & 63), (hi >> 1) >> (31 - sh), __ATOMIC_RELAXED, __HIP_MEMORY_SCOPE_WORKGROUP);
        bool bad = false;
        if (cnt) {
            *(LAS u32x4*)(scr + lane * 32) = (u32x4){E[0], E[1], E[2], E[3]};
            *(LAS u32x4*)(scr + lane * 32 + 16) = (u32x4){E[4], E[5], E[6], E[7]};
            u32 idx = (incl >> 16) - nt, M = T;
            while (M) {
                const u32 bit = __builtin_ctz(M); M &= M - 1;
                ((LAS u16*)(scr + SCR_POS))[idx & 63] = (u16)(lane * 32 + (bit & 7) * 4 + (bit >> 3));
                ++idx;
            }
            asm volatile("" ::: "memory");
            if ((u32)lane < cnt) {
                const u32 pos = ((const LAS u16*)(scr + SCR_POS))[lane] & 2047u, e = scr[pos];
                const int dl = (int)base - (int)e;
                bad = e != 0 && (dl > 24 || dl < -17);
                const u32 c5 = e == 0 ? 15u : (dl > 0 ? (u32)(dl - 10) : (u32)(14 - dl));
                ((LAS u16*)(scr + SCR_R))[127 - lane] = (u16)((pos << 5) | (c5 & 31u));
            }
        }
        asm volatile("" ::: "memory");
        R = rl[lane];
        raw = __ballot(bad) != 0;
    }
    if (raw) {
        R = lane == 0 ? (base | (255u << 8)) : 0u;
        __builtin_amdgcn_raw_buffer_store_b128((u32x4){E[0], E[1], E[2], E[3]}, fb, lane * 16, fbo, AUX_SYS);
        __builtin_amdgcn_raw_buffer_store_b128((u32x4){E[4], E[5], E[6], E[7]}, fb, lane * 16 + 1024, fbo, AUX_SYS);
    }
    return Packed{(u32x4){SM[0], SM[1], SM[2], SM[3]}, (u32x4){SM[4], SM[5], SM[6], SM[7]}, (u32x2){A, B}, R};
}

__device__ __forceinline__ void unpack_chunk(u32 (&d)[16], u32x4 s0, u32x4 s1, u32x2 ab, u32 R, rsrc_t fb, u32 fbo, int lane, bool ok,
                                             LAS unsigned char* scr) {
    const u32 h0 = __builtin_amdgcn_readlane(R, 0);
    const u32 cnt = ok ? (h0 >> 8) & 0xFFu : 0u, base = h0 & 0xFFu;
    u32 E[8];
    if (__builtin_expect(cnt == 255, 0)) {
        u32x4 a = __builtin_amdgcn_raw_buffer_load_b128(fb, lane * 16, fbo, AUX_SYS);
        u32x4 b = __builtin_amdgcn_raw_buffer_load_b128(fb, lane * 16 + 1024, fbo, AUX_SYS);
        asm volatile("" : "+v"(a), "+v"(b));
#pragma unroll
        for (int t = 0; t < 4; ++t) { E[t] = a[t]; E[4 + t] = b[t]; }
    } else {
        const u32 B4 = base * 0x01010101u;
        const u32 lutp = B4 - 0x03020100u, luts_lo = bfi(0xFFu, base + 1, B4 - 0x05040300u), luts_hi = B4 - 0x09080706u;
        const u32 k = __builtin_popcount(ab[0] & (ab[0] >> 1) & 0x55555555u) + __builtin_popcount(ab[1] & (ab[1] >> 1) & 0x55555555u);
        LAS u32* rl = (LAS u32*)(scr + SCR_R);
        rl[lane] = R;
        asm volatile("" ::: "memory");
        const u32 off = 32 + 3 * (wave_scan(k) - k), i = min(off >> 5, 61u);
        const u32 w0 = rl[i], w1 = rl[i + 1], w2 = rl[i + 2];
        const u64 win = ((u64)__builtin_amdgcn_alignbit(w2, w1, off) << 32) | __builtin_amdgcn_alignbit(w1, w0, off);
        u32 r3 = 0, m3 = 0x03030303u, m1 = 0x01010101u;
        asm("" : "+s"(m3), "+s"(m1));
#pragma unroll
        for (int t = 0; t < 8; ++t) {
            const u32 c2 = (ab[t >> 2] >> (2 * (t & 3))) & m3;
            const u32 m = c2 & (c2 >> 1) & m1;
            const u32 sel = *(const LAS u32*)(scr + SCR_SEL + __builtin_amdgcn_udot4(m, 0x20100804u, 0u, false));
            const u32 F = (u32)(win >> (r3 & 63));
            const u32 x = bfi(0x3Fu, F, F << 10);
            const u32 u = bfi(0x00070007u, x, x << 5) & 0x07070707u;
            E[t] = perm(perm(luts_hi, luts_lo, u), perm(lutp, lutp, c2), sel);
            r3 += 3 * __builtin_popcount(m);
        }
        if (__builtin_expect(cnt != 0, 1)) {
            *(LAS u32x4*)(scr + lane * 32) = (u32x4){E[0], E[1], E[2], E[3]};
            *(LAS u32x4*)(scr + lane * 32 + 16) = (u32x4){E[4], E[5], E[6], E[7]};
            asm volatile("" ::: "memory");
#pragma unroll
            for (int h = 0; h < 2; ++h) {
                const u32 ent = (R >> (16 * h)) & 0xFFFFu, c5 = ent & 31u;
                const u32 e = c5 < 16 ? (c5 == 15 ? 0u : base - 10 - c5) : base + c5 - 14;
                if ((u32)(127 - 2 * lane - h) < cnt) scr[ent >> 5] = (unsigned char)e;
            }
            asm volatile("" ::: "memory");
            const u32x4 a = *(const LAS u32x4*)(scr + lane * 32), b = *(const LAS u32x4*)(scr + lane * 32 + 16);
#pragma unroll
            for (int t = 0; t < 4; ++t) { E[t] = a[t]; E[4 + t] = b[t]; }
        }
    }
#pragma unroll
    for (int t = 0; t < 8; ++t) {
        const u32 sm = t < 4 ? s0[t & 3] : s1[t & 3];
        const u32 lo = bfi(0x7F7F7F7Fu, sm, E[t] << 7), hi = bfi(0x80808080u, sm, E[t] >> 1);
        d[2 * t] = perm(hi, lo, 0x05010400u); d[2 * t + 1] = perm(hi, lo, 0x07030602u);
    }
    if (__builtin_expect(!ok, 0)) {
        asm volatile("" ::: "memory");
#pragma unroll
        for (int i = 0; i < 16; ++i) d[i] = 0x7FC07FC0u;
    }
}

__host__ __device__ __forceinline__ void tile_order(u32 m, u32& rg, u32& cg) {
    rg = m < 32 ? (m & 3) : (m >> 3);
    cg = m < 32 ? (m >> 2) : (m & 7);
}
__host__ __device__ __forceinline__ void half_tile(int h, int& rg, int& c64) {
    if (h < 64) { rg = h & 3; c64 = h >> 2; }
    else if (h < 192) { const int k = (h - 64) >> 5, i = h & 31; rg = 4 + (k >> 1) * 4 + (i >> 3); c64 = (k & 1) * 8 + (i & 7); }
    else { rg = 12 + ((h - 192) >> 4); c64 = h & 15; }
}

__global__ __launch_bounds__(NT, 2) void fused_gemm_ar(Params p) {
    extern __shared__ __attribute__((aligned(16))) unsigned char smem[];
    LAS unsigned char* lds = (LAS unsigned char*)smem;
    const int tid = threadIdx.x, lane = tid & 63, wv = (int)rfl(tid >> 6);
    const int b = blockIdx.x, rank = p.rank, ri = rank >> 2, rj = rank & 3;
    char* ws = p.ws[rank];
    char* tmo = ws + OFF_TMO;
    STAMP(ws, b, 0);
    const u64 t_start = rtc();
    const u32 callv = ld_agent(ws + OFF_CTR + b * 4);

    {
        u32 d[16];
        const int isw = b >> 7, g = (b & 127) >> 2, q = b & 3;
        if (!isw) {
            const u16* src = p.x + (size_t)(g * 128 + wv * 16 + (lane & 15)) * KS + q * 128 + (lane >> 4) * 8;
#pragma unroll
            for (int f = 0; f < 4; ++f) {
                const u32x4 v = *(const GAS u32x4*)(src + f * 32);
                d[4 * f] = v[0]; d[4 * f + 1] = v[1]; d[4 * f + 2] = v[2]; d[4 * f + 3] = v[3];
            }
        } else {
            const u16* src = p.w + (size_t)(q * 128 + (tid >> 4)) * NN + g * 128 + (tid & 15) * 8;
            u32x4 v[4];
#pragma unroll
            for (int i = 0; i < 4; ++i) v[i] = *(const GAS u32x4*)(src + (size_t)i * 32 * NN);
#pragma unroll
            for (int i = 0; i < 4; ++i) *(LAS u32x4*)(lds + ((tid >> 4) + 32 * i) * WPITCH + (tid & 15) * 16) = v[i];
            __syncthreads();
            const int a0 = (8 * (lane >> 4) + ((lane & 15) >> 2)) * WPITCH + (wv * 16 + 4 * (lane & 3)) * 2;
#pragma unroll
            for (int f = 0; f < 4; ++f)
#pragma unroll
                for (int h = 0; h < 2; ++h) {
                    const s16x4 r = __builtin_amdgcn_ds_read_tr16_b64_v4i16((LAS s16x4*)(lds + a0 + (f * 32 + h * 4) * WPITCH));
                    const u32x2 rr = __builtin_bit_cast(u32x2, r);
                    d[4 * f + 2 * h] = rr[0]; d[4 * f + 2 * h + 1] = rr[1];
                }
        }
        init_tables(lds + SCR_OFF + wv * SCR_WAVE, lane);
        const Packed pk = pack_chunk(d, mk_rsrc(ws + OFF_FBIN + (size_t)b * FB_UNIT, FB_UNIT), wv * FB_CH, lane, lds + SCR_OFF + wv * SCR_WAVE);
        if (!isw && (g & 15) < NPUSH / 32) {
            for (int j = 0; j < 4; ++j) {
                const u32 dest = (u32)((g >> 4) * 4 + j);
                if (dest != (u32)rank && dest < (u32)NR)
                    store_packed(pk, mk_rsrc(p.ws[dest] + OFF_PUSH + (size_t)rank * NPUSH * CH_BYTES, NPUSH * CH_BYTES),
                                 (u32)(((g & 15) * 4 + q) * 8 + wv) * CH_BYTES, lane);
            }
        } else {
            store_packed(pk, mk_rsrc(ws + OFF_PIN + (size_t)b * UNIT_BYTES, UNIT_BYTES), wv * CH_BYTES, lane);
        }
        const bool mine = isw ? ((g >> 3) == rj) : ((g >> 4) == ri);
        const int item = isw ? 16 + (g & 7) : (g & 15);
        if (mine) {
            const rsrc_t dst = mk_rsrc(ws + (isw ? OFF_WF + ((size_t)(g & 7) << 20) : OFF_XF + ((size_t)(g & 15) << 20)), 1u << 20);
#pragma unroll
            for (int f = 0; f < 4; ++f)
                __builtin_amdgcn_raw_buffer_store_b128((u32x4){d[4 * f], d[4 * f + 1], d[4 * f + 2], d[4 * f + 3]}, dst,
                                                       (u32)(((rank * 16 + q * 4 + f) * 8 + wv) * 1024 + lane * 16), 0, AUX_AGENT);
        }
        drain();
        __syncthreads();
        const u32 call0 = rfl(callv) + 1;
        if (tid < (isw ? 2 : 4)) {
            const u32 dest = isw ? (u32)(tid * 4 + (g >> 3)) : (u32)((g >> 4) * 4 + tid);
            if (dest < (u32)NR) {
                if (dest != (u32)rank) st_sys(p.ws[dest] + OFF_FIN + (size_t)(rank * 256 + b) * 128, call0);
                else __hip_atomic_fetch_add((GAS u32*)(ws + OFF_CNT + (item * 8 + rank) * 128), 8u, __ATOMIC_RELAXED, __HIP_MEMORY_SCOPE_AGENT);
            }
        }
        if (tid == 0) st_agent(ws + OFF_CTR + b * 4, call0);
    }
    const u32 call = rfl(callv) + 1;
    STAMP(ws, b, 1);
    const int xcd = b & 7, slot = b >> 3;

    if (slot < GS) {
        const int h0 = xcd * GS + slot;
        LAS u32* okw = (LAS u32*)(lds + LDS_MISC);
        {
            const u32 i = (u32)h0 + (u32)wv * (8 * GS);
            if (i < 7u * NPUSH) {
                const u32 src = (u32)(rank + 1 + i % 7) & 7u, c = i / 7, uid = ri * 64 + (c >> 3), v = c & 7;
                const bool ok = wait_ge<true>(ws + OFF_FIN + (size_t)(src * 256 + uid) * 128, call, tmo, call);
                asm volatile("" ::: "memory");
                const rsrc_t pr = mk_rsrc(ws + OFF_PUSH + (size_t)(src * NPUSH + c) * CH_BYTES, CH_BYTES);
                const u32x4 s0 = __builtin_amdgcn_raw_buffer_load_b128(pr, lane * 16, 0, AUX_SYS);
                const u32x4 s1 = __builtin_amdgcn_raw_buffer_load_b128(pr, lane * 16 + 1024, 0, AUX_SYS);
                const u32x2 cp = __builtin_amdgcn_raw_buffer_load_b64(pr, lane * 8 + 2048, 0, AUX_SYS);
                const u32 lst = __builtin_amdgcn_raw_buffer_load_b32(pr, lane * 4 + 2560, 0, AUX_SYS);
                u32 d[16];
                init_tables(lds + wv * SCR_WAVE, lane);
                unpack_chunk(d, s0, s1, cp, lst, mk_rsrc(p.ws[src] + OFF_FBIN, 256 * FB_UNIT), (uid * 8 + v) * FB_CH, lane, ok, lds + wv * SCR_WAVE);
                const rsrc_t dst = mk_rsrc(ws + OFF_XF + ((size_t)(c >> 5) << 20), 1u << 20);
#pragma unroll
                for (int f = 0; f < 4; ++f)
                    __builtin_amdgcn_raw_buffer_store_b128((u32x4){d[4 * f], d[4 * f + 1], d[4 * f + 2], d[4 * f + 3]}, dst,
                                                           (u32)(((src * 16 + ((c >> 3) & 3) * 4 + f) * 8 + v) * 1024 + lane * 16), 0, AUX_AGENT);
                drain();
                if (lane == 0) __hip_atomic_fetch_add((GAS u32*)(ws + OFF_CNT + ((c >> 5) * 8 + src) * 128), 1u, __ATOMIC_RELAXED, __HIP_MEMORY_SCOPE_AGENT);
            }
        }
#define G_BAR() do { __builtin_amdgcn_s_barrier(); asm volatile("" ::: "memory"); } while (0)
        for (int round = 0; round < (h0 < 256 - 8 * GS ? 2 : 1); ++round) {
            int rg, c64;
            half_tile(h0 + round * 8 * GS, rg, c64);
            const int cg = c64 >> 1, hn = c64 & 1;
            if (wv == 0) {
                const u32 e = max(rg < 4 ? rg : rg + 8, 4 + cg);
                while (rtc() - t_start < (u64)(GATE0 + GATE1 * e)) __builtin_amdgcn_s_sleep(8);
                wait_all16(ws + OFF_CNT + (((lane & 8) ? 16 + cg : rg) * 8 + (lane & 7)) * 128, call * 32, tmo, call, lane);
            }
            asm volatile("" ::: "memory"); __builtin_amdgcn_s_barrier(); asm volatile("" ::: "memory");
            STAMP(ws, b, 2 + 3 * round);
            f32x4 acc[4][2];
#pragma unroll
            for (int m = 0; m < 4; ++m) { acc[m][0] = (f32x4){0.f, 0.f, 0.f, 0.f}; acc[m][1] = (f32x4){0.f, 0.f, 0.f, 0.f}; }
            const int wr = (wv >> 1) & 1, wc = wv & 1;
            if (wv >= 4) {
                const int lw = wv - 4;
                const char* xg = ws + OFF_XF + ((size_t)rg << 20) + lw * 4096 + lane * 16;
                const char* wg = ws + OFF_WF + ((size_t)cg << 20) + (lw >> 1) * 8192 + hn * 4096 + (lw & 1) * 2048 + lane * 16;
#define L_ISSUE(s, SL) do { const char* _x = xg + (size_t)(s) * 16384; const char* _w = wg + (size_t)(s) * 16384; \
                _Pragma("unroll") for (int i = 0; i < 4; ++i) __builtin_amdgcn_global_load_lds((const GAS u32*)(_x + i * 1024), (LAS u32*)(lds + (SL) * GSLOT + lw * 4096 + i * 1024), 16, 0, GL_AUX); \
                _Pragma("unroll") for (int i = 0; i < 2; ++i) __builtin_amdgcn_global_load_lds((const GAS u32*)(_w + i * 1024), (LAS u32*)(lds + (SL) * GSLOT + 16384 + lw * 2048 + i * 1024), 16, 0, GL_AUX); } while (0)
#define L_WAIT(n) asm volatile("s_waitcnt vmcnt(" #n ")" ::: "memory")
                L_ISSUE(0, 0); L_ISSUE(1, 1); L_ISSUE(2, 2); L_ISSUE(3, 3);
                L_WAIT(18); G_BAR();
                for (int s = 0; s < 60; s += 4) {
                    L_WAIT(12); G_BAR(); L_ISSUE(s + 4, 0);
                    L_WAIT(12); G_BAR(); L_ISSUE(s + 5, 1);
                    L_WAIT(12); G_BAR(); L_ISSUE(s + 6, 2);
                    L_WAIT(12); G_BAR(); L_ISSUE(s + 7, 3);
                }
                L_WAIT(12); G_BAR(); L_WAIT(6); G_BAR(); L_WAIT(0); G_BAR(); G_BAR();
            } else {
                const int xro = wr * 4096 + lane * 16, wro = 16384 + wc * 2048 + lane * 16;
                bf16x8 xa[2][2][4], wb[2][2][2];
#define C_READ(P, SL) do { _Pragma("unroll") for (int k = 0; k < 2; ++k) { \
                _Pragma("unroll") for (int n = 0; n < 2; ++n) wb[P][k][n] = *(const LAS bf16x8*)(lds + (SL) * GSLOT + wro + k * 4096 + n * 1024); \
                _Pragma("unroll") for (int m = 0; m < 4; ++m) xa[P][k][m] = *(const LAS bf16x8*)(lds + (SL) * GSLOT + xro + k * 8192 + m * 1024); } } while (0)
#define C_MMA(P) do { _Pragma("unroll") for (int k = 0; k < 2; ++k) _Pragma("unroll") for (int m = 0; m < 4; ++m) \
                _Pragma("unroll") for (int n = 0; n < 2; ++n) acc[m][n] = __builtin_amdgcn_mfma_f32_16x16x32_bf16(wb[P][k][n], xa[P][k][m], acc[m][n], 0, 0, 0); } while (0)
#define C_SCHED() do { _Pragma("unroll") for (int i = 0; i < 12; ++i) { __builtin_amdgcn_sched_group_barrier(0x008, 1, 0); __builtin_amdgcn_sched_group_barrier(0x100, 1, 0); } \
                __builtin_amdgcn_sched_group_barrier(0x008, 4, 0); } while (0)
#define C_STEP(SL, P, RD) do { asm volatile("s_waitcnt lgkmcnt(0)" ::: "memory"); G_BAR(); if (RD) C_READ((P) ^ 1, ((SL) + 1) & 3); C_MMA(P); if (RD) C_SCHED(); __builtin_amdgcn_sched_barrier(0); } while (0)
                G_BAR();
                C_READ(0, 0);
                for (int s = 0; s < 60; s += 4) { C_STEP(0, 0, true); C_STEP(1, 1, true); C_STEP(2, 0, true); C_STEP(3, 1, true); }
                C_STEP(0, 0, true); C_STEP(1, 1, true); C_STEP(2, 0, true); C_STEP(3, 1, false);
            }
            G_BAR();
            STAMP(ws, b, 3 + 3 * round);
            if (wv < 4) {
#pragma unroll
                for (int m = 0; m < 4; ++m)
#pragma unroll
                    for (int n = 0; n < 2; ++n) {
                        u32x2 o;
                        asm("v_cvt_pk_bf16_f32 %0, %1, %2" : "=v"(o.x) : "v"(acc[m][n][0]), "v"(acc[m][n][1]));
                        asm("v_cvt_pk_bf16_f32 %0, %1, %2" : "=v"(o.y) : "v"(acc[m][n][2]), "v"(acc[m][n][3]));
                        *(LAS u32x2*)(lds + (wr * 64 + m * 16 + (lane & 15)) * OPITCH + (wc * 32 + n * 16 + (lane >> 4) * 4) * 2) = o;
                    }
            }
            if (tid == 0) okw[0] = (ld_agent(tmo) == call) ? 1u : 0u;
            __syncthreads();
            u32 d[16];
            const int j = wv & 3;
#pragma unroll
            for (int f = 0; f < 4; ++f) {
                const u32x4 v = *(const LAS u32x4*)(lds + (j * 32 + f * 8 + (lane >> 3)) * OPITCH + (lane & 7) * 16);
                d[4 * f] = v[0]; d[4 * f + 1] = v[1]; d[4 * f + 2] = v[2]; d[4 * f + 3] = v[3];
            }
            if (okw[0]) {
#pragma unroll
                for (int i = 0; i < 16; ++i) d[i] = 0x7FC07FC0u;
            }
            const int t = rg < 4 ? cg * 4 + rg : rg * 8 + cg;
            if (wv < 4) {
                init_tables(lds + SCR_OFF + wv * SCR_WAVE, lane);
                const Packed pk = pack_chunk(d, mk_rsrc(ws + OFF_FBOUT + (size_t)t * FB_UNIT, FB_UNIT), (hn * 4 + j) * FB_CH, lane, lds + SCR_OFF + wv * SCR_WAVE);
                store_packed(pk, mk_rsrc(ws + OFF_POUT + (size_t)t * UNIT_BYTES, UNIT_BYTES), (hn * 4 + j) * CH_BYTES, lane);
                drain();
            } else {
                const rsrc_t outr = mk_rsrc(p.out, 4096u * NN * 2);
                const u32 oo = (u32)((ri * 2048 + rg * 128 + j * 32 + (lane >> 3)) * NN + rj * 1024 + c64 * 64 + (lane & 7) * 8) * 2;
#pragma unroll
                for (int f = 0; f < 4; ++f)
                    __builtin_amdgcn_raw_buffer_store_b128((u32x4){d[4 * f], d[4 * f + 1], d[4 * f + 2], d[4 * f + 3]}, outr, oo + f * 8 * NN * 2, 0, OUT_AUX);
            }
            __syncthreads();
            if (tid < NR && tid != rank) st_sys(p.ws[tid] + OFF_FOUT + (size_t)(rank * 256 + t * 2 + hn) * 128, call);
            STAMP(ws, b, 4 + 3 * round);
        }
        STAMPC(ws, b, call, 1);
        return;
    }

    const int pi = (slot - GS) * 8 + xcd;
    {
        const int gw = pi * PWPW + wv;
        if (wv >= PWPW || gw >= 7 * NWL) return;
        const int link = gw % 7, wl = gw / 7;
        const u32 src = rfl((u32)(rank + 1 + link) & 7u);
        if (src >= (u32)NR) return;
        const u64 pws = uni64((u64)p.ws[src]), wsu = uni64((u64)ws);
        LAS unsigned char* scr = lds + wv * SCR_WAVE;
        init_tables(scr, lane);
        GAS u32* tick = (GAS u32*)(ws + OFF_TICK + link * 128);
        const u32 tbase = NPUSH + 3 * NWL - (call - 1) * (u32)(NCH - NPUSH - 2 * NWL);
        struct Desc { u32 rem, fb, flg, dst, cnt; };
        const u32 xbase = ri * 64, wbase = 128 + rj * 32;
        const u32 obase = ((src >> 2) * 2048 * NN + (src & 3) * 1024) * 2;
        const u32 fin0 = (u32)OFF_FIN + src * 256 * 128, fout0 = (u32)OFF_FOUT + src * 256 * 128 - IN_UNITS * 256;
        const u32 cnt0 = (u32)OFF_CNT + src * 128, din0 = src * 128 * 1024;
        auto mk = [&](u32 c) -> Desc {
            c = min(c, (u32)NCH - 1);
            const u32 u = c >> 3, v = c & 7, e = u >> 2, m = u - IN_UNITS;
            u32 s4 = e >= 4 ? 4u : 0u, s12 = e >= 12 ? 4u : 0u, dx = e < 4 ? xbase : xbase - 32, dw = wbase - 16;
            u32 rgo = m < 32 ? (m & 3) : (m >> 3), cgo = m < 32 ? (m >> 2) : (m & 7);
            asm("" : "+s"(s4), "+s"(s12), "+s"(dx), "+s"(rgo), "+s"(cgo));
            const bool in = u < IN_UNITS, w = s4 != s12;
            const u32 pan = e - s4 - s12;
            u32 uidi = u + (w ? dw : dx), pbase = w ? (u32)OFF_WF : (u32)OFF_XF, cw = w ? 16u : 0u;
            asm("" : "+s"(uidi), "+s"(pbase), "+s"(cw));
            u32 remi = (u32)OFF_PIN + uidi * UNIT_BYTES, remo = (u32)OFF_POUT + m * UNIT_BYTES;
            u32 fbi = (u32)OFF_FBIN + uidi * FB_UNIT, fbo = (u32)OFF_FBOUT + m * FB_UNIT;
            u32 flgi = fin0 + uidi * 128, flgo = fout0 + u * 256 + (v >> 2) * 128;
            u32 dsti = pbase + (pan << 20) + din0 + ((u & 3) * 32 + v) * 1024;
            u32 dsto = obase + ((rgo * 128 + (v & 3) * 32) * NN + cgo * 128 + (v >> 2) * 64) * 2;
            u32 cnti = cnt0 + (cw + pan) * 1024;
            asm("" : "+s"(remi), "+s"(remo), "+s"(fbi), "+s"(fbo), "+s"(flgi), "+s"(flgo), "+s"(dsti), "+s"(dsto), "+s"(cnti));
            Desc d;
            d.rem = (in ? remi : remo) + v * CH_BYTES;
            d.fb = (in ? fbi : fbo) + v * FB_CH;
            d.flg = in ? flgi : flgo;
            d.dst = in ? dsti : dsto;
            d.cnt = in ? cnti : 0u;
            return d;
        };
        const rsrc_t wsr = mk_rsrc(ws, (u32)WS_NEED), outr = mk_rsrc(p.out, 4096u * NN * 2), pwr = mk_rsrc((const void*)pws, (u32)WS_NEED);
        const u32 vin = lane * 16, vout = (lane >> 3) * (NN * 2) + (lane & 7) * 16, vl4 = 2560 + lane * 4;
        u32 n1 = NPUSH + NWL + wl, n2 = NPUSH + 2 * NWL + wl;
        Desc D0 = mk(NPUSH + wl), D1 = mk(n1), D2 = mk(n2);
#define P_CLOB "a0", "a1", "a2", "a3", "a4", "a5", "a6", "a7", "a8", "a9", "a10", "a11"
#define P_ISSUE(DA) do { \
            const u64 un_ = pws + (DA).rem; \
            asm volatile("s_nop 4\n\tglobal_load_dwordx4 a[0:3], %0, %2 sc0 sc1\n\tglobal_load_dwordx4 a[4:7], %0, %2 offset:1024 sc0 sc1\n\t" \
                         "global_load_dwordx2 a[8:9], %1, %2 offset:2048 sc0 sc1\n\tglobal_load_dword a11, %3, %2 sc0 sc1" \
                         :: "v"(vin), "v"(lane * 8), "s"(un_), "v"(vl4) : P_CLOB, "memory"); } while (0)
#define P_FLAG(DB) do { const u64 fa_ = wsu + (DB).flg; \
            asm volatile("s_nop 4\n\tglobal_load_dword a12, %0, %1 sc0 sc1" :: "v"(0), "s"(fa_) : "a12", "memory"); } while (0)
#define P_TICKET(want_) do { u64 sv_; const u32 em_ = rfl((want_) ? 1u : 0u); \
            asm volatile("v_accvgpr_write_b32 a14, 1\n\ts_mov_b64 %0, exec\n\ts_mov_b32 exec_lo, %2\n\ts_mov_b32 exec_hi, 0\n\ts_nop 1\n\t" \
                         "global_atomic_add a13, %1, a14, off sc0\n\ts_mov_b64 exec, %0" \
                         : "=&s"(sv_) : "v"(tick), "s"(em_) : "a13", "a14", "memory"); } while (0)
#define P_FETCH() asm volatile("v_accvgpr_read_b32 %0, a0\n\tv_accvgpr_read_b32 %1, a1\n\tv_accvgpr_read_b32 %2, a2\n\tv_accvgpr_read_b32 %3, a3\n\t" \
                         "v_accvgpr_read_b32 %4, a4\n\tv_accvgpr_read_b32 %5, a5\n\tv_accvgpr_read_b32 %6, a6\n\tv_accvgpr_read_b32 %7, a7\n\t" \
                         "v_accvgpr_read_b32 %8, a8\n\tv_accvgpr_read_b32 %9, a9\n\tv_accvgpr_read_b32 %10, a11\n\t" \
                         "v_accvgpr_read_b32 %11, a12\n\tv_accvgpr_read_b32 %12, a13" \
                         : "=v"(c0.x), "=v"(c0.y), "=v"(c0.z), "=v"(c0.w), "=v"(c1.x), "=v"(c1.y), "=v"(c1.z), "=v"(c1.w), \
                           "=v"(cc.x), "=v"(cc.y), "=v"(cl), "=v"(fl), "=v"(tk) :: "memory")
#define P_SIGNAL(c_) do { u64 sv_; const u64 ca_ = wsu + (c_); const u32 em_ = rfl((c_) ? 1u : 0u); \
            asm volatile("s_nop 4\n\ts_mov_b64 %0, exec\n\ts_mov_b32 exec_lo, %4\n\ts_mov_b32 exec_hi, 0\n\tglobal_atomic_add %1, %2, %3\n\ts_mov_b64 exec, %0" \
                         : "=&s"(sv_) : "v"(0), "v"(1u), "s"(ca_), "s"(em_) : "memory"); } while (0)
        bool ok = wait_ge<true>(ws + D0.flg, call, tmo, call);
        asm volatile("" ::: "memory");
        P_ISSUE(D0); P_FLAG(D1);
        bool tkv = false;
        drain();
        u32 pendA = 0, pendB = 0;
#define TS() do {} while (0)
        for (;;) {
            asm volatile("s_waitcnt vmcnt(4)" ::: "memory");
            TS();
            u32x4 c0, c1; u32x2 cc; u32 cl, fl, tk;
            P_FETCH();
            const bool cok = ok;
            const bool more = n1 < (u32)NCH;
            const bool early = more && rfl(fl) >= call;
            if (__builtin_expect(early, 1)) { ok = true; asm volatile("" ::: "memory"); P_ISSUE(D1); }
            n2 = tkv ? rfl(tk) + tbase : n2;
            D2 = mk(n2);
            if (__builtin_expect(early, 1)) { P_FLAG(D2); tkv = n2 < (u32)NCH; P_TICKET(tkv); }
            P_SIGNAL(pendA);
            pendA = pendB;
            u32 d[16];
            unpack_chunk(d, c0, c1, cc, cl, pwr, D0.fb, lane, cok, scr);

            {
                const bool isin = D0.cnt != 0;
                const rsrc_t dr = isin ? wsr : outr;
                const u32 vo = isin ? vin : vout, step = isin ? 8192u : 8u * NN * 2;
#pragma unroll
                for (int f = 0; f < 4; ++f)
                    __builtin_amdgcn_raw_buffer_store_b128((u32x4){d[4 * f], d[4 * f + 1], d[4 * f + 2], d[4 * f + 3]}, dr, vo, D0.dst + f * step, AUX_AGENT);
            }
            u32 mine = D0.cnt;
            if (__builtin_expect(!early, 0)) {
                drain(); P_SIGNAL(pendA); P_SIGNAL(mine);
                pendA = 0; mine = 0;
                if (!more) break;
                ok = wait_ge<true>(ws + D1.flg, call, tmo, call);
                asm volatile("" ::: "memory");
                P_ISSUE(D1); P_FLAG(D2); tkv = n2 < (u32)NCH; P_TICKET(tkv);
                drain();
            }
            pendB = mine;
            n1 = n2; if (!tkv) n2 = NCH;
            D0 = D1; D1 = D2;
        }
        STAMPW(ws, b, 4);
        drain();
        STAMPC(ws, b, call, 1);
    }
}
}

#define HIP_CHECK(expr) do { hipError_t _e = (expr); if (_e != hipSuccess) { \
    fprintf(stderr, "HIP error %s at %s:%d\n", hipGetErrorString(_e), __FILE__, __LINE__); \
    return; } } while (0)

extern "C" void kernel_launch(int rank, int nranks, ncclComm_t comm,
                              hipStream_t stream,
                              void* const* d_in, const int* in_sizes, int n_in,
                              void* d_out, int out_size,
                              const int* in_dtypes, int out_dtype,
                              void* const* d_out_all,
                              void* const* d_ws_all, size_t ws_bytes) {
    (void)comm; (void)in_sizes; (void)n_in; (void)out_size; (void)in_dtypes; (void)out_dtype; (void)d_out_all;
    if (nranks != NR || ws_bytes < WS_NEED || rank < 0 || rank >= NR) { fprintf(stderr, "unexpected configuration\n"); return; }
    static bool attr_done[64] = {};
    int dev = 0;
    HIP_CHECK(hipGetDevice(&dev));
    if (dev >= 0 && dev < 64 && !attr_done[dev]) {
        HIP_CHECK(hipFuncSetAttribute((const void*)fused_gemm_ar, hipFuncAttributeMaxDynamicSharedMemorySize, LDS_BYTES));
        attr_done[dev] = true;
    }
    Params p;
    p.x = (const u16*)d_in[0];
    p.w = (const u16*)d_in[1];
    p.out = (u16*)d_out;
    for (int i = 0; i < NR; ++i) p.ws[i] = (char*)d_ws_all[i];
    p.rank = rank; p.pad = 0;
    hipLaunchKernelGGL(fused_gemm_ar, dim3(GRID), dim3(NT), LDS_BYTES, stream, p);
    HIP_CHECK(hipGetLastError());
}
```

```cpp
#include <hip/hip_runtime.h>
#include <rccl/rccl.h>
#include <cstdio>
#include <cstdint>

namespace {
typedef unsigned u32;
typedef unsigned long long u64;
typedef unsigned short u16;
typedef u32 u32x4 __attribute__((ext_vector_type(4)));
typedef u32 u32x2 __attribute__((ext_vector_type(2)));
typedef u32 u32x3 __attribute__((ext_vector_type(3)));
typedef short bf16x8 __attribute__((ext_vector_type(8)));
typedef short s16x4 __attribute__((ext_vector_type(4)));
typedef u16 u16x2 __attribute__((ext_vector_type(2)));
typedef float f32x4 __attribute__((ext_vector_type(4)));
#define LAS __attribute__((address_space(3)))
#define GAS __attribute__((address_space(1)))

constexpr int NR = 8;
constexpr int NN = 4096, KS = 512;
constexpr int NT = 512, GRID = 256;
#define PULL_W 26
constexpr int NWL = PULL_W;
#define PULL_SLOTS 8
constexpr int GS = 32 - PULL_SLOTS;
static_assert(GS % 2 == 0 && GS >= 16, "rows");
constexpr int NPULL = 8 * PULL_SLOTS;
#define PULL_WPW 3
constexpr int PWPW = PULL_WPW;
static_assert(7 * NWL <= NPULL * PWPW, "puller waves");
constexpr u64 DEADLINE = 40000000ull;

constexpr u32 CH_BYTES = 2816, UNIT_BYTES = 8 * CH_BYTES;
constexpr u32 FB_CH = 2048, FB_UNIT = 8 * FB_CH;
constexpr int IN_UNITS = 96, OUT_UNITS = 128, ALL_UNITS = IN_UNITS + OUT_UNITS;
constexpr int NCH = ALL_UNITS * 8;
#define PUSH_N 64
constexpr int NPUSH = PUSH_N;

constexpr size_t OFF_CTR = 0;
constexpr size_t OFF_TMO = 4096;
constexpr size_t OFF_TICK = 5120;
constexpr size_t OFF_CNT = 8192;
constexpr size_t OFF_FIN = OFF_CNT + 24 * 8 * 128;
constexpr size_t OFF_FOUT = OFF_FIN + (size_t)8 * 256 * 128;
constexpr size_t OFF_DBG = OFF_FOUT + (size_t)8 * 256 * 128;
constexpr size_t OFF_DBG2 = OFF_DBG + 65536;
constexpr size_t OFF_DBG3 = OFF_DBG2 + 16384;
constexpr size_t OFF_PIN = OFF_DBG3 + 384 * 512;
constexpr size_t OFF_PUSH = OFF_PIN + (size_t)256 * UNIT_BYTES;
constexpr size_t OFF_POUT = OFF_PUSH + (size_t)8 * NPUSH * CH_BYTES;
constexpr size_t OFF_FBIN = OFF_POUT + (size_t)128 * UNIT_BYTES;
constexpr size_t OFF_FBOUT = OFF_FBIN + (size_t)256 * FB_UNIT;
constexpr size_t OFF_XF = OFF_FBOUT + (size_t)128 * FB_UNIT;
constexpr size_t OFF_WF = OFF_XF + ((size_t)16 << 20);
constexpr size_t WS_NEED = OFF_WF + ((size_t)8 << 20);
static_assert(OFF_PIN % 128 == 0 && OFF_XF % 1024 == 0, "alignment");

constexpr int LDS_BYTES = 131072 + 1024;
constexpr int LDS_MISC = 131072;
constexpr int GSLOT = 24576;
constexpr int WPITCH = 288;
constexpr int OPITCH = 144;
constexpr int SCR_OFF = 65536, SCR_R = 2048, SCR_POS = 2304, SCR_SEL = 2432, SCR_CMP = 2496, SCR_WAVE = 2560;

struct Params {
    const u16* x; const u16* w; u16* out;
    char* ws[NR];
    int rank; int pad;
};

typedef __amdgpu_buffer_rsrc_t rsrc_t;
__device__ __forceinline__ rsrc_t mk_rsrc(const void* p, u32 bytes) {
    return __builtin_amdgcn_make_buffer_rsrc((void*)p, 0, bytes, 0x00020000);
}
constexpr int AUX_SYS = 17, AUX_AGENT = 16;
#define GATE0 1000
#define GATE1 150
#define POLL_SLEEP 8
#define PULL_GATE 0
#define GL_AUX 17
#define OUT_AUX 16
__device__ __forceinline__ u64 rtc() { return __builtin_amdgcn_s_memrealtime(); }
__device__ __forceinline__ u32 rfl(u32 v) { return __builtin_amdgcn_readfirstlane(v); }
__device__ __forceinline__ u64 uni64(u64 v) { return ((u64)rfl((u32)(v >> 32)) << 32) | rfl((u32)v); }
__device__ __forceinline__ u32 bfi(u32 m, u32 a, u32 b) { return (m & a) | (~m & b); }
__device__ __forceinline__ u32 perm(u32 a, u32 b, u32 s) { return __builtin_amdgcn_perm(a, b, s); }
__device__ __forceinline__ void drain() { asm volatile("s_waitcnt vmcnt(0)" ::: "memory"); }
__device__ __forceinline__ u32 ld_sys(const char* p) { return __hip_atomic_load((const GAS u32*)p, __ATOMIC_RELAXED, __HIP_MEMORY_SCOPE_SYSTEM); }
__device__ __forceinline__ u32 ld_agent(const char* p) { return __hip_atomic_load((const GAS u32*)p, __ATOMIC_RELAXED, __HIP_MEMORY_SCOPE_AGENT); }
__device__ __forceinline__ void st_sys(char* p, u32 v) { __hip_atomic_store((GAS u32*)p, v, __ATOMIC_RELAXED, __HIP_MEMORY_SCOPE_SYSTEM); }
__device__ __forceinline__ void st_agent(char* p, u32 v) { __hip_atomic_store((GAS u32*)p, v, __ATOMIC_RELAXED, __HIP_MEMORY_SCOPE_AGENT); }

#define STAMPC(ws, b, call, i) do {} while (0)
#define STAMP(ws, b, i) do {} while (0)
#define STAMPW(ws, b, i) do {} while (0)

template <bool SYS>
__device__ __forceinline__ bool wait_ge(const char* flag, u32 want, char* tmo, u32 call) {
    const u64 t0 = rtc();
    for (u32 n = 1;; ++n) {
        const u32 v = rfl(SYS ? ld_sys(flag) : ld_agent(flag));
        if (v >= want) return true;
        if ((n & 7) == 0 && (rfl(ld_agent(tmo)) == call || rtc() - t0 > DEADLINE)) { st_agent(tmo, call); return false; }
        __builtin_amdgcn_s_sleep(1);
    }
}
__device__ __forceinline__ bool wait_all16(const char* cnt_lane, u32 want, char* tmo, u32 call, int lane) {
    const u64 t0 = rtc();
    for (u32 n = 1;; ++n) {
        const u32 v = lane < 16 ? ld_agent(cnt_lane) : want;
        if (__all(v >= want)) return true;
        if ((n & 7) == 0 && (rfl(ld_agent(tmo)) == call || rtc() - t0 > DEADLINE)) { st_agent(tmo, call); return false; }
        __builtin_amdgcn_s_sleep(POLL_SLEEP);
    }
}

#define DPP_ADD(v, ctrl, rm) v += (u32)__builtin_amdgcn_update_dpp(0, (int)(v), ctrl, rm, 0xF, true)
__device__ __forceinline__ u32 wave_sum(u32 v) {
    DPP_ADD(v, 0xB1, 0xF); DPP_ADD(v, 0x4E, 0xF); DPP_ADD(v, 0x141, 0xF); DPP_ADD(v, 0x140, 0xF);
    return (__builtin_amdgcn_readlane(v, 0) + __builtin_amdgcn_readlane(v, 16)) + (__builtin_amdgcn_readlane(v, 32) + __builtin_amdgcn_readlane(v, 48));
}
__device__ __forceinline__ u32 wave_scan(u32 v) {
    DPP_ADD(v, 0x111, 0xF); DPP_ADD(v, 0x112, 0xF); DPP_ADD(v, 0x114, 0xF); DPP_ADD(v, 0x118, 0xF);
    DPP_ADD(v, 0x142, 0xA); DPP_ADD(v, 0x143, 0xC);
    return v;
}
__device__ __forceinline__ void init_tables(LAS unsigned char* scr, int lane) {
    if (lane < 16) {
        const u32 m = ((u32)lane * 0x00204081u) & 0x01010101u;
        ((LAS u32*)(scr + SCR_SEL))[lane] = bfi((m << 8) - m, (m << 8) + (m << 16) + (m << 24) + 0x04040404u, 0x03020100u);
        u32 c = 0x0C0C0C0Cu; int q = 0;
#pragma unroll
        for (int b = 0; b < 4; ++b) if ((lane >> b) & 1) { c = bfi(0xFFu << (8 * q), (u32)b << (8 * q), c); ++q; }
        ((LAS u32*)(scr + SCR_CMP))[lane] = c;
    }
    asm volatile("" ::: "memory");
}
struct Packed { u32x4 s0, s1; u32x2 ab; u32 r; };
__device__ __forceinline__ void store_sm(const Packed& k, rsrc_t dst, u32 off, int lane) {
    __builtin_amdgcn_raw_buffer_store_b128(k.s0, dst, lane * 16, off, AUX_SYS);
    __builtin_amdgcn_raw_buffer_store_b128(k.s1, dst, lane * 16 + 1024, off, AUX_SYS);
}
__device__ __forceinline__ void store_exp(const Packed& k, rsrc_t dst, u32 off, int lane) {
    __builtin_amdgcn_raw_buffer_store_b64(k.ab, dst, lane * 8 + 2048, off, AUX_SYS);
    __builtin_amdgcn_raw_buffer_store_b32(k.r, dst, lane * 4 + 2560, off, AUX_SYS);
}
__device__ __forceinline__ void pack_sm(const u32 (&d)[16], Packed& k, u32 (&E)[8]) {
    u32 SM[8];
#pragma unroll
    for (int t = 0; t < 8; ++t) {
        const u32 lo = perm(d[2 * t + 1], d[2 * t], 0x06040200u), hi = perm(d[2 * t + 1], d[2 * t], 0x07050301u);
        SM[t] = bfi(0x7F7F7F7Fu, lo, hi);
        E[t] = bfi(0xFEFEFEFEu, hi << 1, lo >> 7);
    }
    k.s0 = (u32x4){SM[0], SM[1], SM[2], SM[3]}; k.s1 = (u32x4){SM[4], SM[5], SM[6], SM[7]};
}
__device__ __forceinline__ void pack_exp(const u32 (&E)[8], Packed& pk, rsrc_t fb, u32 fbo, int lane, LAS unsigned char* scr) {
    u32 sum = 0;
#pragma unroll
    for (int t = 0; t < 8; ++t) sum = __builtin_amdgcn_udot4(E[t], 0x01010101u, sum, false);
    const u32 base = min((wave_sum(sum) + 4035u) >> 11, 255u);
    const u16x2 b2 = {(u16)base, (u16)base}, c1 = {1, 1}, c3 = {3, 3}, c11 = {11, 11};
    u32 A = 0, B = 0, T = 0, k = 0;
    u64 acc = 0;
#pragma unroll
    for (int t = 0; t < 8; ++t) {
        const u16x2 da = b2 - __builtin_bit_cast(u16x2, E[t] & 0x00FF00FFu), db = b2 - __builtin_bit_cast(u16x2, perm(E[t], E[t], 0x0C030C01u));
        const u32 c2 = __builtin_bit_cast(u32, __builtin_elementwise_min(da, c3)) | (__builtin_bit_cast(u32, __builtin_elementwise_min(db, c3)) << 8);
        const u16x2 sa = __builtin_elementwise_max(__builtin_elementwise_min((u16x2)(da + c1), c11), c3) - c3;
        const u16x2 sb = __builtin_elementwise_max(__builtin_elementwise_min((u16x2)(db + c1), c11), c3) - c3;
        const u32 s8 = __builtin_bit_cast(u32, sa) | (__builtin_bit_cast(u32, sb) << 8);
        const u32 m = c2 & (c2 >> 1) & 0x01010101u;
        T |= ((s8 >> 3) & 0x01010101u) << t;
        const u32 cs = *(const LAS u32*)(scr + SCR_CMP + __builtin_amdgcn_udot4(m, 0x20100804u, 0u, false));
        const u32 x = perm(0u, s8 & 0x07070707u, cs);
        u32 v = (x | (x >> 5)) & 0x003F003Fu;
        v = (v | (v >> 10)) & 0xFFFu;
        acc |= (u64)v << min(3u * k, 63u);
        k += __builtin_popcount(m);
        if (t < 4) A |= c2 << (2 * t); else B |= c2 << (2 * (t - 4));
    }
    const u32 nt = __builtin_popcount(T);
    const u32 incl = wave_scan(k | (nt << 16));
    const u32 tot = __builtin_amdgcn_readlane(incl, 63), K = tot & 0xFFFFu;
    u32 cnt = tot >> 16, R = 0;
    bool raw = __ballot(k > 21) != 0 || cnt > 63 || 32 + 3 * K > 16 * (128 - cnt);
    if (!raw) {
        LAS u32* rl = (LAS u32*)(scr + SCR_R);
        rl[lane] = lane == 0 ? (base | (cnt << 8)) : 0u;
        asm volatile("" ::: "memory");
        const u32 off = 32 + 3 * ((incl & 0xFFFFu) - k), i = off >> 5, sh = off & 31;
        const u32 lo = (u32)acc, hi = (u32)(acc >> 32);
        __hip_atomic_fetch_or(rl + i, lo << sh, __ATOMIC_RELAXED, __HIP_MEMORY_SCOPE_WORKGROUP);
        __hip_atomic_fetch_or(rl + ((i + 1) & 63), (u32)((acc << sh) >> 32), __ATOMIC_RELAXED, __HIP_MEMORY_SCOPE_WORKGROUP);
        __hip_atomic_fetch_or(rl + ((i + 2) & 63), (hi >> 1) >> (31 - sh), __ATOMIC_RELAXED, __HIP_MEMORY_SCOPE_WORKGROUP);
        bool bad = false;
        if (cnt) {
            *(LAS u32x4*)(scr + lane * 32) = (u32x4){E[0], E[1], E[2], E[3]};
            *(LAS u32x4*)(scr + lane * 32 + 16) = (u32x4){E[4], E[5], E[6], E[7]};
            u32 idx = (incl >> 16) - nt, M = T;
            while (M) {
                const u32 bit = __builtin_ctz(M); M &= M - 1;
                ((LAS u16*)(scr + SCR_POS))[idx & 63] = (u16)(lane * 32 + (bit & 7) * 4 + (bit >> 3));
                ++idx;
            }
            asm volatile("" ::: "memory");
            if ((u32)lane < cnt) {
                const u32 pos = ((const LAS u16*)(scr + SCR_POS))[lane] & 2047u, e = scr[pos];
                const int dl = (int)base - (int)e;
                bad = e != 0 && (dl > 24 || dl < -17);
                const u32 c5 = e == 0 ? 15u : (dl > 0 ? (u32)(dl - 10) : (u32)(14 - dl));
                ((LAS u16*)(scr + SCR_R))[127 - lane] = (u16)((pos << 5) | (c5 & 31u));
            }
        }
        asm volatile("" ::: "memory");
        R = rl[lane];
        raw = __ballot(bad) != 0;
    }
    if (raw) {
        R = lane == 0 ? (base | (255u << 8)) : 0u;
        __builtin_amdgcn_raw_buffer_store_b128((u32x4){E[0], E[1], E[2], E[3]}, fb, lane * 16, fbo, AUX_SYS);
        __builtin_amdgcn_raw_buffer_store_b128((u32x4){E[4], E[5], E[6], E[7]}, fb, lane * 16 + 1024, fbo, AUX_SYS);
    }
    pk.ab = (u32x2){A, B}; pk.r = R;
}

__device__ __forceinline__ void unpack_chunk(u32 (&d)[16], u32x4 s0, u32x4 s1, u32x2 ab, u32 R, rsrc_t fb, u32 fbo, int lane, bool ok,
                                             LAS unsigned char* scr) {
    const u32 h0 = __builtin_amdgcn_readlane(R, 0);
    const u32 cnt = ok ? (h0 >> 8) & 0xFFu : 0u, base = h0 & 0xFFu;
    u32 E[8];
    if (__builtin_expect(cnt == 255, 0)) {
        u32x4 a = __builtin_amdgcn_raw_buffer_load_b128(fb, lane * 16, fbo, AUX_SYS);
        u32x4 b = __builtin_amdgcn_raw_buffer_load_b128(fb, lane * 16 + 1024, fbo, AUX_SYS);
        asm volatile("" : "+v"(a), "+v"(b));
#pragma unroll
        for (int t = 0; t < 4; ++t) { E[t] = a[t]; E[4 + t] = b[t]; }
    } else {
        const u32 B4 = base * 0x01010101u;
        const u32 lutp = B4 - 0x03020100u, luts_lo = bfi(0xFFu, base + 1, B4 - 0x05040300u), luts_hi = B4 - 0x09080706u;
        const u32 k = __builtin_popcount(ab[0] & (ab[0] >> 1) & 0x55555555u) + __builtin_popcount(ab[1] & (ab[1] >> 1) & 0x55555555u);
        LAS u32* rl = (LAS u32*)(scr + SCR_R);
        rl[lane] = R;
        asm volatile("" ::: "memory");
        const u32 off = 32 + 3 * (wave_scan(k) - k), i = min(off >> 5, 61u);
        const u32 w0 = rl[i], w1 = rl[i + 1], w2 = rl[i + 2];
        const u64 win = ((u64)__builtin_amdgcn_alignbit(w2, w1, off) << 32) | __builtin_amdgcn_alignbit(w1, w0, off);
        u32 r3 = 0, m3 = 0x03030303u, m1 = 0x01010101u;
        asm("" : "+s"(m3), "+s"(m1));
#pragma unroll
        for (int t = 0; t < 8; ++t) {
            const u32 c2 = (ab[t >> 2] >> (2 * (t & 3))) & m3;
            const u32 m = c2 & (c2 >> 1) & m1;
            const u32 sel = *(const LAS u32*)(scr + SCR_SEL + __builtin_amdgcn_udot4(m, 0x20100804u, 0u, false));
            const u32 F = (u32)(win >> (r3 & 63));
            const u32 x = bfi(0x3Fu, F, F << 10);
            const u32 u = bfi(0x00070007u, x, x << 5) & 0x07070707u;
            E[t] = perm(perm(luts_hi, luts_lo, u), perm(lutp, lutp, c2), sel);
            r3 += 3 * __builtin_popcount(m);
        }
        if (__builtin_expect(cnt != 0, 1)) {
            *(LAS u32x4*)(scr + lane * 32) = (u32x4){E[0], E[1], E[2], E[3]};
            *(LAS u32x4*)(scr + lane * 32 + 16) = (u32x4){E[4], E[5], E[6], E[7]};
            asm volatile("" ::: "memory");
#pragma unroll
            for (int h = 0; h < 2; ++h) {
                const u32 ent = (R >> (16 * h)) & 0xFFFFu, c5 = ent & 31u;
                const u32 e = c5 < 16 ? (c5 == 15 ? 0u : base - 10 - c5) : base + c5 - 14;
                if ((u32)(127 - 2 * lane - h) < cnt) scr[ent >> 5] = (unsigned char)e;
            }
            asm volatile("" ::: "memory");
            const u32x4 a = *(const LAS u32x4*)(scr + lane * 32), b = *(const LAS u32x4*)(scr + lane * 32 + 16);
#pragma unroll
            for (int t = 0; t < 4; ++t) { E[t] = a[t]; E[4 + t] = b[t]; }
        }
    }
#pragma unroll
    for (int t = 0; t < 8; ++t) {
        const u32 sm = t < 4 ? s0[t & 3] : s1[t & 3];
        const u32 lo = bfi(0x7F7F7F7Fu, sm, E[t] << 7), hi = bfi(0x80808080u, sm, E[t] >> 1);
        d[2 * t] = perm(hi, lo, 0x05010400u); d[2 * t + 1] = perm(hi, lo, 0x07030602u);
    }
    if (__builtin_expect(!ok, 0)) {
        asm volatile("" ::: "memory");
#pragma unroll
        for (int i = 0; i < 16; ++i) d[i] = 0x7FC07FC0u;
    }
}

__host__ __device__ __forceinline__ void tile_order(u32 m, u32& rg, u32& cg) {
    rg = m < 32 ? (m & 3) : (m >> 3);
    cg = m < 32 ? (m >> 2) : (m & 7);
}
__host__ __device__ __forceinline__ void half_tile(int h, int& rg, int& c64) {
    if (h < 64) { rg = h & 3; c64 = h >> 2; }
    else if (h < 192) { const int k = (h - 64) >> 5, i = h & 31; rg = 4 + (k >> 1) * 4 + (i >> 3); c64 = (k & 1) * 8 + (i & 7); }
    else { rg = 12 + ((h - 192) >> 4); c64 = h & 15; }
}

__global__ __launch_bounds__(NT, 2) void fused_gemm_ar(Params p) {
    extern __shared__ __attribute__((aligned(16))) unsigned char smem[];
    LAS unsigned char* lds = (LAS unsigned char*)smem;
    const int tid = threadIdx.x, lane = tid & 63, wv = (int)rfl(tid >> 6);
    const int b = blockIdx.x, rank = p.rank, ri = rank >> 2, rj = rank & 3;
    char* ws = p.ws[rank];
    char* tmo = ws + OFF_TMO;
    STAMP(ws, b, 0);
    const u64 t_start = rtc();
    const u32 callv = ld_agent(ws + OFF_CTR + b * 4);

    {
        u32 d[16];
        const int isw = b >> 7, g = (b & 127) >> 2, q = b & 3;
        if (!isw) {
            const u16* src = p.x + (size_t)(g * 128 + wv * 16 + (lane & 15)) * KS + q * 128 + (lane >> 4) * 8;
#pragma unroll
            for (int f = 0; f < 4; ++f) {
                const u32x4 v = *(const GAS u32x4*)(src + f * 32);
                d[4 * f] = v[0]; d[4 * f + 1] = v[1]; d[4 * f + 2] = v[2]; d[4 * f + 3] = v[3];
            }
        } else {
            const u16* src = p.w + (size_t)(q * 128 + (tid >> 4)) * NN + g * 128 + (tid & 15) * 8;
            u32x4 v[4];
#pragma unroll
            for (int i = 0; i < 4; ++i) v[i] = *(const GAS u32x4*)(src + (size_t)i * 32 * NN);
#pragma unroll
            for (int i = 0; i < 4; ++i) *(LAS u32x4*)(lds + ((tid >> 4) + 32 * i) * WPITCH + (tid & 15) * 16) = v[i];
            __syncthreads();
            const int a0 = (8 * (lane >> 4) + ((lane & 15) >> 2)) * WPITCH + (wv * 16 + 4 * (lane & 3)) * 2;
#pragma unroll
            for (int f = 0; f < 4; ++f)
#pragma unroll
                for (int h = 0; h < 2; ++h) {
                    const s16x4 r = __builtin_amdgcn_ds_read_tr16_b64_v4i16((LAS s16x4*)(lds + a0 + (f * 32 + h * 4) * WPITCH));
                    const u32x2 rr = __builtin_bit_cast(u32x2, r);
                    d[4 * f + 2 * h] = rr[0]; d[4 * f + 2 * h + 1] = rr[1];
                }
        }
        init_tables(lds + SCR_OFF + wv * SCR_WAVE, lane);
        Packed pk; u32 E[8];
        pack_sm(d, pk, E);
        const bool push = !isw && (g & 15) < NPUSH / 32;
        const u32 poff = (u32)(((g & 15) * 4 + q) * 8 + wv) * CH_BYTES;
        const rsrc_t pin = mk_rsrc(ws + OFF_PIN + (size_t)b * UNIT_BYTES, UNIT_BYTES);
        if (push) {
            for (int j = 0; j < 4; ++j) {
                const u32 dest = (u32)((g >> 4) * 4 + j);
                if (dest != (u32)rank && dest < (u32)NR) store_sm(pk, mk_rsrc(p.ws[dest] + OFF_PUSH + (size_t)rank * NPUSH * CH_BYTES, NPUSH * CH_BYTES), poff, lane);
            }
        } else store_sm(pk, pin, wv * CH_BYTES, lane);
        pack_exp(E, pk, mk_rsrc(ws + OFF_FBIN + (size_t)b * FB_UNIT, FB_UNIT), wv * FB_CH, lane, lds + SCR_OFF + wv * SCR_WAVE);
        if (push) {
            for (int j = 0; j < 4; ++j) {
                const u32 dest = (u32)((g >> 4) * 4 + j);
                if (dest != (u32)rank && dest < (u32)NR) store_exp(pk, mk_rsrc(p.ws[dest] + OFF_PUSH + (size_t)rank * NPUSH * CH_BYTES, NPUSH * CH_BYTES), poff, lane);
            }
        } else store_exp(pk, pin, wv * CH_BYTES, lane);
        const bool mine = isw ? ((g >> 3) == rj) : ((g >> 4) == ri);
        const int item = isw ? 16 + (g & 7) : (g & 15);
        if (mine) {
            const rsrc_t dst = mk_rsrc(ws + (isw ? OFF_WF + ((size_t)(g & 7) << 20) : OFF_XF + ((size_t)(g & 15) << 20)), 1u << 20);
#pragma unroll
            for (int f = 0; f < 4; ++f)
                __builtin_amdgcn_raw_buffer_store_b128((u32x4){d[4 * f], d[4 * f + 1], d[4 * f + 2], d[4 * f + 3]}, dst,
                                                       (u32)(((rank * 16 + q * 4 + f) * 8 + wv) * 1024 + lane * 16), 0, AUX_AGENT);
        }
        drain();
        __syncthreads();
        const u32 call0 = rfl(callv) + 1;
        if (tid < (isw ? 2 : 4)) {
            const u32 dest = isw ? (u32)(tid * 4 + (g >> 3)) : (u32)((g >> 4) * 4 + tid);
            if (dest < (u32)NR) {
                if (dest != (u32)rank) st_sys(p.ws[dest] + OFF_FIN + (size_t)(rank * 256 + b) * 128, call0);
                else __hip_atomic_fetch_add((GAS u32*)(ws + OFF_CNT + (item * 8 + rank) * 128), 8u, __ATOMIC_RELAXED, __HIP_MEMORY_SCOPE_AGENT);
            }
        }
        if (tid == 0) st_agent(ws + OFF_CTR + b * 4, call0);
    }
    const u32 call = rfl(callv) + 1;
    STAMP(ws, b, 1);
    const int xcd = b & 7, slot = b >> 3;

    if (slot < GS) {
        const int h0 = xcd * GS + slot;
        LAS u32* okw = (LAS u32*)(lds + LDS_MISC);
        {
            const u32 i = (u32)h0 + (u32)wv * (8 * GS);
            if (i < 7u * NPUSH) {
                const u32 src = (u32)(rank + 1 + i % 7) & 7u, c = i / 7, uid = ri * 64 + (c >> 3), v = c & 7;
                const bool ok = wait_ge<true>(ws + OFF_FIN + (size_t)(src * 256 + uid) * 128, call, tmo, call);
                asm volatile("" ::: "memory");
                const rsrc_t pr = mk_rsrc(ws + OFF_PUSH + (size_t)(src * NPUSH + c) * CH_BYTES, CH_BYTES);
                const u32x4 s0 = __builtin_amdgcn_raw_buffer_load_b128(pr, lane * 16, 0, AUX_SYS);
                const u32x4 s1 = __builtin_amdgcn_raw_buffer_load_b128(pr, lane * 16 + 1024, 0, AUX_SYS);
                const u32x2 cp = __builtin_amdgcn_raw_buffer_load_b64(pr, lane * 8 + 2048, 0, AUX_SYS);
                const u32 lst = __builtin_amdgcn_raw_buffer_load_b32(pr, lane * 4 + 2560, 0, AUX_SYS);
                u32 d[16];
                init_tables(lds + wv * SCR_WAVE, lane);
                unpack_chunk(d, s0, s1, cp, lst, mk_rsrc(p.ws[src] + OFF_FBIN, 256 * FB_UNIT), (uid * 8 + v) * FB_CH, lane, ok, lds + wv * SCR_WAVE);
                const rsrc_t dst = mk_rsrc(ws + OFF_XF + ((size_t)(c >> 5) << 20), 1u << 20);
#pragma unroll
                for (int f = 0; f < 4; ++f)
                    __builtin_amdgcn_raw_buffer_store_b128((u32x4){d[4 * f], d[4 * f + 1], d[4 * f + 2], d[4 * f + 3]}, dst,
                                                           (u32)(((src * 16 + ((c >> 3) & 3) * 4 + f) * 8 + v) * 1024 + lane * 16), 0, AUX_AGENT);
                drain();
                if (lane == 0) __hip_atomic_fetch_add((GAS u32*)(ws + OFF_CNT + ((c >> 5) * 8 + src) * 128), 1u, __ATOMIC_RELAXED, __HIP_MEMORY_SCOPE_AGENT);
            }
        }
#define G_BAR() do { __builtin_amdgcn_s_barrier(); asm volatile("" ::: "memory"); } while (0)
        for (int round = 0; round < (h0 < 256 - 8 * GS ? 2 : 1); ++round) {
            int rg, c64;
            half_tile(h0 + round * 8 * GS, rg, c64);
            const int cg = c64 >> 1, hn = c64 & 1;
            if (wv == 0) {
                const u32 e = max(rg < 4 ? rg : rg + 8, 4 + cg);
                while (rtc() - t_start < (u64)(GATE0 + GATE1 * e)) __builtin_amdgcn_s_sleep(8);
                wait_all16(ws + OFF_CNT + (((lane & 8) ? 16 + cg : rg) * 8 + (lane & 7)) * 128, call * 32, tmo, call, lane);
            }
            asm volatile("" ::: "memory"); __builtin_amdgcn_s_barrier(); asm volatile("" ::: "memory");
            STAMP(ws, b, 2 + 3 * round);
            f32x4 acc[4][2];
#pragma unroll
            for (int m = 0; m < 4; ++m) { acc[m][0] = (f32x4){0.f, 0.f, 0.f, 0.f}; acc[m][1] = (f32x4){0.f, 0.f, 0.f, 0.f}; }
            const int wr = (wv >> 1) & 1, wc = wv & 1;
            if (wv >= 4) {
                const int lw = wv - 4;
                const char* xg = ws + OFF_XF + ((size_t)rg << 20) + lw * 4096 + lane * 16;
                const char* wg = ws + OFF_WF + ((size_t)cg << 20) + (lw >> 1) * 8192 + hn * 4096 + (lw & 1) * 2048 + lane * 16;
#define L_ISSUE(s, SL) do { const char* _x = xg + (size_t)(s) * 16384; const char* _w = wg + (size_t)(s) * 16384; \
                _Pragma("unroll") for (int i = 0; i < 4; ++i) __builtin_amdgcn_global_load_lds((const GAS u32*)(_x + i * 1024), (LAS u32*)(lds + (SL) * GSLOT + lw * 4096 + i * 1024), 16, 0, GL_AUX); \
                _Pragma("unroll") for (int i = 0; i < 2; ++i) __builtin_amdgcn_global_load_lds((const GAS u32*)(_w + i * 1024), (LAS u32*)(lds + (SL) * GSLOT + 16384 + lw * 2048 + i * 1024), 16, 0, GL_AUX); } while (0)
#define L_WAIT(n) asm volatile("s_waitcnt vmcnt(" #n ")" ::: "memory")
                L_ISSUE(0, 0); L_ISSUE(1, 1); L_ISSUE(2, 2); L_ISSUE(3, 3);
                L_WAIT(18); G_BAR();
                for (int s = 0; s < 60; s += 4) {
                    L_WAIT(12); G_BAR(); L_ISSUE(s + 4, 0);
                    L_WAIT(12); G_BAR(); L_ISSUE(s + 5, 1);
                    L_WAIT(12); G_BAR(); L_ISSUE(s + 6, 2);
                    L_WAIT(12); G_BAR(); L_ISSUE(s + 7, 3);
                }
                L_WAIT(12); G_BAR(); L_WAIT(6); G_BAR(); L_WAIT(0); G_BAR(); G_BAR();
            } else {
                const int xro = wr * 4096 + lane * 16, wro = 16384 + wc * 2048 + lane * 16;
                bf16x8 xa[2][2][4], wb[2][2][2];
#define C_READ(P, SL) do { _Pragma("unroll") for (int k = 0; k < 2; ++k) { \
                _Pragma("unroll") for (int n = 0; n < 2; ++n) wb[P][k][n] = *(const LAS bf16x8*)(lds + (SL) * GSLOT + wro + k * 4096 + n * 1024); \
                _Pragma("unroll") for (int m = 0; m < 4; ++m) xa[P][k][m] = *(const LAS bf16x8*)(lds + (SL) * GSLOT + xro + k * 8192 + m * 1024); } } while (0)
#define C_MMA(P) do { _Pragma("unroll") for (int k = 0; k < 2; ++k) _Pragma("unroll") for (int m = 0; m < 4; ++m) \
                _Pragma("unroll") for (int n = 0; n < 2; ++n) acc[m][n] = __builtin_amdgcn_mfma_f32_16x16x32_bf16(wb[P][k][n], xa[P][k][m], acc[m][n], 0, 0, 0); } while (0)
#define C_SCHED() do { _Pragma("unroll") for (int i = 0; i < 12; ++i) { __builtin_amdgcn_sched_group_barrier(0x008, 1, 0); __builtin_amdgcn_sched_group_barrier(0x100, 1, 0); } \
                __builtin_amdgcn_sched_group_barrier(0x008, 4, 0); } while (0)
#define C_STEP(SL, P, RD) do { asm volatile("s_waitcnt lgkmcnt(0)" ::: "memory"); G_BAR(); if (RD) C_READ((P) ^ 1, ((SL) + 1) & 3); C_MMA(P); if (RD) C_SCHED(); __builtin_amdgcn_sched_barrier(0); } while (0)
                G_BAR();
                C_READ(0, 0);
                for (int s = 0; s < 60; s += 4) { C_STEP(0, 0, true); C_STEP(1, 1, true); C_STEP(2, 0, true); C_STEP(3, 1, true); }
                C_STEP(0, 0, true); C_STEP(1, 1, true); C_STEP(2, 0, true); C_STEP(3, 1, false);
            }
            G_BAR();
            STAMP(ws, b, 3 + 3 * round);
            if (wv < 4) {
#pragma unroll
                for (int m = 0; m < 4; ++m)
#pragma unroll
                    for (int n = 0; n < 2; ++n) {
                        u32x2 o;
                        asm("v_cvt_pk_bf16_f32 %0, %1, %2" : "=v"(o.x) : "v"(acc[m][n][0]), "v"(acc[m][n][1]));
                        asm("v_cvt_pk_bf16_f32 %0, %1, %2" : "=v"(o.y) : "v"(acc[m][n][2]), "v"(acc[m][n][3]));
                        *(LAS u32x2*)(lds + (wr * 64 + m * 16 + (lane & 15)) * OPITCH + (wc * 32 + n * 16 + (lane >> 4) * 4) * 2) = o;
                    }
            }
            if (tid == 0) okw[0] = (ld_agent(tmo) == call) ? 1u : 0u;
            __syncthreads();
            u32 d[16];
            const int j = wv & 3;
#pragma unroll
            for (int f = 0; f < 4; ++f) {
                const u32x4 v = *(const LAS u32x4*)(lds + (j * 32 + f * 8 + (lane >> 3)) * OPITCH + (lane & 7) * 16);
                d[4 * f] = v[0]; d[4 * f + 1] = v[1]; d[4 * f + 2] = v[2]; d[4 * f + 3] = v[3];
            }
            if (okw[0]) {
#pragma unroll
                for (int i = 0; i < 16; ++i) d[i] = 0x7FC07FC0u;
            }
            const int t = rg < 4 ? cg * 4 + rg : rg * 8 + cg;
            if (wv < 4) {
                init_tables(lds + SCR_OFF + wv * SCR_WAVE, lane);
                Packed pk; u32 E[8];
                const rsrc_t pout = mk_rsrc(ws + OFF_POUT + (size_t)t * UNIT_BYTES, UNIT_BYTES);
                pack_sm(d, pk, E);
                store_sm(pk, pout, (hn * 4 + j) * CH_BYTES, lane);
                pack_exp(E, pk, mk_rsrc(ws + OFF_FBOUT + (size_t)t * FB_UNIT, FB_UNIT), (hn * 4 + j) * FB_CH, lane, lds + SCR_OFF + wv * SCR_WAVE);
                store_exp(pk, pout, (hn * 4 + j) * CH_BYTES, lane);
                drain();
            } else {
                const rsrc_t outr = mk_rsrc(p.out, 4096u * NN * 2);
                const u32 oo = (u32)((ri * 2048 + rg * 128 + j * 32 + (lane >> 3)) * NN + rj * 1024 + c64 * 64 + (lane & 7) * 8) * 2;
#pragma unroll
                for (int f = 0; f < 4; ++f)
                    __builtin_amdgcn_raw_buffer_store_b128((u32x4){d[4 * f], d[4 * f + 1], d[4 * f + 2], d[4 * f + 3]}, outr, oo + f * 8 * NN * 2, 0, OUT_AUX);
            }
            __syncthreads();
            if (tid < NR && tid != rank) st_sys(p.ws[tid] + OFF_FOUT + (size_t)(rank * 256 + t * 2 + hn) * 128, call);
            STAMP(ws, b, 4 + 3 * round);
        }
        STAMPC(ws, b, call, 1);
        return;
    }

    const int pi = (slot - GS) * 8 + xcd;
    {
        const int gw = pi * PWPW + wv;
        if (wv >= PWPW || gw >= 7 * NWL) return;
        const int link = gw % 7, wl = gw / 7;
        const u32 src = rfl((u32)(rank + 1 + link) & 7u);
        if (src >= (u32)NR) return;
        const u64 pws = uni64((u64)p.ws[src]), wsu = uni64((u64)ws);
        LAS unsigned char* scr = lds + wv * SCR_WAVE;
        init_tables(scr, lane);
        GAS u32* tick = (GAS u32*)(ws + OFF_TICK + link * 128);
        const u32 tbase = NPUSH + 3 * NWL - (call - 1) * (u32)(NCH - NPUSH - 2 * NWL);
        struct Desc { u32 rem, fb, flg, dst, cnt; };
        const u32 xbase = ri * 64, wbase = 128 + rj * 32;
        const u32 obase = ((src >> 2) * 2048 * NN + (src & 3) * 1024) * 2;
        const u32 fin0 = (u32)OFF_FIN + src * 256 * 128, fout0 = (u32)OFF_FOUT + src * 256 * 128 - IN_UNITS * 256;
        const u32 cnt0 = (u32)OFF_CNT + src * 128, din0 = src * 128 * 1024;
        auto mk = [&](u32 c) -> Desc {
            c = min(c, (u32)NCH - 1);
            const u32 u = c >> 3, v = c & 7, e = u >> 2, m = u - IN_UNITS;
            u32 s4 = e >= 4 ? 4u : 0u, s12 = e >= 12 ? 4u : 0u, dx = e < 4 ? xbase : xbase - 32, dw = wbase - 16;
            u32 rgo = m < 32 ? (m & 3) : (m >> 3), cgo = m < 32 ? (m >> 2) : (m & 7);
            asm("" : "+s"(s4), "+s"(s12), "+s"(dx), "+s"(rgo), "+s"(cgo));
            const bool in = u < IN_UNITS, w = s4 != s12;
            const u32 pan = e - s4 - s12;
            u32 uidi = u + (w ? dw : dx), pbase = w ? (u32)OFF_WF : (u32)OFF_XF, cw = w ? 16u : 0u;
            asm("" : "+s"(uidi), "+s"(pbase), "+s"(cw));
            u32 remi = (u32)OFF_PIN + uidi * UNIT_BYTES, remo = (u32)OFF_POUT + m * UNIT_BYTES;
            u32 fbi = (u32)OFF_FBIN + uidi * FB_UNIT, fbo = (u32)OFF_FBOUT + m * FB_UNIT;
            u32 flgi = fin0 + uidi * 128, flgo = fout0 + u * 256 + (v >> 2) * 128;
            u32 dsti = pbase + (pan << 20) + din0 + ((u & 3) * 32 + v) * 1024;
            u32 dsto = obase + ((rgo * 128 + (v & 3) * 32) * NN + cgo * 128 + (v >> 2) * 64) * 2;
            u32 cnti = cnt0 + (cw + pan) * 1024;
            asm("" : "+s"(remi), "+s"(remo), "+s"(fbi), "+s"(fbo), "+s"(flgi), "+s"(flgo), "+s"(dsti), "+s"(dsto), "+s"(cnti));
            Desc d;
            d.rem = (in ? remi : remo) + v * CH_BYTES;
            d.fb = (in ? fbi : fbo) + v * FB_CH;
            d.flg = in ? flgi : flgo;
            d.dst = in ? dsti : dsto;
            d.cnt = in ? cnti : 0u;
            return d;
        };
        const rsrc_t wsr = mk_rsrc(ws, (u32)WS_NEED), outr = mk_rsrc(p.out, 4096u * NN * 2), pwr = mk_rsrc((const void*)pws, (u32)WS_NEED);
        const u32 vin = lane * 16, vout = (lane >> 3) * (NN * 2) + (lane & 7) * 16, vl4 = 2560 + lane * 4;
        u32 n1 = NPUSH + NWL + wl, n2 = NPUSH + 2 * NWL + wl;
        Desc D0 = mk(NPUSH + wl), D1 = mk(n1), D2 = mk(n2);
#define P_CLOB "a0", "a1", "a2", "a3", "a4", "a5", "a6", "a7", "a8", "a9", "a10", "a11"
#define P_ISSUE(DA) do { \
            const u64 un_ = pws + (DA).rem; \
            asm volatile("s_nop 4\n\tglobal_load_dwordx4 a[0:3], %0, %2 sc0 sc1\n\tglobal_load_dwordx4 a[4:7], %0, %2 offset:1024 sc0 sc1\n\t" \
                         "global_load_dwordx2 a[8:9], %1, %2 offset:2048 sc0 sc1\n\tglobal_load_dword a11, %3, %2 sc0 sc1" \
                         :: "v"(vin), "v"(lane * 8), "s"(un_), "v"(vl4) : P_CLOB, "memory"); } while (0)
#define P_FLAG(DB) do { const u64 fa_ = wsu + (DB).flg; \
            asm volatile("s_nop 4\n\tglobal_load_dword a12, %0, %1 sc0 sc1" :: "v"(0), "s"(fa_) : "a12", "memory"); } while (0)
#define P_TICKET(want_) do { u64 sv_; const u32 em_ = rfl((want_) ? 1u : 0u); \
            asm volatile("v_accvgpr_write_b32 a14, 1\n\ts_mov_b64 %0, exec\n\ts_mov_b32 exec_lo, %2\n\ts_mov_b32 exec_hi, 0\n\ts_nop 1\n\t" \
                         "global_atomic_add a13, %1, a14, off sc0\n\ts_mov_b64 exec, %0" \
                         : "=&s"(sv_) : "v"(tick), "s"(em_) : "a13", "a14", "memory"); } while (0)
#define P_FETCH() asm volatile("v_accvgpr_read_b32 %0, a0\n\tv_accvgpr_read_b32 %1, a1\n\tv_accvgpr_read_b32 %2, a2\n\tv_accvgpr_read_b32 %3, a3\n\t" \
                         "v_accvgpr_read_b32 %4, a4\n\tv_accvgpr_read_b32 %5, a5\n\tv_accvgpr_read_b32 %6, a6\n\tv_accvgpr_read_b32 %7, a7\n\t" \
                         "v_accvgpr_read_b32 %8, a8\n\tv_accvgpr_read_b32 %9, a9\n\tv_accvgpr_read_b32 %10, a11\n\t" \
                         "v_accvgpr_read_b32 %11, a12\n\tv_accvgpr_read_b32 %12, a13" \
                         : "=v"(c0.x), "=v"(c0.y), "=v"(c0.z), "=v"(c0.w), "=v"(c1.x), "=v"(c1.y), "=v"(c1.z), "=v"(c1.w), \
                           "=v"(cc.x), "=v"(cc.y), "=v"(cl), "=v"(fl), "=v"(tk) :: "memory")
#define P_SIGNAL(c_) do { u64 sv_; const u64 ca_ = wsu + (c_); const u32 em_ = rfl((c_) ? 1u : 0u); \
            asm volatile("s_nop 4\n\ts_mov_b64 %0, exec\n\ts_mov_b32 exec_lo, %4\n\ts_mov_b32 exec_hi, 0\n\tglobal_atomic_add %1, %2, %3\n\ts_mov_b64 exec, %0" \
                         : "=&s"(sv_) : "v"(0), "v"(1u), "s"(ca_), "s"(em_) : "memory"); } while (0)
#define TS() do {} while (0)
        bool ok = wait_ge<true>(ws + D0.flg, call, tmo, call);
        asm volatile("" ::: "memory");
        TS();
        P_ISSUE(D0); P_FLAG(D1);
        bool tkv = false;
        drain();
        u32 pendA = 0, pendB = 0;
        for (;;) {
            asm volatile("s_waitcnt vmcnt(4)" ::: "memory");
            TS();
            u32x4 c0, c1; u32x2 cc; u32 cl, fl, tk;
            P_FETCH();
            const bool cok = ok;
            const bool more = n1 < (u32)NCH;
            const bool early = more && rfl(fl) >= call;
            if (__builtin_expect(early, 1)) { ok = true; asm volatile("" ::: "memory"); P_ISSUE(D1); }
            n2 = tkv ? rfl(tk) + tbase : n2;
            D2 = mk(n2);
            if (__builtin_expect(early, 1)) { P_FLAG(D2); tkv = n2 < (u32)NCH; P_TICKET(tkv); }
            P_SIGNAL(pendA);
            pendA = pendB;
            u32 d[16];
            unpack_chunk(d, c0, c1, cc, cl, pwr, D0.fb, lane, cok, scr);

            {
                const bool isin = D0.cnt != 0;
                const rsrc_t dr = isin ? wsr : outr;
                const u32 vo = isin ? vin : vout, step = isin ? 8192u : 8u * NN * 2;
#pragma unroll
                for (int f = 0; f < 4; ++f)
                    __builtin_amdgcn_raw_buffer_store_b128((u32x4){d[4 * f], d[4 * f + 1], d[4 * f + 2], d[4 * f + 3]}, dr, vo, D0.dst + f * step, AUX_AGENT);
            }
            u32 mine = D0.cnt;
            if (__builtin_expect(!early, 0)) {
                drain(); P_SIGNAL(pendA); P_SIGNAL(mine);
                pendA = 0; mine = 0;
                if (!more) break;
                ok = wait_ge<true>(ws + D1.flg, call, tmo, call);
                asm volatile("" ::: "memory");
                P_ISSUE(D1); P_FLAG(D2); tkv = n2 < (u32)NCH; P_TICKET(tkv);
                drain();
            }
            pendB = mine;
            n1 = n2; if (!tkv) n2 = NCH;
            D0 = D1; D1 = D2;
        }
        STAMPW(ws, b, 4);
        drain();
        STAMPC(ws, b, call, 1);
    }
}
}

#define HIP_CHECK(expr) do { hipError_t _e = (expr); if (_e != hipSuccess) { \
    fprintf(stderr, "HIP error %s at %s:%d\n", hipGetErrorString(_e), __FILE__, __LINE__); \
    return; } } while (0)

extern "C" void kernel_launch(int rank, int nranks, ncclComm_t comm,
                              hipStream_t stream,
                              void* const* d_in, const int* in_sizes, int n_in,
                              void* d_out, int out_size,
                              const int* in_dtypes, int out_dtype,
                              void* const* d_out_all,
                              void* const* d_ws_all, size_t ws_bytes) {
    (void)comm; (void)in_sizes; (void)n_in; (void)out_size; (void)in_dtypes; (void)out_dtype; (void)d_out_all;
    if (nranks != NR || ws_bytes < WS_NEED || rank < 0 || rank >= NR) { fprintf(stderr, "unexpected configuration\n"); return; }
    static bool attr_done[64] = {};
    int dev = 0;
    HIP_CHECK(hipGetDevice(&dev));
    if (dev >= 0 && dev < 64 && !attr_done[dev]) {
        HIP_CHECK(hipFuncSetAttribute((const void*)fused_gemm_ar, hipFuncAttributeMaxDynamicSharedMemorySize, LDS_BYTES));
        attr_done[dev] = true;
    }
    Params p;
    p.x = (const u16*)d_in[0];
    p.w = (const u16*)d_in[1];
    p.out = (u16*)d_out;
    for (int i = 0; i < NR; ++i) p.ws[i] = (char*)d_ws_all[i];
    p.rank = rank; p.pad = 0;
    hipLaunchKernelGGL(fused_gemm_ar, dim3(GRID), dim3(NT), LDS_BYTES, stream, p);
    HIP_CHECK(hipGetLastError());
}
```

```cpp
#include <hip/hip_runtime.h>
#include <rccl/rccl.h>
#include <cstdio>
#include <cstdint>

namespace {
typedef unsigned u32;
typedef unsigned long long u64;
typedef unsigned short u16;
typedef u32 u32x4 __attribute__((ext_vector_type(4)));
typedef u32 u32x2 __attribute__((ext_vector_type(2)));
typedef short bf16x8 __attribute__((ext_vector_type(8)));
typedef short s16x4 __attribute__((ext_vector_type(4)));
typedef u16 u16x2 __attribute__((ext_vector_type(2)));
typedef float f32x4 __attribute__((ext_vector_type(4)));
#define LAS __attribute__((address_space(3)))
#define GAS __attribute__((address_space(1)))

constexpr int NR = 8;
constexpr int NN = 4096, KS = 512;
constexpr int NT = 512, GRID = 256;
constexpr int NWL = 27;
constexpr int GS = 18;
constexpr int NPULL = 8 * (32 - GS);
constexpr int PWPW = 2;
static_assert(GS % 2 == 0 && GS >= 16 && 7 * NWL <= NPULL * PWPW, "roles");
constexpr u64 DEADLINE = 40000000ull;

constexpr u32 CH_BYTES = 2816, UNIT_BYTES = 8 * CH_BYTES;
constexpr u32 FB_CH = 2048, FB_UNIT = 8 * FB_CH;
constexpr int IN_UNITS = 96, OUT_UNITS = 128, ALL_UNITS = IN_UNITS + OUT_UNITS;
constexpr int NCH = ALL_UNITS * 8;
constexpr int NPUSH = 64;

constexpr size_t OFF_CTR = 0;
constexpr size_t OFF_TMO = 4096;
constexpr size_t OFF_TICK = 5120;
constexpr size_t OFF_CNT = 8192;
constexpr size_t OFF_FIN = OFF_CNT + 24 * 8 * 128;
constexpr size_t OFF_FOUT = OFF_FIN + (size_t)8 * 256 * 128;
constexpr size_t OFF_PIN = OFF_FOUT + (size_t)8 * 256 * 128;
constexpr size_t OFF_PUSH = OFF_PIN + (size_t)256 * UNIT_BYTES;
constexpr size_t OFF_POUT = OFF_PUSH + (size_t)8 * NPUSH * CH_BYTES;
constexpr size_t OFF_FBIN = OFF_POUT + (size_t)128 * UNIT_BYTES;
constexpr size_t OFF_FBOUT = OFF_FBIN + (size_t)256 * FB_UNIT;
constexpr size_t OFF_XF = OFF_FBOUT + (size_t)128 * FB_UNIT;
constexpr size_t OFF_WF = OFF_XF + ((size_t)16 << 20);
constexpr size_t WS_NEED = OFF_WF + ((size_t)8 << 20);
static_assert(OFF_PIN % 128 == 0 && OFF_XF % 1024 == 0, "alignment");

constexpr int LDS_BYTES = 131072 + 1024;
constexpr int LDS_MISC = 131072;
constexpr int GSLOT = 24576;
constexpr int WPITCH = 288;
constexpr int OPITCH = 144;
constexpr int SCR_OFF = 65536, SCR_R = 2048, SCR_POS = 2304, SCR_SEL = 2432, SCR_CMP = 2496, SCR_WAVE = 2560;

struct Params {
    const u16* x; const u16* w; u16* out;
    char* ws[NR];
    int rank; int pad;
};

typedef __amdgpu_buffer_rsrc_t rsrc_t;
__device__ __forceinline__ rsrc_t mk_rsrc(const void* p, u32 bytes) {
    return __builtin_amdgcn_make_buffer_rsrc((void*)p, 0, bytes, 0x00020000);
}
constexpr int AUX_SYS = 17, AUX_AGENT = 16;
constexpr u64 GATE0 = 1000, GATE1 = 150;
__device__ __forceinline__ u64 rtc() { return __builtin_amdgcn_s_memrealtime(); }
__device__ __forceinline__ u32 rfl(u32 v) { return __builtin_amdgcn_readfirstlane(v); }
__device__ __forceinline__ u64 uni64(u64 v) { return ((u64)rfl((u32)(v >> 32)) << 32) | rfl((u32)v); }
__device__ __forceinline__ u32 bfi(u32 m, u32 a, u32 b) { return (m & a) | (~m & b); }
__device__ __forceinline__ u32 perm(u32 a, u32 b, u32 s) { return __builtin_amdgcn_perm(a, b, s); }
__device__ __forceinline__ void drain() { asm volatile("s_waitcnt vmcnt(0)" ::: "memory"); }
__device__ __forceinline__ u32 ld_sys(const char* p) { return __hip_atomic_load((const GAS u32*)p, __ATOMIC_RELAXED, __HIP_MEMORY_SCOPE_SYSTEM); }
__device__ __forceinline__ u32 ld_agent(const char* p) { return __hip_atomic_load((const GAS u32*)p, __ATOMIC_RELAXED, __HIP_MEMORY_SCOPE_AGENT); }
__device__ __forceinline__ void st_sys(char* p, u32 v) { __hip_atomic_store((GAS u32*)p, v, __ATOMIC_RELAXED, __HIP_MEMORY_SCOPE_SYSTEM); }
__device__ __forceinline__ void st_agent(char* p, u32 v) { __hip_atomic_store((GAS u32*)p, v, __ATOMIC_RELAXED, __HIP_MEMORY_SCOPE_AGENT); }

template <bool SYS>
__device__ __forceinline__ bool wait_ge(const char* flag, u32 want, char* tmo, u32 call) {
    const u64 t0 = rtc();
    for (u32 n = 1;; ++n) {
        const u32 v = rfl(SYS ? ld_sys(flag) : ld_agent(flag));
        if (v >= want) return true;
        if ((n & 7) == 0 && (rfl(ld_agent(tmo)) == call || rtc() - t0 > DEADLINE)) { st_agent(tmo, call); return false; }
        __builtin_amdgcn_s_sleep(1);
    }
}
__device__ __forceinline__ bool wait_all16(const char* cnt_lane, u32 want, char* tmo, u32 call, int lane) {
    const u64 t0 = rtc();
    for (u32 n = 1;; ++n) {
        const u32 v = lane < 16 ? ld_agent(cnt_lane) : want;
        if (__all(v >= want)) return true;
        if ((n & 7) == 0 && (rfl(ld_agent(tmo)) == call || rtc() - t0 > DEADLINE)) { st_agent(tmo, call); return false; }
        __builtin_amdgcn_s_sleep(8);
    }
}

#define DPP_ADD(v, ctrl, rm) v += (u32)__builtin_amdgcn_update_dpp(0, (int)(v), ctrl, rm, 0xF, true)
__device__ __forceinline__ u32 wave_sum(u32 v) {
    DPP_ADD(v, 0xB1, 0xF); DPP_ADD(v, 0x4E, 0xF); DPP_ADD(v, 0x141, 0xF); DPP_ADD(v, 0x140, 0xF);
    return (__builtin_amdgcn_readlane(v, 0) + __builtin_amdgcn_readlane(v, 16)) + (__builtin_amdgcn_readlane(v, 32) + __builtin_amdgcn_readlane(v, 48));
}
__device__ __forceinline__ u32 wave_scan(u32 v) {
    DPP_ADD(v, 0x111, 0xF); DPP_ADD(v, 0x112, 0xF); DPP_ADD(v, 0x114, 0xF); DPP_ADD(v, 0x118, 0xF);
    DPP_ADD(v, 0x142, 0xA); DPP_ADD(v, 0x143, 0xC);
    return v;
}
__device__ __forceinline__ void init_tables(LAS unsigned char* scr, int lane) {
    if (lane < 16) {
        const u32 m = ((u32)lane * 0x00204081u) & 0x01010101u;
        ((LAS u32*)(scr + SCR_SEL))[lane] = bfi((m << 8) - m, (m << 8) + (m << 16) + (m << 24) + 0x04040404u, 0x03020100u);
        u32 c = 0x0C0C0C0Cu; int q = 0;
#pragma unroll
        for (int b = 0; b < 4; ++b) if ((lane >> b) & 1) { c = bfi(0xFFu << (8 * q), (u32)b << (8 * q), c); ++q; }
        ((LAS u32*)(scr + SCR_CMP))[lane] = c;
    }
    asm volatile("" ::: "memory");
}
struct Packed { u32x4 s0, s1; u32x2 ab; u32 r; };
__device__ __forceinline__ void store_sm(const Packed& k, rsrc_t dst, u32 off, int lane) {
    __builtin_amdgcn_raw_buffer_store_b128(k.s0, dst, lane * 16, off, AUX_SYS);
    __builtin_amdgcn_raw_buffer_store_b128(k.s1, dst, lane * 16 + 1024, off, AUX_SYS);
}
__device__ __forceinline__ void store_exp(const Packed& k, rsrc_t dst, u32 off, int lane) {
    __builtin_amdgcn_raw_buffer_store_b64(k.ab, dst, lane * 8 + 2048, off, AUX_SYS);
    __builtin_amdgcn_raw_buffer_store_b32(k.r, dst, lane * 4 + 2560, off, AUX_SYS);
}
__device__ __forceinline__ void pack_sm(const u32 (&d)[16], Packed& k, u32 (&E)[8]) {
    u32 SM[8];
#pragma unroll
    for (int t = 0; t < 8; ++t) {
        const u32 lo = perm(d[2 * t + 1], d[2 * t], 0x06040200u), hi = perm(d[2 * t + 1], d[2 * t], 0x07050301u);
        SM[t] = bfi(0x7F7F7F7Fu, lo, hi);
        E[t] = bfi(0xFEFEFEFEu, hi << 1, lo >> 7);
    }
    k.s0 = (u32x4){SM[0], SM[1], SM[2], SM[3]}; k.s1 = (u32x4){SM[4], SM[5], SM[6], SM[7]};
}
__device__ __forceinline__ void pack_exp(const u32 (&E)[8], Packed& pk, rsrc_t fb, u32 fbo, int lane, LAS unsigned char* scr) {
    u32 sum = 0;
#pragma unroll
    for (int t = 0; t < 8; ++t) sum = __builtin_amdgcn_udot4(E[t], 0x01010101u, sum, false);
    const u32 base = min((wave_sum(sum) + 4035u) >> 11, 255u);
    const u16x2 b2 = {(u16)base, (u16)base}, c1 = {1, 1}, c3 = {3, 3}, c11 = {11, 11};
    u32 A = 0, B = 0, T = 0, k3 = 0;
    u64 acc = 0;
#pragma unroll
    for (int t = 0; t < 8; ++t) {
        const u16x2 da = b2 - __builtin_bit_cast(u16x2, E[t] & 0x00FF00FFu), db = b2 - __builtin_bit_cast(u16x2, perm(E[t], E[t], 0x0C030C01u));
        const u32 c2 = __builtin_bit_cast(u32, __builtin_elementwise_min(da, c3)) | (__builtin_bit_cast(u32, __builtin_elementwise_min(db, c3)) << 8);
        const u16x2 sa = __builtin_elementwise_max(__builtin_elementwise_min((u16x2)(da + c1), c11), c3) - c3;
        const u16x2 sb = __builtin_elementwise_max(__builtin_elementwise_min((u16x2)(db + c1), c11), c3) - c3;
        const u32 s8 = __builtin_bit_cast(u32, sa) | (__builtin_bit_cast(u32, sb) << 8);
        const u32 m = c2 & (c2 >> 1) & 0x01010101u;
        T |= ((s8 >> 3) & 0x01010101u) << t;
        const u32 cs = *(const LAS u32*)(scr + SCR_CMP + __builtin_amdgcn_udot4(m, 0x20100804u, 0u, false));
        const u32 x = perm(0u, s8 & 0x07070707u, cs);
        u32 v = (x | (x >> 5)) & 0x003F003Fu;
        v = (v | (v >> 10)) & 0xFFFu;
        acc |= (u64)v << min(k3, 63u);
        k3 = __builtin_amdgcn_udot4(m, 0x03030303u, k3, false);
        if (t < 4) A |= c2 << (2 * t); else B |= c2 << (2 * (t - 4));
    }
    const u32 nt = __builtin_popcount(T);
    const u32 incl = wave_scan(k3 | (nt << 16));
    const u32 tot = __builtin_amdgcn_readlane(incl, 63), K3 = tot & 0xFFFFu;
    u32 cnt = tot >> 16, R = 0;
    bool raw = __ballot(k3 > 63) != 0 || cnt > 63 || 32 + K3 > 16 * (128 - cnt);
    if (!raw) {
        LAS u32* rl = (LAS u32*)(scr + SCR_R);
        rl[lane] = lane == 0 ? (base | (cnt << 8)) : 0u;
        asm volatile("" ::: "memory");
        const u32 off = 32 + (incl & 0xFFFFu) - k3, i = off >> 5, sh = off & 31;
        const u32 lo = (u32)acc, hi = (u32)(acc >> 32);
        __hip_atomic_fetch_or(rl + i, lo << sh, __ATOMIC_RELAXED, __HIP_MEMORY_SCOPE_WORKGROUP);
        __hip_atomic_fetch_or(rl + ((i + 1) & 63), (u32)((acc << sh) >> 32), __ATOMIC_RELAXED, __HIP_MEMORY_SCOPE_WORKGROUP);
        __hip_atomic_fetch_or(rl + ((i + 2) & 63), (hi >> 1) >> (31 - sh), __ATOMIC_RELAXED, __HIP_MEMORY_SCOPE_WORKGROUP);
        bool bad = false;
        if (cnt) {
            *(LAS u32x4*)(scr + lane * 32) = (u32x4){E[0], E[1], E[2], E[3]};
            *(LAS u32x4*)(scr + lane * 32 + 16) = (u32x4){E[4], E[5], E[6], E[7]};
            u32 idx = (incl >> 16) - nt, M = T;
            while (M) {
                const u32 bit = __builtin_ctz(M); M &= M - 1;
                ((LAS u16*)(scr + SCR_POS))[idx & 63] = (u16)(lane * 32 + (bit & 7) * 4 + (bit >> 3));
                ++idx;
            }
            asm volatile("" ::: "memory");
            if ((u32)lane < cnt) {
                const u32 pos = ((const LAS u16*)(scr + SCR_POS))[lane] & 2047u, e = scr[pos];
                const int dl = (int)base - (int)e;
                bad = e != 0 && (dl > 24 || dl < -17);
                const u32 c5 = e == 0 ? 15u : (dl > 0 ? (u32)(dl - 10) : (u32)(14 - dl));
                ((LAS u16*)(scr + SCR_R))[127 - lane] = (u16)((pos << 5) | (c5 & 31u));
            }
        }
        asm volatile("" ::: "memory");
        R = rl[lane];
        raw = __ballot(bad) != 0;
    }
    if (raw) {
        R = lane == 0 ? (base | (255u << 8)) : 0u;
        __builtin_amdgcn_raw_buffer_store_b128((u32x4){E[0], E[1], E[2], E[3]}, fb, lane * 16, fbo, AUX_SYS);
        __builtin_amdgcn_raw_buffer_store_b128((u32x4){E[4], E[5], E[6], E[7]}, fb, lane * 16 + 1024, fbo, AUX_SYS);
    }
    pk.ab = (u32x2){A, B}; pk.r = R;
}

__device__ __forceinline__ void unpack_chunk(u32 (&d)[16], u32x4 s0, u32x4 s1, u32x2 ab, u32 R, rsrc_t fb, u32 fbo, int lane, bool ok,
                                             LAS unsigned char* scr) {
    const u32 h0 = __builtin_amdgcn_readlane(R, 0);
    const u32 cnt = ok ? (h0 >> 8) & 0xFFu : 0u, base = h0 & 0xFFu;
    u32 E[8];
    if (__builtin_expect(cnt == 255, 0)) {
        u32x4 a = __builtin_amdgcn_raw_buffer_load_b128(fb, lane * 16, fbo, AUX_SYS);
        u32x4 b = __builtin_amdgcn_raw_buffer_load_b128(fb, lane * 16 + 1024, fbo, AUX_SYS);
        asm volatile("" : "+v"(a), "+v"(b));
#pragma unroll
        for (int t = 0; t < 4; ++t) { E[t] = a[t]; E[4 + t] = b[t]; }
    } else {
        const u32 B4 = base * 0x01010101u;
        const u32 lutp = B4 - 0x03020100u, luts_lo = bfi(0xFFu, base + 1, B4 - 0x05040300u), luts_hi = B4 - 0x09080706u;
        const u32 k = __builtin_popcount(ab[0] & (ab[0] >> 1) & 0x55555555u) + __builtin_popcount(ab[1] & (ab[1] >> 1) & 0x55555555u);
        LAS u32* rl = (LAS u32*)(scr + SCR_R);
        rl[lane] = R;
        asm volatile("" ::: "memory");
        const u32 off = 32 + 3 * (wave_scan(k) - k), i = min(off >> 5, 61u);
        const u32 w0 = rl[i], w1 = rl[i + 1], w2 = rl[i + 2];
        const u64 win = ((u64)__builtin_amdgcn_alignbit(w2, w1, off) << 32) | __builtin_amdgcn_alignbit(w1, w0, off);
        u32 r3 = 0, m3 = 0x03030303u, m1 = 0x01010101u;
        asm("" : "+s"(m3), "+s"(m1));
#pragma unroll
        for (int t = 0; t < 8; ++t) {
            const u32 c2 = (ab[t >> 2] >> (2 * (t & 3))) & m3;
            const u32 m = c2 & (c2 >> 1) & m1;
            const u32 sel = *(const LAS u32*)(scr + SCR_SEL + __builtin_amdgcn_udot4(m, 0x20100804u, 0u, false));
            const u32 F = (u32)(win >> (r3 & 63));
            const u32 x = bfi(0x3Fu, F, F << 10);
            const u32 u = bfi(0x00070007u, x, x << 5) & 0x07070707u;
            E[t] = perm(perm(luts_hi, luts_lo, u), perm(lutp, lutp, c2), sel);
            r3 = __builtin_amdgcn_udot4(m, 0x03030303u, r3, false);
        }
        if (__builtin_expect(cnt != 0, 1)) {
            *(LAS u32x4*)(scr + lane * 32) = (u32x4){E[0], E[1], E[2], E[3]};
            *(LAS u32x4*)(scr + lane * 32 + 16) = (u32x4){E[4], E[5], E[6], E[7]};
            asm volatile("" ::: "memory");
#pragma unroll
            for (int h = 0; h < 2; ++h) {
                const u32 ent = (R >> (16 * h)) & 0xFFFFu, c5 = ent & 31u;
                const u32 e = c5 < 16 ? (c5 == 15 ? 0u : base - 10 - c5) : base + c5 - 14;
                if ((u32)(127 - 2 * lane - h) < cnt) scr[ent >> 5] = (unsigned char)e;
            }
            asm volatile("" ::: "memory");
            const u32x4 a = *(const LAS u32x4*)(scr + lane * 32), b = *(const LAS u32x4*)(scr + lane * 32 + 16);
#pragma unroll
            for (int t = 0; t < 4; ++t) { E[t] = a[t]; E[4 + t] = b[t]; }
        }
    }
#pragma unroll
    for (int t = 0; t < 8; ++t) {
        const u32 sm = t < 4 ? s0[t & 3] : s1[t & 3];
        const u32 lo = bfi(0x7F7F7F7Fu, sm, E[t] << 7), hi = bfi(0x80808080u, sm, E[t] >> 1);
        d[2 * t] = perm(hi, lo, 0x05010400u); d[2 * t + 1] = perm(hi, lo, 0x07030602u);
    }
    if (__builtin_expect(!ok, 0)) {
        asm volatile("" ::: "memory");
#pragma unroll
        for (int i = 0; i < 16; ++i) d[i] = 0x7FC07FC0u;
    }
}

__host__ __device__ __forceinline__ void half_tile(int h, int& rg, int& c64) {
    if (h < 64) { rg = h & 3; c64 = h >> 2; }
    else if (h < 192) { const int k = (h - 64) >> 5, i = h & 31; rg = 4 + (k >> 1) * 4 + (i >> 3); c64 = (k & 1) * 8 + (i & 7); }
    else { rg = 12 + ((h - 192) >> 4); c64 = h & 15; }
}

__global__ __launch_bounds__(NT, 2) void fused_gemm_ar(Params p) {
    extern __shared__ __attribute__((aligned(16))) unsigned char smem[];
    LAS unsigned char* lds = (LAS unsigned char*)smem;
    const int tid = threadIdx.x, lane = tid & 63, wv = (int)rfl(tid >> 6);
    const int b = blockIdx.x, rank = p.rank, ri = rank >> 2, rj = rank & 3;
    char* ws = p.ws[rank];
    char* tmo = ws + OFF_TMO;
    const u64 t_start = rtc();
    const u32 callv = ld_agent(ws + OFF_CTR + b * 4);

    {
        u32 d[16];
        const int isw = b >> 7, g = (b & 127) >> 2, q = b & 3;
        if (!isw) {
            const u16* src = p.x + (size_t)(g * 128 + wv * 16 + (lane & 15)) * KS + q * 128 + (lane >> 4) * 8;
#pragma unroll
            for (int f = 0; f < 4; ++f) {
                const u32x4 v = *(const GAS u32x4*)(src + f * 32);
                d[4 * f] = v[0]; d[4 * f + 1] = v[1]; d[4 * f + 2] = v[2]; d[4 * f + 3] = v[3];
            }
        } else {
            const u16* src = p.w + (size_t)(q * 128 + (tid >> 4)) * NN + g * 128 + (tid & 15) * 8;
            u32x4 v[4];
#pragma unroll
            for (int i = 0; i < 4; ++i) v[i] = *(const GAS u32x4*)(src + (size_t)i * 32 * NN);
#pragma unroll
            for (int i = 0; i < 4; ++i) *(LAS u32x4*)(lds + ((tid >> 4) + 32 * i) * WPITCH + (tid & 15) * 16) = v[i];
            __syncthreads();
            const int a0 = (8 * (lane >> 4) + ((lane & 15) >> 2)) * WPITCH + (wv * 16 + 4 * (lane & 3)) * 2;
#pragma unroll
            for (int f = 0; f < 4; ++f)
#pragma unroll
                for (int h = 0; h < 2; ++h) {
                    const s16x4 r = __builtin_amdgcn_ds_read_tr16_b64_v4i16((LAS s16x4*)(lds + a0 + (f * 32 + h * 4) * WPITCH));
                    const u32x2 rr = __builtin_bit_cast(u32x2, r);
                    d[4 * f + 2 * h] = rr[0]; d[4 * f + 2 * h + 1] = rr[1];
                }
        }
        init_tables(lds + SCR_OFF + wv * SCR_WAVE, lane);
        Packed pk; u32 E[8];
        pack_sm(d, pk, E);
        const bool push = !isw && (g & 15) < NPUSH / 32;
        const u32 poff = (u32)(((g & 15) * 4 + q) * 8 + wv) * CH_BYTES;
        const rsrc_t pin = mk_rsrc(ws + OFF_PIN + (size_t)b * UNIT_BYTES, UNIT_BYTES);
        if (push) {
            for (int j = 0; j < 4; ++j) {
                const u32 dest = (u32)((g >> 4) * 4 + j);
                if (dest != (u32)rank && dest < (u32)NR) store_sm(pk, mk_rsrc(p.ws[dest] + OFF_PUSH + (size_t)rank * NPUSH * CH_BYTES, NPUSH * CH_BYTES), poff, lane);
            }
        } else store_sm(pk, pin, wv * CH_BYTES, lane);
        pack_exp(E, pk, mk_rsrc(ws + OFF_FBIN + (size_t)b * FB_UNIT, FB_UNIT), wv * FB_CH, lane, lds + SCR_OFF + wv * SCR_WAVE);
        if (push) {
            for (int j = 0; j < 4; ++j) {
                const u32 dest = (u32)((g >> 4) * 4 + j);
                if (dest != (u32)rank && dest < (u32)NR) store_exp(pk, mk_rsrc(p.ws[dest] + OFF_PUSH + (size_t)rank * NPUSH * CH_BYTES, NPUSH * CH_BYTES), poff, lane);
            }
        } else store_exp(pk, pin, wv * CH_BYTES, lane);
        const bool mine = isw ? ((g >> 3) == rj) : ((g >> 4) == ri);
        const int item = isw ? 16 + (g & 7) : (g & 15);
        if (mine) {
            const rsrc_t dst = mk_rsrc(ws + (isw ? OFF_WF + ((size_t)(g & 7) << 20) : OFF_XF + ((size_t)(g & 15) << 20)), 1u << 20);
#pragma unroll
            for (int f = 0; f < 4; ++f)
                __builtin_amdgcn_raw_buffer_store_b128((u32x4){d[4 * f], d[4 * f + 1], d[4 * f + 2], d[4 * f + 3]}, dst,
                                                       (u32)(((rank * 16 + q * 4 + f) * 8 + wv) * 1024 + lane * 16), 0, AUX_AGENT);
        }
        drain();
        __syncthreads();
        const u32 call0 = rfl(callv) + 1;
        if (tid < (isw ? 2 : 4)) {
            const u32 dest = isw ? (u32)(tid * 4 + (g >> 3)) : (u32)((g >> 4) * 4 + tid);
            if (dest < (u32)NR) {
                if (dest != (u32)rank) st_sys(p.ws[dest] + OFF_FIN + (size_t)(rank * 256 + b) * 128, call0);
                else __hip_atomic_fetch_add((GAS u32*)(ws + OFF_CNT + (item * 8 + rank) * 128), 8u, __ATOMIC_RELAXED, __HIP_MEMORY_SCOPE_AGENT);
            }
        }
        if (tid == 0) st_agent(ws + OFF_CTR + b * 4, call0);
    }
    const u32 call = rfl(callv) + 1;
    const int xcd = b & 7, slot = b >> 3;

    if (slot < GS) {
        const int h0 = xcd * GS + slot;
        LAS u32* okw = (LAS u32*)(lds + LDS_MISC);
        {
            const u32 i = (u32)h0 + (u32)wv * (8 * GS);
            if (i < 7u * NPUSH) {
                const u32 src = (u32)(rank + 1 + i % 7) & 7u, c = i / 7, uid = ri * 64 + (c >> 3), v = c & 7;
                const bool ok = wait_ge<true>(ws + OFF_FIN + (size_t)(src * 256 + uid) * 128, call, tmo, call);
                asm volatile("" ::: "memory");
                const rsrc_t pr = mk_rsrc(ws + OFF_PUSH + (size_t)(src * NPUSH + c) * CH_BYTES, CH_BYTES);
                const u32x4 s0 = __builtin_amdgcn_raw_buffer_load_b128(pr, lane * 16, 0, AUX_SYS);
                const u32x4 s1 = __builtin_amdgcn_raw_buffer_load_b128(pr, lane * 16 + 1024, 0, AUX_SYS);
                const u32x2 cp = __builtin_amdgcn_raw_buffer_load_b64(pr, lane * 8 + 2048, 0, AUX_SYS);
                const u32 lst = __builtin_amdgcn_raw_buffer_load_b32(pr, lane * 4 + 2560, 0, AUX_SYS);
                u32 d[16];
                init_tables(lds + wv * SCR_WAVE, lane);
                unpack_chunk(d, s0, s1, cp, lst, mk_rsrc(p.ws[src] + OFF_FBIN, 256 * FB_UNIT), (uid * 8 + v) * FB_CH, lane, ok, lds + wv * SCR_WAVE);
                const rsrc_t dst = mk_rsrc(ws + OFF_XF + ((size_t)(c >> 5) << 20), 1u << 20);
#pragma unroll
                for (int f = 0; f < 4; ++f)
                    __builtin_amdgcn_raw_buffer_store_b128((u32x4){d[4 * f], d[4 * f + 1], d[4 * f + 2], d[4 * f + 3]}, dst,
                                                           (u32)(((src * 16 + ((c >> 3) & 3) * 4 + f) * 8 + v) * 1024 + lane * 16), 0, AUX_AGENT);
                drain();
                if (lane == 0) __hip_atomic_fetch_add((GAS u32*)(ws + OFF_CNT + ((c >> 5) * 8 + src) * 128), 1u, __ATOMIC_RELAXED, __HIP_MEMORY_SCOPE_AGENT);
            }
        }
#define G_BAR() do { __builtin_amdgcn_s_barrier(); asm volatile("" ::: "memory"); } while (0)
        for (int round = 0; round < (h0 < 256 - 8 * GS ? 2 : 1); ++round) {
            int rg, c64;
            half_tile(h0 + round * 8 * GS, rg, c64);
            const int cg = c64 >> 1, hn = c64 & 1;
            if (wv == 0) {
                const u32 e = max(rg < 4 ? rg : rg + 8, 4 + cg);
                while (rtc() - t_start < GATE0 + GATE1 * e) __builtin_amdgcn_s_sleep(8);
                wait_all16(ws + OFF_CNT + (((lane & 8) ? 16 + cg : rg) * 8 + (lane & 7)) * 128, call * 32, tmo, call, lane);
            }
            asm volatile("" ::: "memory"); __builtin_amdgcn_s_barrier(); asm volatile("" ::: "memory");
            f32x4 acc[4][2];
#pragma unroll
            for (int m = 0; m < 4; ++m) { acc[m][0] = (f32x4){0.f, 0.f, 0.f, 0.f}; acc[m][1] = (f32x4){0.f, 0.f, 0.f, 0.f}; }
            const int wr = (wv >> 1) & 1, wc = wv & 1;
            if (wv >= 4) {
                const int lw = wv - 4;
                const char* xg = ws + OFF_XF + ((size_t)rg << 20) + lw * 4096 + lane * 16;
                const char* wg = ws + OFF_WF + ((size_t)cg << 20) + (lw >> 1) * 8192 + hn * 4096 + (lw & 1) * 2048 + lane * 16;
#define L_ISSUE(s, SL) do { const char* _x = xg + (size_t)(s) * 16384; const char* _w = wg + (size_t)(s) * 16384; \
                _Pragma("unroll") for (int i = 0; i < 4; ++i) __builtin_amdgcn_global_load_lds((const GAS u32*)(_x + i * 1024), (LAS u32*)(lds + (SL) * GSLOT + lw * 4096 + i * 1024), 16, 0, AUX_SYS); \
                _Pragma("unroll") for (int i = 0; i < 2; ++i) __builtin_amdgcn_global_load_lds((const GAS u32*)(_w + i * 1024), (LAS u32*)(lds + (SL) * GSLOT + 16384 + lw * 2048 + i * 1024), 16, 0, AUX_SYS); } while (0)
#define L_WAIT(n) asm volatile("s_waitcnt vmcnt(" #n ")" ::: "memory")
                L_ISSUE(0, 0); L_ISSUE(1, 1); L_ISSUE(2, 2); L_ISSUE(3, 3);
                L_WAIT(18); G_BAR();
                for (int s = 0; s < 60; s += 4) {
                    L_WAIT(12); G_BAR(); L_ISSUE(s + 4, 0);
                    L_WAIT(12); G_BAR(); L_ISSUE(s + 5, 1);
                    L_WAIT(12); G_BAR(); L_ISSUE(s + 6, 2);
                    L_WAIT(12); G_BAR(); L_ISSUE(s + 7, 3);
                }
                L_WAIT(12); G_BAR(); L_WAIT(6); G_BAR(); L_WAIT(0); G_BAR(); G_BAR();
            } else {
                const int xro = wr * 4096 + lane * 16, wro = 16384 + wc * 2048 + lane * 16;
                bf16x8 xa[2][2][4], wb[2][2][2];
#define C_READ(P, SL) do { _Pragma("unroll") for (int k = 0; k < 2; ++k) { \
                _Pragma("unroll") for (int n = 0; n < 2; ++n) wb[P][k][n] = *(const LAS bf16x8*)(lds + (SL) * GSLOT + wro + k * 4096 + n * 1024); \
                _Pragma("unroll") for (int m = 0; m < 4; ++m) xa[P][k][m] = *(const LAS bf16x8*)(lds + (SL) * GSLOT + xro + k * 8192 + m * 1024); } } while (0)
#define C_MMA(P) do { _Pragma("unroll") for (int k = 0; k < 2; ++k) _Pragma("unroll") for (int m = 0; m < 4; ++m) \
                _Pragma("unroll") for (int n = 0; n < 2; ++n) acc[m][n] = __builtin_amdgcn_mfma_f32_16x16x32_bf16(wb[P][k][n], xa[P][k][m], acc[m][n], 0, 0, 0); } while (0)
#define C_SCHED() do { _Pragma("unroll") for (int i = 0; i < 12; ++i) { __builtin_amdgcn_sched_group_barrier(0x008, 1, 0); __builtin_amdgcn_sched_group_barrier(0x100, 1, 0); } \
                __builtin_amdgcn_sched_group_barrier(0x008, 4, 0); } while (0)
#define C_STEP(SL, P, RD) do { asm volatile("s_waitcnt lgkmcnt(0)" ::: "memory"); G_BAR(); if (RD) C_READ((P) ^ 1, ((SL) + 1) & 3); C_MMA(P); if (RD) C_SCHED(); __builtin_amdgcn_sched_barrier(0); } while (0)
                G_BAR();
                C_READ(0, 0);
                for (int s = 0; s < 60; s += 4) { C_STEP(0, 0, true); C_STEP(1, 1, true); C_STEP(2, 0, true); C_STEP(3, 1, true); }
                C_STEP(0, 0, true); C_STEP(1, 1, true); C_STEP(2, 0, true); C_STEP(3, 1, false);
            }
            G_BAR();
            if (wv < 4) {
#pragma unroll
                for (int m = 0; m < 4; ++m)
#pragma unroll
                    for (int n = 0; n < 2; ++n) {
                        u32x2 o;
                        asm("v_cvt_pk_bf16_f32 %0, %1, %2" : "=v"(o.x) : "v"(acc[m][n][0]), "v"(acc[m][n][1]));
                        asm("v_cvt_pk_bf16_f32 %0, %1, %2" : "=v"(o.y) : "v"(acc[m][n][2]), "v"(acc[m][n][3]));
                        *(LAS u32x2*)(lds + (wr * 64 + m * 16 + (lane & 15)) * OPITCH + (wc * 32 + n * 16 + (lane >> 4) * 4) * 2) = o;
                    }
            }
            if (tid == 0) okw[0] = (ld_agent(tmo) == call) ? 1u : 0u;
            __syncthreads();
            u32 d[16];
            const int j = wv & 3;
#pragma unroll
            for (int f = 0; f < 4; ++f) {
                const u32x4 v = *(const LAS u32x4*)(lds + (j * 32 + f * 8 + (lane >> 3)) * OPITCH + (lane & 7) * 16);
                d[4 * f] = v[0]; d[4 * f + 1] = v[1]; d[4 * f + 2] = v[2]; d[4 * f + 3] = v[3];
            }
            if (okw[0]) {
#pragma unroll
                for (int i = 0; i < 16; ++i) d[i] = 0x7FC07FC0u;
            }
            const int t = rg < 4 ? cg * 4 + rg : rg * 8 + cg;
            if (wv < 4) {
                init_tables(lds + SCR_OFF + wv * SCR_WAVE, lane);
                Packed pk; u32 E[8];
                const rsrc_t pout = mk_rsrc(ws + OFF_POUT + (size_t)t * UNIT_BYTES, UNIT_BYTES);
                pack_sm(d, pk, E);
                store_sm(pk, pout, (hn * 4 + j) * CH_BYTES, lane);
                pack_exp(E, pk, mk_rsrc(ws + OFF_FBOUT + (size_t)t * FB_UNIT, FB_UNIT), (hn * 4 + j) * FB_CH, lane, lds + SCR_OFF + wv * SCR_WAVE);
                store_exp(pk, pout, (hn * 4 + j) * CH_BYTES, lane);
                drain();
            } else {
                const rsrc_t outr = mk_rsrc(p.out, 4096u * NN * 2);
                const u32 oo = (u32)((ri * 2048 + rg * 128 + j * 32 + (lane >> 3)) * NN + rj * 1024 + c64 * 64 + (lane & 7) * 8) * 2;
#pragma unroll
                for (int f = 0; f < 4; ++f)
                    __builtin_amdgcn_raw_buffer_store_b128((u32x4){d[4 * f], d[4 * f + 1], d[4 * f + 2], d[4 * f + 3]}, outr, oo + f * 8 * NN * 2, 0, AUX_AGENT);
            }
            __syncthreads();
            if (tid < NR && tid != rank) st_sys(p.ws[tid] + OFF_FOUT + (size_t)(rank * 256 + t * 2 + hn) * 128, call);
        }
        return;
    }

    const int pi = (slot - GS) * 8 + xcd;
    {
        const int gw = pi * PWPW + wv;
        if (wv >= PWPW || gw >= 7 * NWL) return;
        const int link = gw % 7, wl = gw / 7;
        const u32 src = rfl((u32)(rank + 1 + link) & 7u);
        if (src >= (u32)NR) return;
        const u64 pws = uni64((u64)p.ws[src]), wsu = uni64((u64)ws);
        LAS unsigned char* scr = lds + wv * SCR_WAVE;
        init_tables(scr, lane);
        GAS u32* tick = (GAS u32*)(ws + OFF_TICK + link * 128 + (call & 3) * 32);
        if (wl == 0 && lane == 0) st_agent(ws + OFF_TICK + link * 128 + ((call + 2) & 3) * 32, 0u);
        const u32 tbase = NPUSH + 3 * NWL;
        struct Desc { u32 rem, fb, flg, dst, cnt; };
        const u32 xbase = ri * 64, wbase = 128 + rj * 32;
        const u32 obase = ((src >> 2) * 2048 * NN + (src & 3) * 1024) * 2;
        const u32 fin0 = (u32)OFF_FIN + src * 256 * 128, fout0 = (u32)OFF_FOUT + src * 256 * 128 - IN_UNITS * 256;
        const u32 cnt0 = (u32)OFF_CNT + src * 128, din0 = src * 128 * 1024;
        auto mk = [&](u32 c) -> Desc {
            c = min(c, (u32)NCH - 1);
            const u32 u = c >> 3, v = c & 7, e = u >> 2, m = u - IN_UNITS;
            u32 s4 = e >= 4 ? 4u : 0u, s12 = e >= 12 ? 4u : 0u, dx = e < 4 ? xbase : xbase - 32, dw = wbase - 16;
            u32 rgo = m < 32 ? (m & 3) : (m >> 3), cgo = m < 32 ? (m >> 2) : (m & 7);
            asm("" : "+s"(s4), "+s"(s12), "+s"(dx), "+s"(rgo), "+s"(cgo));
            const bool in = u < IN_UNITS, w = s4 != s12;
            const u32 pan = e - s4 - s12;
            u32 uidi = u + (w ? dw : dx), pbase = w ? (u32)OFF_WF : (u32)OFF_XF, cw = w ? 16u : 0u;
            asm("" : "+s"(uidi), "+s"(pbase), "+s"(cw));
            u32 remi = (u32)OFF_PIN + uidi * UNIT_BYTES, remo = (u32)OFF_POUT + m * UNIT_BYTES;
            u32 fbi = (u32)OFF_FBIN + uidi * FB_UNIT, fbo = (u32)OFF_FBOUT + m * FB_UNIT;
            u32 flgi = fin0 + uidi * 128, flgo = fout0 + u * 256 + (v >> 2) * 128;
            u32 dsti = pbase + (pan << 20) + din0 + ((u & 3) * 32 + v) * 1024;
            u32 dsto = obase + ((rgo * 128 + (v & 3) * 32) * NN + cgo * 128 + (v >> 2) * 64) * 2;
            u32 cnti = cnt0 + (cw + pan) * 1024;
            asm("" : "+s"(remi), "+s"(remo), "+s"(fbi), "+s"(fbo), "+s"(flgi), "+s"(flgo), "+s"(dsti), "+s"(dsto), "+s"(cnti));
            Desc d;
            d.rem = (in ? remi : remo) + v * CH_BYTES;
            d.fb = (in ? fbi : fbo) + v * FB_CH;
            d.flg = in ? flgi : flgo;
            d.dst = in ? dsti : dsto;
            d.cnt = in ? cnti : 0u;
            return d;
        };
        const rsrc_t wsr = mk_rsrc(ws, (u32)WS_NEED), outr = mk_rsrc(p.out, 4096u * NN * 2), pwr = mk_rsrc((const void*)pws, (u32)WS_NEED);
        const u32 vin = lane * 16, vout = (lane >> 3) * (NN * 2) + (lane & 7) * 16, vl4 = 2560 + lane * 4;
        u32 n1 = NPUSH + NWL + wl, n2 = NPUSH + 2 * NWL + wl;
        Desc D0 = mk(NPUSH + wl), D1 = mk(n1), D2 = mk(n2);
#define P_CLOB "a0", "a1", "a2", "a3", "a4", "a5", "a6", "a7", "a8", "a9", "a10", "a11"
#define P_ISSUE(DA) do { \
            const u64 un_ = pws + (DA).rem; \
            asm volatile("s_nop 4\n\tglobal_load_dwordx4 a[0:3], %0, %2 sc0 sc1\n\tglobal_load_dwordx4 a[4:7], %0, %2 offset:1024 sc0 sc1\n\t" \
                         "global_load_dwordx2 a[8:9], %1, %2 offset:2048 sc0 sc1\n\tglobal_load_dword a11, %3, %2 sc0 sc1" \
                         :: "v"(vin), "v"(lane * 8), "s"(un_), "v"(vl4) : P_CLOB, "memory"); } while (0)
#define P_FLAG(DB) do { const u64 fa_ = wsu + (DB).flg; \
            asm volatile("s_nop 4\n\tglobal_load_dword a12, %0, %1 sc0 sc1" :: "v"(0), "s"(fa_) : "a12", "memory"); } while (0)
#define P_TICKET(want_) do { u64 sv_; const u32 em_ = rfl((want_) ? 1u : 0u); \
            asm volatile("v_accvgpr_write_b32 a14, 1\n\ts_mov_b64 %0, exec\n\ts_mov_b32 exec_lo, %2\n\ts_mov_b32 exec_hi, 0\n\ts_nop 1\n\t" \
                         "global_atomic_add a13, %1, a14, off sc0\n\ts_mov_b64 exec, %0" \
                         : "=&s"(sv_) : "v"(tick), "s"(em_) : "a13", "a14", "memory"); } while (0)
#define P_FETCH() asm volatile("v_accvgpr_read_b32 %0, a0\n\tv_accvgpr_read_b32 %1, a1\n\tv_accvgpr_read_b32 %2, a2\n\tv_accvgpr_read_b32 %3, a3\n\t" \
                         "v_accvgpr_read_b32 %4, a4\n\tv_accvgpr_read_b32 %5, a5\n\tv_accvgpr_read_b32 %6, a6\n\tv_accvgpr_read_b32 %7, a7\n\t" \
                         "v_accvgpr_read_b32 %8, a8\n\tv_accvgpr_read_b32 %9, a9\n\tv_accvgpr_read_b32 %10, a11\n\t" \
                         "v_accvgpr_read_b32 %11, a12\n\tv_accvgpr_read_b32 %12, a13" \
                         : "=v"(c0.x), "=v"(c0.y), "=v"(c0.z), "=v"(c0.w), "=v"(c1.x), "=v"(c1.y), "=v"(c1.z), "=v"(c1.w), \
                           "=v"(cc.x), "=v"(cc.y), "=v"(cl), "=v"(fl), "=v"(tk) :: "memory")
#define P_SIGNAL(c_) do { u64 sv_; const u64 ca_ = wsu + (c_); const u32 em_ = rfl((c_) ? 1u : 0u); \
            asm volatile("s_nop 4\n\ts_mov_b64 %0, exec\n\ts_mov_b32 exec_lo, %4\n\ts_mov_b32 exec_hi, 0\n\tglobal_atomic_add %1, %2, %3\n\ts_mov_b64 exec, %0" \
                         : "=&s"(sv_) : "v"(0), "v"(1u), "s"(ca_), "s"(em_) : "memory"); } while (0)
        bool ok = wait_ge<true>(ws + D0.flg, call, tmo, call);
        asm volatile("" ::: "memory");
        P_ISSUE(D0); P_FLAG(D1);
        bool tkv = false;
        drain();
        u32 pendA = 0, pendB = 0;
        for (;;) {
            asm volatile("s_waitcnt vmcnt(4)" ::: "memory");
            u32x4 c0, c1; u32x2 cc; u32 cl, fl, tk;
            P_FETCH();
            const bool cok = ok;
            const bool more = n1 < (u32)NCH;
            const bool early = more && rfl(fl) >= call;
            if (__builtin_expect(early, 1)) { ok = true; asm volatile("" ::: "memory"); P_ISSUE(D1); }
            n2 = tkv ? rfl(tk) + tbase : n2;
            D2 = mk(n2);
            if (__builtin_expect(early, 1)) { P_FLAG(D2); tkv = n2 < (u32)NCH; P_TICKET(tkv); }
            P_SIGNAL(pendA);
            pendA = pendB;
            u32 d[16];
            unpack_chunk(d, c0, c1, cc, cl, pwr, D0.fb, lane, cok, scr);

            {
                const bool isin = D0.cnt != 0;
                const rsrc_t dr = isin ? wsr : outr;
                const u32 vo = isin ? vin : vout, step = isin ? 8192u : 8u * NN * 2;
#pragma unroll
                for (int f = 0; f < 4; ++f)
                    __builtin_amdgcn_raw_buffer_store_b128((u32x4){d[4 * f], d[4 * f + 1], d[4 * f + 2], d[4 * f + 3]}, dr, vo, D0.dst + f * step, AUX_AGENT);
            }
            u32 mine = D0.cnt;
            if (__builtin_expect(!early, 0)) {
                drain(); P_SIGNAL(pendA); P_SIGNAL(mine);
                pendA = 0; mine = 0;
                if (!more) break;
                ok = wait_ge<true>(ws + D1.flg, call, tmo, call);
                asm volatile("" ::: "memory");
                P_ISSUE(D1); P_FLAG(D2); tkv = n2 < (u32)NCH; P_TICKET(tkv);
                drain();
            }
            pendB = mine;
            n1 = n2; if (!tkv) n2 = NCH;
            D0 = D1; D1 = D2;
        }
        drain();
    }
}
}

#define HIP_CHECK(expr) do { hipError_t _e = (expr); if (_e != hipSuccess) { \
    fprintf(stderr, "HIP error %s at %s:%d\n", hipGetErrorString(_e), __FILE__, __LINE__); \
    return; } } while (0)

extern "C" void kernel_launch(int rank, int nranks, ncclComm_t comm,
                              hipStream_t stream,
                              void* const* d_in, const int* in_sizes, int n_in,
                              void* d_out, int out_size,
                              const int* in_dtypes, int out_dtype,
                              void* const* d_out_all,
                              void* const* d_ws_all, size_t ws_bytes) {
    (void)comm; (void)in_sizes; (void)n_in; (void)out_size; (void)in_dtypes; (void)out_dtype; (void)d_out_all;
    if (nranks != NR || ws_bytes < WS_NEED || rank < 0 || rank >= NR) { fprintf(stderr, "unexpected configuration\n"); return; }
    static bool attr_done[64] = {};
    int dev = 0;
    HIP_CHECK(hipGetDevice(&dev));
    if (dev >= 0 && dev < 64 && !attr_done[dev]) {
        HIP_CHECK(hipFuncSetAttribute((const void*)fused_gemm_ar, hipFuncAttributeMaxDynamicSharedMemorySize, LDS_BYTES));
        attr_done[dev] = true;
    }
    Params p;
    p.x = (const u16*)d_in[0];
    p.w = (const u16*)d_in[1];
    p.out = (u16*)d_out;
    for (int i = 0; i < NR; ++i) p.ws[i] = (char*)d_ws_all[i];
    p.rank = rank; p.pad = 0;
    hipLaunchKernelGGL(fused_gemm_ar, dim3(GRID), dim3(NT), LDS_BYTES, stream, p);
    HIP_CHECK(hipGetLastError());
}
```
